```python
import numpy as np

kernel_name = "gpu_gemm_ar_m4096_k4096_n32768_bf16_gelu"


ABSMAX_RTOL = 0.008

COMPUTE_DTYPE = "bfloat16"
EPILOGUE = "gelu"
IN_QUANT = None
OUT_QUANT = None
IN_DTYPES = {"x": "bfloat16", "w_mat": "bfloat16"}
WORLD_SIZE = 8

def _to_bf16(x):
    u = np.ascontiguousarray(x, dtype=np.float32).view(np.uint32)
    return ((u + 0x7FFF + ((u >> 16) & 1)) & 0xFFFF0000).view(np.float32)

def setup_inputs():

    x = _to_bf16(np.random.randn(4096, 4096).astype(np.float32))
    w_mat = _to_bf16((np.random.randn(4096, 32768) / np.sqrt(4096)).astype(np.float32))
    return {"x": x, "w_mat": w_mat}

def _epilogue(y):
    c = 0.7978845608028654
    return (0.5 * y * (1.0 + np.tanh(c * (y + 0.044715 * y ** 3)))).astype(np.float32)

def reference(x, w_mat):
    y = (x @ w_mat).astype(np.float32)
    return _epilogue(y)
```

```cpp
#include <hip/hip_runtime.h>
#include <rccl/rccl.h>
#include <cstdio>

#define HIP_CHECK(expr) do { hipError_t _e = (expr); if (_e != hipSuccess) { \
    fprintf(stderr, "HIP error %s at %s:%d\n", hipGetErrorString(_e), __FILE__, __LINE__); \
    return; } } while (0)

namespace gk {

typedef unsigned u32x4 __attribute__((ext_vector_type(4)));
typedef unsigned u32x3 __attribute__((ext_vector_type(3)));
typedef float f32x4 __attribute__((ext_vector_type(4)));
typedef float f32x2 __attribute__((ext_vector_type(2)));
typedef __bf16 bf16x2v __attribute__((ext_vector_type(2)));
typedef short bf16x8 __attribute__((ext_vector_type(8)));
typedef short s16x2 __attribute__((ext_vector_type(2)));
typedef unsigned short u16x2 __attribute__((ext_vector_type(2)));
typedef __amdgpu_buffer_rsrc_t rsrc_t;
#define LAS __attribute__((address_space(3)))

constexpr int NR = 8;
constexpr int MM = 4096, KK = 4096, NN = 32768, KS = 512, NS = 4096;
constexpr int NTHR = 512, GRID = 256;
#define PULL_PER_PEER 8
#define PULL_WAVES 4
constexpr int NPP = PULL_PER_PEER;
constexpr int NPULL = NPP * 7;
constexpr int NGEMM = GRID - NPULL;
constexpr int GSLOTS = NGEMM / 8;
static_assert(NPULL % 8 == 0, "pullers must be a multiple of 8");
constexpr int IN_UNITS = 64, OUT_UNITS = 256, UNITS = IN_UNITS + OUT_UNITS;
constexpr int ENC_JOBS = 16 * 18;
#define ENC_STAGGER 100
constexpr int LDS_BYTES = 131072 + 8192;

constexpr unsigned SLOT = 17408;
constexpr unsigned PK_OFF = 1024, SUBB = 2816;
constexpr unsigned PATCH0 = 4, PATCH_CAP = 252;

constexpr size_t OFF_TMO = 256, OFF_XCNT = 4096, OFF_WCNT = 8192, OFF_TICKET = 16384;
constexpr size_t OFF_FLAGS = 65536;
constexpr size_t OFF_DIAG = 262144;
constexpr size_t OFF_XF = 1u << 20;
constexpr size_t OFF_WT = OFF_XF + (size_t)MM * KK * 2;
constexpr size_t OFF_PK = OFF_WT + (size_t)NS * KK * 2;
constexpr size_t XP_BYTES = (size_t)32 * 8 * SLOT;
constexpr size_t WP_BYTES = (size_t)NR * 32 * 8 * SLOT;
constexpr size_t OP_BYTES = (size_t)OUT_UNITS * 8 * SLOT;
constexpr size_t PAR_BYTES = XP_BYTES + WP_BYTES + OP_BYTES;
constexpr size_t WS_USED = OFF_PK + 2 * PAR_BYTES;
constexpr size_t OUT_BYTES = (size_t)MM * NN * 2;
static_assert(WS_USED < (1ull << 31), "32-bit offsets");
constexpr unsigned long long DEADLINE = 100000000ull;

constexpr int AUX_SYS = 17;

struct Params {
    const void* x; const void* w; void* out;
    char* ws[NR];
    int rank; int pad;
};

__device__ __forceinline__ rsrc_t mkrsrc(const void* p, unsigned bytes) {
    return __builtin_amdgcn_make_buffer_rsrc((void*)p, 0, bytes, 0x00020000);
}
__device__ __forceinline__ unsigned pk_add16(unsigned a, unsigned b) {
    return __builtin_bit_cast(unsigned, (u16x2)(__builtin_bit_cast(u16x2, a) + __builtin_bit_cast(u16x2, b)));
}
__device__ __forceinline__ unsigned pk_sub16(unsigned a, unsigned b) {
    return __builtin_bit_cast(unsigned, (u16x2)(__builtin_bit_cast(u16x2, a) - __builtin_bit_cast(u16x2, b)));
}
__device__ __forceinline__ unsigned perm(unsigned s0, unsigned s1, unsigned sel) { return __builtin_amdgcn_perm(s0, s1, sel); }
__device__ __forceinline__ int uni(int v) { return __builtin_amdgcn_readfirstlane(v); }
#define WAIT_VM0() asm volatile("s_waitcnt vmcnt(0)" ::: "memory")
#define CBAR() asm volatile("" ::: "memory")

__device__ __forceinline__ void shell44(int i, int& a, int& b) {
    a = (int)((0x3210333210221010ull >> (4 * i)) & 15);
    b = (int)((0x3333210222101100ull >> (4 * i)) & 15);
}
__device__ __forceinline__ void unit_tile(int m, int& tm, int& tn) {
    int bi, bj, i, j; shell44(m >> 4, bi, bj); shell44(m & 15, i, j);
    tm = bi * 4 + i; tn = bj * 4 + j;
}

__device__ __forceinline__ bool give_up(unsigned* tmo, unsigned call, unsigned long long t0) {
    if (__hip_atomic_load(tmo, __ATOMIC_RELAXED, __HIP_MEMORY_SCOPE_AGENT) == call) return true;
    if (__builtin_amdgcn_s_memrealtime() - t0 > DEADLINE) {
        __hip_atomic_store(tmo, call, __ATOMIC_RELAXED, __HIP_MEMORY_SCOPE_AGENT);
        return true;
    }
    return false;
}

__device__ __forceinline__ int robust_top(int e) {
    int t = 0;
#pragma unroll
    for (int bit = 128; bit; bit >>= 1) {
        const int c = t | bit;
        if (__builtin_popcountll(__builtin_amdgcn_ballot_w64(e >= c)) >= 4) t = c;
    }
    return t;
}

__device__ __forceinline__ unsigned img_off(unsigned r, unsigned l) { const unsigned row = 2 * r + (l >> 5); return row * 512 + (((l & 31) ^ (row & 7)) << 4); }

template <bool IS_OUT>
__device__ __forceinline__ void encode_wb(const u32x4 (&d)[16], rsrc_t dst, unsigned slot_off, const LAS unsigned char* img, LAS unsigned* list, int lane) {
    s16x2 pm = __builtin_bit_cast(s16x2, d[0].x); u16x2 nm = __builtin_bit_cast(u16x2, d[0].x);
#pragma unroll
    for (int r = 0; r < 4; ++r)
#pragma unroll
        for (int j = 0; j < 4; ++j) {
            pm = __builtin_elementwise_max(pm, __builtin_bit_cast(s16x2, d[r][j]));
            nm = __builtin_elementwise_max(nm, __builtin_bit_cast(u16x2, d[r][j]));
        }
    const int p16 = pm.x > pm.y ? pm.x : pm.y;
    const unsigned n16 = nm.x > nm.y ? nm.x : nm.y;
    const int pe = p16 > 0 ? (p16 >> 7) & 255 : 0;
    const int ne = (n16 & 0x8000u) ? (int)(n16 >> 7) & 255 : 0;
    const int tp = robust_top(pe), tn = robust_top(ne);
    int off, pb, nb; unsigned T0, T1, nb4;
    if (IS_OUT) {
        off = (tn & 1) ^ 1;
        pb = ((tp + off) >> 1) - 4; nb = ((tn + off) >> 1) - 2;
        pb = pb < 0 ? 0 : (pb > 123 ? 123 : pb); nb = nb < 5 ? 5 : (nb > 125 ? 125 : nb);
        T0 = (unsigned)pb * 0x01010101u + 0x03020100u;
        T1 = (unsigned)(pb + 4) | ((unsigned)(0x80 | nb) << 8) | ((unsigned)(0x80 | (nb + 1)) << 16) | ((unsigned)(0x80 | (nb + 2)) << 24);
        nb4 = (unsigned)(nb - 5) * 0x01010101u;
    } else {
        const int t = tp > tn ? tp : tn;
        off = (t & 1) ^ 1;
        pb = ((t + off) >> 1) - 3; nb = pb;
        pb = pb < 0 ? 0 : (pb > 124 ? 124 : pb); nb = nb < 4 ? 4 : (nb > 124 ? 124 : nb);
        T0 = (unsigned)pb * 0x01010101u + 0x03020100u;
        T1 = ((unsigned)nb * 0x01010101u + 0x03020100u) | 0x80808080u;
        nb4 = (unsigned)(nb - 4) * 0x01010101u;
    }
    const unsigned pb4 = (unsigned)pb * 0x01010101u;
    const unsigned koff = off ? 0x00800080u : 0u;
    unsigned cnt = 0;
#pragma unroll
    for (int s = 0; s < 4; ++s) {
        unsigned c[8]; unsigned L[8]; unsigned mk = 0;
#pragma unroll
        for (int g = 0; g < 8; ++g) {
            const int r = 4 * s + (g >> 1), h = g & 1;
            const unsigned e0 = pk_add16(d[r][2 * h], koff), e1 = pk_add16(d[r][2 * h + 1], koff);
            L[g] = perm(e1, e0, 0x06040200u);
            const unsigned H = perm(e1, e0, 0x07050301u);
            const unsigned M = perm(e1, e0, 0x0b0a0908u);
            const unsigned base = (M & nb4) | (~M & pb4);
            const unsigned cc = ((H | 0x80808080u) - base) & 0x07070707u;
            c[g] = cc;
            const unsigned bad = perm(T1, T0, cc) ^ H;
            const unsigned nz = ((bad & 0x7f7f7f7fu) + 0x7f7f7f7fu) | bad;
            mk |= (nz >> (7 - g)) & (0x01010101u << g);
        }
        u32x3 P;
        P.x = c[0] | (c[1] << 3) | ((c[2] << 6) & 0xc0c0c0c0u);
        P.y = ((c[2] >> 2) & 0x01010101u) | (c[3] << 1) | (c[4] << 4) | ((c[5] << 7) & 0x80808080u);
        P.z = ((c[5] >> 1) & 0x03030303u) | (c[6] << 2) | (c[7] << 5);
        const unsigned o = slot_off + PK_OFF + s * SUBB;
        __builtin_amdgcn_raw_buffer_store_b128((u32x4){L[0], L[1], L[2], L[3]}, dst, o + lane * 16, 0, AUX_SYS);
        __builtin_amdgcn_raw_buffer_store_b128((u32x4){L[4], L[5], L[6], L[7]}, dst, o + 1024 + lane * 16, 0, AUX_SYS);
        __builtin_amdgcn_raw_buffer_store_b96(P, dst, o + 2048 + lane * 12, 0, AUX_SYS);
        unsigned long long m = __builtin_amdgcn_ballot_w64(mk != 0);
        while (m) {
            const unsigned pos = cnt + __builtin_amdgcn_mbcnt_hi((unsigned)(m >> 32), __builtin_amdgcn_mbcnt_lo((unsigned)m, 0));
            if (mk != 0) {
                const unsigned bit = (unsigned)__builtin_ctz(mk), g = bit & 7, j = bit >> 3;
                const unsigned r = 4 * s + (g >> 1), jj = 4 * (g & 1) + j;
                const unsigned val = *(const LAS unsigned short*)(img + img_off(r, (unsigned)lane) + jj * 2);
                if (pos < PATCH_CAP) list[PATCH0 + pos] = ((unsigned)lane << 23) | (r << 19) | (jj << 16) | val;
                mk &= mk - 1;
            }
            cnt += (unsigned)__builtin_popcountll(m);
            m = __builtin_amdgcn_ballot_w64(mk != 0);
        }
    }
    const bool raw = cnt > PATCH_CAP;
    if (raw) {
        WAIT_VM0();
#pragma unroll
        for (int r = 0; r < 16; ++r) __builtin_amdgcn_raw_buffer_store_b128(d[r], dst, slot_off + PK_OFF + r * 1024 + lane * 16, 0, AUX_SYS);
        cnt = 0;
    }
    if (lane == 0) { list[0] = cnt | ((unsigned)off << 16) | (raw ? 0x80000000u : 0u); list[1] = T0; list[2] = T1; list[3] = 0; }
    for (unsigned b = 0; b < cnt + PATCH0; b += 64) {
        const unsigned e = list[b + lane];
        __builtin_amdgcn_raw_buffer_store_b32(e, dst, slot_off + (b + lane) * 4, 0, AUX_SYS);
    }
}

struct Item { rsrc_t dr; unsigned so, base, sh, A, B, s1, s2; unsigned* cnt; };
struct Sub { u32x4 La, Lb; u32x3 P; };

__device__ __forceinline__ void load_sub(Sub& v, rsrc_t src, unsigned so, int s, int lane) {
    const unsigned o = so + PK_OFF + s * SUBB;
    v.La = __builtin_amdgcn_raw_buffer_load_b128(src, o + lane * 16, 0, AUX_SYS);
    v.Lb = __builtin_amdgcn_raw_buffer_load_b128(src, o + 1024 + lane * 16, 0, AUX_SYS);
    v.P = __builtin_amdgcn_raw_buffer_load_b96(src, o + 2048 + lane * 12, 0, AUX_SYS);
}

__device__ __forceinline__ void unpack_sub(const Sub& v, int s, unsigned T0, unsigned T1, unsigned koff, unsigned cnt, const unsigned (&e)[4],
                                           const Item& it, unsigned lanepart, LAS unsigned char* stg, int lane) {
    const u32x3 p = v.P;
    unsigned c[8];
    c[0] = p.x & 0x07070707u;
    c[1] = (p.x >> 3) & 0x07070707u;
    c[2] = ((p.x >> 6) & 0x03030303u) | ((p.y & 0x01010101u) << 2);
    c[3] = (p.y >> 1) & 0x07070707u;
    c[4] = (p.y >> 4) & 0x07070707u;
    c[5] = ((p.y >> 7) & 0x01010101u) | ((p.z & 0x03030303u) << 1);
    c[6] = (p.z >> 2) & 0x07070707u;
    c[7] = (p.z >> 5) & 0x07070707u;
#pragma unroll
    for (int rr = 0; rr < 4; ++rr) {
        const unsigned L0 = rr < 2 ? v.La[2 * rr] : v.Lb[2 * rr - 4], L1 = rr < 2 ? v.La[2 * rr + 1] : v.Lb[2 * rr - 3];
        const unsigned H0 = perm(T1, T0, c[2 * rr]), H1 = perm(T1, T0, c[2 * rr + 1]);
        u32x4 o;
        o.x = pk_sub16(perm(H0, L0, 0x05010400u), koff);
        o.y = pk_sub16(perm(H0, L0, 0x07030602u), koff);
        o.z = pk_sub16(perm(H1, L1, 0x05010400u), koff);
        o.w = pk_sub16(perm(H1, L1, 0x07030602u), koff);
        *(LAS u32x4*)(stg + rr * 1024 + lane * 16) = o;
    }
    CBAR();
#pragma unroll
    for (int q = 0; q < 4; ++q) {
        if (cnt + PATCH0 > (unsigned)(q * 64)) {
            const unsigned idx = q * 64 + lane, x = e[q];
            if (idx >= PATCH0 && idx < cnt + PATCH0 && ((x >> 21) & 3u) == (unsigned)s)
                *(LAS unsigned short*)(stg + ((x >> 19) & 3u) * 1024 + (x >> 23) * 16 + ((x >> 16) & 7u) * 2) = (unsigned short)x;
        }
    }
    CBAR();
#pragma unroll
    for (int rr = 0; rr < 4; ++rr) {
        const int r = 4 * s + rr;
        const u32x4 o = *(const LAS u32x4*)(stg + rr * 1024 + lane * 16);
        __builtin_amdgcn_raw_buffer_store_b128(o, it.dr, lanepart + (r & 7) * it.s1 + (r >> 3) * it.s2, 0, AUX_SYS);
    }
}

constexpr int BM = 256, BK = 64, HALF = 128, HTB = HALF * BK * 2, STAGE_BYTES = 8 * HTB;
__device__ __forceinline__ int lds_byte(int r, int c) { const int st = (r >> 4) * 2 + (c >> 5), rr = r & 15, cc = c & 31, ob = rr * 64 + cc * 2; return st * 1024 + (ob ^ (((ob >> 9) & 1) << 5)); }
__device__ __forceinline__ void stage_rc(int b, int& R, int& C) { const int st = b / 1024, sb = b % 1024, swz = sb ^ (((sb >> 9) & 1) << 5); R = (st >> 1) * 16 + swz / 64; C = (st & 1) * 32 + (swz % 64) / 2; }
__device__ __forceinline__ int perm32(int rho) { const int n = rho >> 4, i = rho & 15; return 8 * (i >> 2) + 4 * n + (i & 3); }

__device__ __forceinline__ float gelu(float y) {
    const float c0 = -2.3022081985f, c1 = -0.1029432397f;
    const float z = y * __builtin_fmaf(y * y, c1, c0);
    return y * __builtin_amdgcn_rcpf(1.0f + __builtin_amdgcn_exp2f(z));
}
__device__ __forceinline__ unsigned cvt2(float a, float b) {
    return __builtin_bit_cast(unsigned, __builtin_convertvector((f32x2){a, b}, bf16x2v));
}

__device__ __forceinline__ void gemm_tile(LAS unsigned char* lds, const char* Abase, const char* Bbase, f32x4 (&acc)[2][2][4][2]) {
    const int tid = threadIdx.x, wid = uni(tid >> 6), lane = tid & 63, wr = wid >> 2, wc = wid & 3, fr = lane & 15, fq = lane >> 4;
    constexpr int K = KK, nt = K / BK;
    unsigned voffA[2], voffB[2];
#pragma unroll
    for (int i = 0; i < 2; ++i) { int R, C; stage_rc(tid * 16 + i * 8192, R, C); const int Rb = (R & ~31) + perm32(R & 31);
        voffA[i] = (unsigned)(R * K + C) * 2u; voffB[i] = (unsigned)(Rb * K + C) * 2u; }
    constexpr size_t kstep = (size_t)(BK * 2), hstep = (size_t)HALF * K * 2;
    const unsigned ldsw = (unsigned)wid * 1024u;
    const int aoff = lds_byte(wr * 64 + fr, fq * 8), boff = lds_byte(wc * 32 + fr, fq * 8);
#define G_SA(b, h) (((b) * 2 + (h)) * HTB)
#define G_SB(b, h) ((4 + (b) * 2 + (h)) * HTB)
#define G_STAGE(bufoff, gbase, voff) do { _Pragma("unroll") for (int _i = 0; _i < 2; ++_i) \
        __builtin_amdgcn_global_load_lds((const unsigned*)((const char*)(gbase) + (voff)[_i]), (LAS unsigned*)(lds + (bufoff) + ldsw + _i * 8192), 16, 0, 0); } while (0)
#define G_LDA(dst, b, h) do { _Pragma("unroll") for (int m = 0; m < 4; ++m) _Pragma("unroll") for (int k = 0; k < 2; ++k) dst[m][k] = *(const LAS bf16x8*)(lds + G_SA(b, h) + aoff + m * 2048 + k * 1024); } while (0)
#define G_LDB(dst, b, h) do { _Pragma("unroll") for (int n = 0; n < 2; ++n) _Pragma("unroll") for (int k = 0; k < 2; ++k) dst[n][k] = *(const LAS bf16x8*)(lds + G_SB(b, h) + boff + n * 2048 + k * 1024); } while (0)
#define G_MMA(ai, bj, At, Bt) do { __builtin_amdgcn_s_setprio(1); _Pragma("unroll") for (int m = 0; m < 4; ++m) _Pragma("unroll") for (int n = 0; n < 2; ++n) _Pragma("unroll") for (int k = 0; k < 2; ++k) \
        acc[ai][bj][m][n] = __builtin_amdgcn_mfma_f32_16x16x32_bf16(Bt[n][k], At[m][k], acc[ai][bj][m][n], 0, 0, 0); __builtin_amdgcn_s_setprio(0); } while (0)
#define G_WAIT_V(n) asm volatile("s_waitcnt vmcnt(" #n ")" ::: "memory")
#define G_WAIT_L(n) asm volatile("s_waitcnt lgkmcnt(" #n ")" ::: "memory")
#define G_BAR __builtin_amdgcn_s_barrier()
#define G_SCHED __builtin_amdgcn_sched_barrier(0)
#pragma unroll
    for (int a = 0; a < 2; ++a)
#pragma unroll
        for (int b = 0; b < 2; ++b)
#pragma unroll
            for (int m = 0; m < 4; ++m)
#pragma unroll
                for (int n = 0; n < 2; ++n) acc[a][b][m][n] = (f32x4){0.f, 0.f, 0.f, 0.f};
    bf16x8 At[4][2], B0[2][2], B1[2][2];
    const char* cA = Abase; const char* cB = Bbase;
    G_STAGE(G_SB(0, 0), cB, voffB); G_STAGE(G_SA(0, 0), cA, voffA); G_STAGE(G_SB(0, 1), cB + hstep, voffB); G_STAGE(G_SA(0, 1), cA + hstep, voffA);
    if (wr == 1) G_BAR;
    G_WAIT_V(4); G_BAR;
    G_STAGE(G_SB(1, 0), cB + kstep, voffB); G_STAGE(G_SA(1, 0), cA + kstep, voffA); G_STAGE(G_SB(1, 1), cB + hstep + kstep, voffB);
    G_WAIT_V(6); G_BAR;
    for (int t = 0; t < nt; t += 2) {
        const bool last = (t == nt - 2);
        const char* a1 = cA + (size_t)(t + 1) * kstep;
        const char* a2 = last ? cA : cA + (size_t)(t + 2) * kstep; const char* b2 = last ? cB : cB + (size_t)(t + 2) * kstep;
        const char* a3 = a2 + kstep; const char* b3 = b2 + kstep;
        G_LDB(B0, 0, 0); G_SCHED; G_LDA(At, 0, 0); G_STAGE(G_SA(1, 1), a1 + hstep, voffA);
        G_WAIT_L(8); G_BAR; G_WAIT_L(0); G_MMA(0, 0, At, B0); G_BAR; G_SCHED;
        G_LDB(B1, 0, 1); G_STAGE(G_SB(0, 0), b2, voffB);
        G_BAR; G_WAIT_L(0); G_MMA(0, 1, At, B1); G_BAR;
        G_LDA(At, 0, 1); G_STAGE(G_SA(0, 0), a2, voffA);
        G_BAR; G_WAIT_L(0); G_MMA(1, 0, At, B0); G_BAR; G_SCHED;
        G_STAGE(G_SB(0, 1), b2 + hstep, voffB);
        G_WAIT_V(6); G_BAR; G_MMA(1, 1, At, B1); G_BAR;
        G_LDB(B0, 1, 0); G_SCHED; G_LDA(At, 1, 0); G_STAGE(G_SA(0, 1), a2 + hstep, voffA);
        G_WAIT_L(8); G_BAR; G_WAIT_L(0); G_MMA(0, 0, At, B0); G_BAR; G_SCHED;
        G_LDB(B1, 1, 1); G_STAGE(G_SB(1, 0), b3, voffB);
        G_BAR; G_WAIT_L(0); G_MMA(0, 1, At, B1); G_BAR;
        G_LDA(At, 1, 1); G_STAGE(G_SA(1, 0), a3, voffA);
        G_BAR; G_WAIT_L(0); G_MMA(1, 0, At, B0); G_BAR; G_SCHED;
        G_STAGE(G_SB(1, 1), b3 + hstep, voffB);
        G_WAIT_V(6); G_BAR; G_MMA(1, 1, At, B1); G_BAR;
    }
    G_WAIT_V(0);
    if (wr == 0) G_BAR;
    G_BAR;
#undef G_SA
#undef G_SB
#undef G_STAGE
#undef G_LDA
#undef G_LDB
#undef G_MMA
}

#define STAMP(p, idx) do { } while (0)
#define STAMPW(p, idx) do { } while (0)

__global__ __launch_bounds__(NTHR, 2) void fused_kernel(Params p) {
    extern __shared__ __attribute__((aligned(16))) unsigned char shm[];
    LAS unsigned char* lds = (LAS unsigned char*)shm;
    const int tid = threadIdx.x, wid = uni(tid >> 6), lane = tid & 63;
    const int rank = p.rank;
    char* const wsl = p.ws[rank];
    unsigned* const tmo = (unsigned*)(wsl + OFF_TMO);
    unsigned* const my_ctr = (unsigned*)(wsl + OFF_TICKET + (size_t)blockIdx.x * 128);
    const unsigned call = (unsigned)uni((int)__hip_atomic_load(my_ctr, __ATOMIC_RELAXED, __HIP_MEMORY_SCOPE_AGENT)) + 1u;
    __syncthreads();
    if (tid == 0) __hip_atomic_store(my_ctr, call, __ATOMIC_RELAXED, __HIP_MEMORY_SCOPE_AGENT);
    const unsigned par = call & 1u;
    const unsigned pk_base = (unsigned)(OFF_PK + par * PAR_BYTES);
    const unsigned xp_off = pk_base, wp_off = pk_base + (unsigned)XP_BYTES, op_off = wp_off + (unsigned)WP_BYTES;
    const unsigned flag_row = (unsigned)(OFF_FLAGS + ((size_t)par * NR) * UNITS * 4);
    const int b = blockIdx.x;
    STAMP(p, 0);

    if (b < NPULL) {
        if (wid >= PULL_WAVES) return;
        const int pi = b % 7, sub = b / 7;
        const int src = (rank + 1 + pi) & 7;
        const rsrc_t rs = mkrsrc(p.ws[src], (unsigned)WS_USED);
        const unsigned* fl = (const unsigned*)(wsl + flag_row + (size_t)src * UNITS * 4);
        LAS unsigned char* stg = lds + wid * 4096;
        constexpr int WPU = 8 / PULL_WAVES;
        constexpr int NK = (UNITS + NPP - 1) / NPP;
        const int nitems = ((UNITS - sub + NPP - 1) / NPP) * WPU;
        int known = 0;
        bool dead = false;
        auto make_item = [&](int i, Item& it) {
            const int k = i / WPU, w = wid + (i % WPU) * PULL_WAVES, u = sub + k * NPP;
            if (u < IN_UNITS) {
                const int c = u >> 2, h = u & 1;
                it.dr = mkrsrc(wsl + OFF_XF, (unsigned)(OFF_PK - OFF_XF));
                it.s1 = 8192;
                if ((u & 2) == 0) {
                    it.so = xp_off + (unsigned)((c * 2 + h) * 8 + w) * SLOT;
                    it.base = (unsigned)(((256 * c + 128 * h + 16 * w) * KK + KS * src) * 2);
                    it.sh = 6; it.A = 0; it.B = 16; it.s2 = 65536;
                    it.cnt = (unsigned*)(wsl + OFF_XCNT + c * 128);
                } else {
                    it.so = wp_off + (unsigned)((rank * 32 + c * 2 + h) * 8 + w) * SLOT;
                    it.base = (unsigned)(MM * KK * 2) + (unsigned)(((256 * c + 64 * (2 * h + (w >> 2))) * KK + KS * src + 128 * (w & 3)) * 2);
                    it.sh = 3; it.A = 16; it.B = 65536; it.s2 = 128;
                    it.cnt = (unsigned*)(wsl + OFF_WCNT + c * 128);
                }
            } else {
                const int m = u - IN_UNITS; int tm, tn; unit_tile(m, tm, tn);
                it.so = op_off + (unsigned)(m * 8 + w) * SLOT;
                it.dr = mkrsrc(p.out, (unsigned)OUT_BYTES);
                it.base = (unsigned)(((size_t)(256 * tm + 32 * w) * NN + NS * src + 256 * tn) * 2);
                it.sh = 5; it.A = 65536; it.B = 16; it.s1 = 131072; it.s2 = 1048576;
                it.cnt = nullptr;
            }
        };
        auto poll = [&](int k) {
            const int uu = sub + (k + (lane & 15)) * NPP;
            const unsigned f = uu < UNITS ? __hip_atomic_load(fl + uu, __ATOMIC_RELAXED, __HIP_MEMORY_SCOPE_SYSTEM) : 0u;
            const unsigned long long mk = __builtin_amdgcn_ballot_w64(f >= call) & 0xffffull;
            known = k + __builtin_ctzll(~mk);
            __builtin_amdgcn_fence(__ATOMIC_ACQUIRE, "wavefront");
            CBAR();
        };
        unsigned* pending = nullptr;
        Item it; Sub v; unsigned e[4]; unsigned en0 = 0;
        bool have_next = false;
        bool started = false;
        unsigned hdr = 0, T0 = 0, T1 = 0;
        auto start_item = [&](const Item& x, unsigned e0v) {
            e[0] = e0v;
            hdr = (unsigned)__builtin_amdgcn_readlane((int)e0v, 0); T0 = (unsigned)__builtin_amdgcn_readlane((int)e0v, 1); T1 = (unsigned)__builtin_amdgcn_readlane((int)e0v, 2);
            const unsigned c = hdr & 0xffffu;
            e[1] = e[2] = e[3] = 0;
            if (c + PATCH0 > 64) e[1] = __builtin_amdgcn_raw_buffer_load_b32(rs, x.so + 256 + lane * 4, 0, AUX_SYS);
            if (c + PATCH0 > 128) { e[2] = __builtin_amdgcn_raw_buffer_load_b32(rs, x.so + 512 + lane * 4, 0, AUX_SYS);
                                    e[3] = __builtin_amdgcn_raw_buffer_load_b32(rs, x.so + 768 + lane * 4, 0, AUX_SYS); }
            load_sub(v, rs, x.so, 0, lane);
        };
        for (int i = 0; i < nitems; ++i) {
            if (!started) {
                const int k = i / WPU;
                if (k >= known && !dead) {
                    if (pending) { WAIT_VM0(); if (lane == 0) __hip_atomic_fetch_add(pending, 1u, __ATOMIC_RELAXED, __HIP_MEMORY_SCOPE_AGENT); pending = nullptr; }
                    const unsigned long long t0 = __builtin_amdgcn_s_memrealtime();
                    for (;;) {
                        poll(k);
                        if (known > k) break;
                        if (__builtin_amdgcn_readfirstlane((int)give_up(tmo, call, t0))) { dead = true; break; }
                        __builtin_amdgcn_s_sleep(4);
                    }
                }
                make_item(i, it);
                if (dead) {
                    if (it.cnt == nullptr) {
                        const unsigned lp = it.base + ((unsigned)lane >> it.sh) * it.A + ((unsigned)lane & ((1u << it.sh) - 1u)) * it.B;
                        for (int r = 0; r < 16; ++r)
                            __builtin_amdgcn_raw_buffer_store_b128((u32x4){0x7fc07fc0u, 0x7fc07fc0u, 0x7fc07fc0u, 0x7fc07fc0u}, it.dr, lp + (r & 7) * it.s1 + (r >> 3) * it.s2, 0, 0);
                    } else if (lane == 0) __hip_atomic_fetch_add(it.cnt, 1u, __ATOMIC_RELAXED, __HIP_MEMORY_SCOPE_AGENT);
                    continue;
                }
                if (!have_next) en0 = __builtin_amdgcn_raw_buffer_load_b32(rs, it.so + lane * 4, 0, AUX_SYS);
                start_item(it, en0);
                have_next = false;
            }
            started = false;
            const unsigned lanepart = it.base + ((unsigned)lane >> it.sh) * it.A + ((unsigned)lane & ((1u << it.sh) - 1u)) * it.B;
            const unsigned chdr = hdr, cT0 = T0, cT1 = T1;
            const unsigned koff = (chdr & 0x10000u) ? 0x00800080u : 0u, cnt = chdr & 0xffffu;
            Item nx; bool nx_ok = false;
            if (i + 1 < nitems) {
                const int kn = (i + 1) / WPU;
                if (kn >= known) poll(kn);
                if (kn < known) { make_item(i + 1, nx); en0 = __builtin_amdgcn_raw_buffer_load_b32(rs, nx.so + lane * 4, 0, AUX_SYS); nx_ok = true; }
            }
            if (chdr & 0x80000000u) {
                if (pending) { WAIT_VM0(); if (lane == 0) __hip_atomic_fetch_add(pending, 1u, __ATOMIC_RELAXED, __HIP_MEMORY_SCOPE_AGENT); pending = nullptr; }
#pragma unroll 4
                for (int r = 0; r < 16; ++r) {
                    const u32x4 x = __builtin_amdgcn_raw_buffer_load_b128(rs, it.so + PK_OFF + r * 1024 + lane * 16, 0, AUX_SYS);
                    __builtin_amdgcn_raw_buffer_store_b128(x, it.dr, lanepart + (r & 7) * it.s1 + (r >> 3) * it.s2, 0, AUX_SYS);
                }
                pending = it.cnt;
                have_next = nx_ok;
                continue;
            }
            unsigned ec[4] = {e[0], e[1], e[2], e[3]};
            Sub cur = v;
            CBAR(); load_sub(v, rs, it.so, 1, lane); CBAR();
            unpack_sub(cur, 0, cT0, cT1, koff, cnt, ec, it, lanepart, stg, lane);
            cur = v;
            CBAR(); load_sub(v, rs, it.so, 2, lane); CBAR();
            if (pending) {
                asm volatile("s_waitcnt vmcnt(10)" ::: "memory");
                if (lane == 0) __hip_atomic_fetch_add(pending, 1u, __ATOMIC_RELAXED, __HIP_MEMORY_SCOPE_AGENT);
                pending = nullptr;
            }
            unpack_sub(cur, 1, cT0, cT1, koff, cnt, ec, it, lanepart, stg, lane);
            cur = v;
            CBAR(); load_sub(v, rs, it.so, 3, lane); CBAR();
            unpack_sub(cur, 2, cT0, cT1, koff, cnt, ec, it, lanepart, stg, lane);
            cur = v;
            CBAR();
            if (nx_ok) { start_item(nx, en0); started = true; }
            CBAR();
            unpack_sub(cur, 3, cT0, cT1, koff, cnt, ec, it, lanepart, stg, lane);
            CBAR();
            pending = it.cnt;
            if (nx_ok) it = nx;
        }
        if (pending) { WAIT_VM0(); if (lane == 0) __hip_atomic_fetch_add(pending, 1u, __ATOMIC_RELAXED, __HIP_MEMORY_SCOPE_AGENT); }
        return;
    }

    const int g = b - NPULL;
    {
        const rsrc_t rws = mkrsrc(wsl, (unsigned)WS_USED);
        LAS unsigned char* img = lds + wid * 16384;
        LAS unsigned* list = (LAS unsigned*)(lds + STAGE_BYTES + wid * 1024);
        const unsigned long long tstart = __builtin_amdgcn_s_memrealtime();
        for (int rnd = 0; rnd < 2; ++rnd) {
            const int job = rnd == 0 ? g : NGEMM + (NGEMM - 1 - g);
            if (job >= ENC_JOBS) break;
            const int c = job / 18, q = job % 18;
            if (rnd == 0) { while (__builtin_amdgcn_s_memrealtime() - tstart < (unsigned long long)(c * ENC_STAGGER)) __builtin_amdgcn_s_sleep(2); }
            u32x4 d[16];
            if (q < 2) {
                const int h = q;
                const rsrc_t rx = mkrsrc(p.x, (unsigned)(MM * KS * 2));
                const unsigned row0 = 256 * c + 128 * h + 16 * wid;
#pragma unroll
                for (int r = 0; r < 16; ++r) d[r] = __builtin_amdgcn_raw_buffer_load_b128(rx, (row0 + r) * (KS * 2) + lane * 16, 0, 0);
#pragma unroll
                for (int r = 0; r < 16; ++r) *(LAS u32x4*)(img + img_off(r, lane)) = d[r];
                CBAR();
                encode_wb<false>(d, rws, xp_off + (unsigned)((c * 2 + h) * 8 + wid) * SLOT, img, list, lane);
#pragma unroll
                for (int r = 0; r < 16; ++r)
                    __builtin_amdgcn_raw_buffer_store_b128(d[r], rws, (unsigned)OFF_XF + ((row0 + r) * KK + KS * rank) * 2 + lane * 16, 0, AUX_SYS);
                WAIT_VM0();
                if (lane == 0) __hip_atomic_fetch_add((unsigned*)(wsl + OFF_XCNT + c * 128), 1u, __ATOMIC_RELAXED, __HIP_MEMORY_SCOPE_AGENT);
                __syncthreads();
                if (tid < NR && tid != rank)
                    __hip_atomic_store((unsigned*)(p.ws[tid] + flag_row + (size_t)rank * UNITS * 4) + (c * 4 + h), call, __ATOMIC_RELAXED, __HIP_MEMORY_SCOPE_SYSTEM);
            } else {
                const int h = q & 1, dest = (rank + 1 + ((q - 2) >> 1)) & 7;
                const rsrc_t rw = mkrsrc(p.w, (unsigned)((size_t)KS * NN * 2));
                const int ng = 2 * h + (wid >> 2), kq = wid & 3;
                const unsigned k0 = 128 * kq + 8 * (lane >> 3), n0 = NS * dest + 256 * c + 64 * ng + 8 * (lane & 7);
                u32x4 in[16];
#pragma unroll
                for (int i = 0; i < 16; ++i) in[i] = __builtin_amdgcn_raw_buffer_load_b128(rw, ((k0 + 64 * (i >> 3) + (i & 7)) * NN + n0) * 2, 0, 0);
#pragma unroll
                for (int bh = 0; bh < 2; ++bh)
#pragma unroll
                    for (int c8 = 0; c8 < 8; ++c8)
#pragma unroll
                        for (int kk = 0; kk < 4; ++kk)
                            d[bh * 8 + c8][kk] = perm(in[bh * 8 + 2 * kk + 1][c8 >> 1], in[bh * 8 + 2 * kk][c8 >> 1], (c8 & 1) ? 0x07060302u : 0x05040100u);
                if (dest == rank) {
                    const unsigned base = (unsigned)OFF_WT + ((256 * c + 64 * ng + 8 * (lane & 7)) * KK + KS * rank + k0) * 2;
#pragma unroll
                    for (int r = 0; r < 16; ++r) __builtin_amdgcn_raw_buffer_store_b128(d[r], rws, base + (r & 7) * 8192 + (r >> 3) * 128, 0, AUX_SYS);
                    WAIT_VM0();
                    if (lane == 0) __hip_atomic_fetch_add((unsigned*)(wsl + OFF_WCNT + c * 128), 1u, __ATOMIC_RELAXED, __HIP_MEMORY_SCOPE_AGENT);
                } else {
#pragma unroll
                    for (int r = 0; r < 16; ++r) *(LAS u32x4*)(img + img_off(r, lane)) = d[r];
                    CBAR();
                    encode_wb<false>(d, rws, wp_off + (unsigned)((dest * 32 + c * 2 + h) * 8 + wid) * SLOT, img, list, lane);
                    WAIT_VM0();
                    __syncthreads();
                    if (tid == 0)
                        __hip_atomic_store((unsigned*)(p.ws[dest] + flag_row + (size_t)rank * UNITS * 4) + (c * 4 + 2 + h), call, __ATOMIC_RELAXED, __HIP_MEMORY_SCOPE_SYSTEM);
                }
            }
            STAMP(p, 8 + rnd);
        }
    }
    STAMP(p, 1);

    const int xl = b & 7, slot = g >> 3;
    for (int pos = slot; pos < 32; pos += GSLOTS) {
        const int m = (pos < 16) ? xl * 16 + pos : 128 + xl * 16 + (pos - 16);
        int tm, tn; unit_tile(m, tm, tn);
        __syncthreads();
        bool dead = false;
        if (wid == 0) {
            const unsigned* xc = (const unsigned*)(wsl + OFF_XCNT + tm * 128); const unsigned* wc_ = (const unsigned*)(wsl + OFF_WCNT + tn * 128);
            const unsigned want = call * 128u;
            const unsigned long long t0 = __builtin_amdgcn_s_memrealtime();
            for (;;) {
                const unsigned v = __hip_atomic_load(lane & 1 ? wc_ : xc, __ATOMIC_RELAXED, __HIP_MEMORY_SCOPE_AGENT);
                if (__builtin_amdgcn_ballot_w64(v < want) == 0ull) break;
                if (__builtin_amdgcn_readfirstlane((int)give_up(tmo, call, t0))) break;
                __builtin_amdgcn_s_sleep(8);
            }
            __builtin_amdgcn_fence(__ATOMIC_ACQUIRE, "agent");
            WAIT_VM0();
        }
        CBAR(); __builtin_amdgcn_s_barrier(); CBAR();
        STAMP(p, pos < GSLOTS ? 2 : 5);
        f32x4 acc[2][2][4][2];
        gemm_tile(lds, wsl + OFF_XF + (size_t)tm * 256 * KK * 2, wsl + OFF_WT + (size_t)tn * 256 * KK * 2, acc);
        STAMP(p, pos < GSLOTS ? 3 : 6);
        dead = __hip_atomic_load(tmo, __ATOMIC_RELAXED, __HIP_MEMORY_SCOPE_AGENT) == call;
        {
            int tz = threadIdx.x; asm volatile("" : "+v"(tz));
            const int ln = tz & 63, wr = wid >> 2, wc = wid & 3, fr = ln & 15, fq = ln >> 4;
#pragma unroll
            for (int ai = 0; ai < 2; ++ai)
#pragma unroll
                for (int mm = 0; mm < 4; ++mm)
#pragma unroll
                    for (int bj = 0; bj < 2; ++bj) {
                        const f32x4 v0 = acc[ai][bj][mm][0], v1 = acc[ai][bj][mm][1];
                        u32x4 w4;
                        w4.x = cvt2(gelu(v0[0]), gelu(v0[1])); w4.y = cvt2(gelu(v0[2]), gelu(v0[3]));
                        w4.z = cvt2(gelu(v1[0]), gelu(v1[1])); w4.w = cvt2(gelu(v1[2]), gelu(v1[3]));
                        const int row = ai * 128 + wr * 64 + mm * 16 + fr, ch = bj * 16 + wc * 4 + fq;
                        *(LAS u32x4*)(lds + row * 512 + ((ch ^ (row & 7)) << 4)) = w4;
                    }
            __syncthreads();
            u32x4 d[16];
#pragma unroll
            for (int r = 0; r < 16; ++r) {
                const int row = wid * 32 + 2 * r + (ln >> 5), ch = ln & 31;
                d[r] = *(const LAS u32x4*)(lds + row * 512 + ((ch ^ (row & 7)) << 4));
            }
            if (dead) {
#pragma unroll
                for (int r = 0; r < 16; ++r) d[r] = (u32x4){0x7fc07fc0u, 0x7fc07fc0u, 0x7fc07fc0u, 0x7fc07fc0u};
            }
            const rsrc_t ro = mkrsrc(p.out, (unsigned)OUT_BYTES);
            const unsigned ob = (unsigned)(((size_t)(256 * tm + 32 * wid + (ln >> 5)) * NN + NS * rank + 256 * tn + 8 * (ln & 31)) * 2);
#pragma unroll
            for (int r = 0; r < 16; ++r) __builtin_amdgcn_raw_buffer_store_b128(d[r], ro, ob + r * (2 * NN * 2), 0, 0);
            const rsrc_t rws = mkrsrc(wsl, (unsigned)WS_USED);
            encode_wb<true>(d, rws, op_off + (unsigned)(m * 8 + wid) * SLOT, lds + wid * 16384, (LAS unsigned*)(lds + STAGE_BYTES + wid * 1024), ln);
            WAIT_VM0();
            __syncthreads();
            if (tid < NR && tid != rank)
                __hip_atomic_store((unsigned*)(p.ws[tid] + flag_row + (size_t)rank * UNITS * 4) + (IN_UNITS + m), call, __ATOMIC_RELAXED, __HIP_MEMORY_SCOPE_SYSTEM);
        }
        STAMP(p, pos < GSLOTS ? 4 : 7);
    }
}

}

extern "C" void kernel_launch(int rank, int nranks, ncclComm_t comm, hipStream_t stream,
                              void* const* d_in, const int* in_sizes, int n_in,
                              void* d_out, int out_size, const int* in_dtypes, int out_dtype,
                              void* const* d_out_all, void* const* d_ws_all, size_t ws_bytes) {
    (void)comm; (void)in_sizes; (void)n_in; (void)out_size; (void)in_dtypes; (void)out_dtype; (void)d_out_all;
    if (nranks != gk::NR || rank < 0 || rank >= gk::NR || ws_bytes < gk::WS_USED) { fprintf(stderr, "unexpected configuration\n"); return; }
    static bool attr_done[64] = {};
    int dev = 0; HIP_CHECK(hipGetDevice(&dev));
    if (dev >= 0 && dev < 64 && !attr_done[dev]) {
        HIP_CHECK(hipFuncSetAttribute((const void*)gk::fused_kernel, hipFuncAttributeMaxDynamicSharedMemorySize, gk::LDS_BYTES));
        attr_done[dev] = true;
    }
    gk::Params p;
    p.x = d_in[0]; p.w = d_in[1]; p.out = d_out;
    for (int i = 0; i < gk::NR; ++i) p.ws[i] = (char*)d_ws_all[i];
    p.rank = rank; p.pad = 0;
    hipLaunchKernelGGL(gk::fused_kernel, dim3(gk::GRID), dim3(gk::NTHR), gk::LDS_BYTES, stream, p);
    HIP_CHECK(hipGetLastError());
}
```

```cpp
#include <hip/hip_runtime.h>
#include <rccl/rccl.h>
#include <cstdio>

#define HIP_CHECK(expr) do { hipError_t _e = (expr); if (_e != hipSuccess) { \
    fprintf(stderr, "HIP error %s at %s:%d\n", hipGetErrorString(_e), __FILE__, __LINE__); \
    return; } } while (0)

namespace gk {

typedef unsigned u32x4 __attribute__((ext_vector_type(4)));
typedef unsigned u32x3 __attribute__((ext_vector_type(3)));
typedef float f32x4 __attribute__((ext_vector_type(4)));
typedef float f32x2 __attribute__((ext_vector_type(2)));
typedef __bf16 bf16x2v __attribute__((ext_vector_type(2)));
typedef short bf16x8 __attribute__((ext_vector_type(8)));
typedef short s16x2 __attribute__((ext_vector_type(2)));
typedef unsigned short u16x2 __attribute__((ext_vector_type(2)));
typedef __amdgpu_buffer_rsrc_t rsrc_t;
#define LAS __attribute__((address_space(3)))

constexpr int NR = 8;
constexpr int MM = 4096, KK = 4096, NN = 32768, KS = 512, NS = 4096;
constexpr int NTHR = 512, GRID = 256;
#define PULL_PER_PEER 8
#define PULL_WAVES 4
constexpr int NPP = PULL_PER_PEER;
constexpr int NPULL = NPP * 7;
constexpr int NGEMM = GRID - NPULL;
constexpr int GSLOTS = NGEMM / 8;
static_assert(NPULL % 8 == 0, "pullers must be a multiple of 8");
constexpr int IN_UNITS = 64, OUT_UNITS = 256, UNITS = IN_UNITS + OUT_UNITS;
constexpr int ENC_JOBS = 16 * 18;
#define PACE_LEAD 70
#define SYNC_FROM 2
#define PEND_LATE 1
#define AUX_LOC 17
#define ENC_STAGGER 100
constexpr int LDS_BYTES = 131072 + 8192;

constexpr unsigned SLOT = 17408;
constexpr unsigned PK_OFF = 1024, SUBB = 2816;
constexpr unsigned PATCH0 = 4, PATCH_CAP = 252;

constexpr size_t OFF_TMO = 256, OFF_XCNT = 4096, OFF_WCNT = 8192, OFF_PTICK = 12288, OFF_TICKET = 16384;
constexpr size_t OFF_FLAGS = 65536;
constexpr size_t OFF_DIAG = 262144;
constexpr size_t OFF_XF = 1u << 20;
constexpr size_t OFF_WT = OFF_XF + (size_t)MM * KK * 2;
constexpr size_t OFF_PK = OFF_WT + (size_t)NS * KK * 2;
constexpr size_t XP_BYTES = (size_t)32 * 8 * SLOT;
constexpr size_t WP_BYTES = (size_t)NR * 32 * 8 * SLOT;
constexpr size_t OP_BYTES = (size_t)OUT_UNITS * 8 * SLOT;
constexpr size_t PAR_BYTES = XP_BYTES + WP_BYTES + OP_BYTES;
constexpr size_t WS_USED = OFF_PK + 2 * PAR_BYTES;
constexpr size_t OUT_BYTES = (size_t)MM * NN * 2;
static_assert(WS_USED < (1ull << 31), "32-bit offsets");
constexpr unsigned long long DEADLINE = 100000000ull;

constexpr int AUX_SYS = 17;

struct Params {
    const void* x; const void* w; void* out;
    char* ws[NR];
    int rank; int pad;
};

__device__ __forceinline__ rsrc_t mkrsrc(const void* p, unsigned bytes) {
    return __builtin_amdgcn_make_buffer_rsrc((void*)p, 0, bytes, 0x00020000);
}
__device__ __forceinline__ unsigned pk_add16(unsigned a, unsigned b) {
    return __builtin_bit_cast(unsigned, (u16x2)(__builtin_bit_cast(u16x2, a) + __builtin_bit_cast(u16x2, b)));
}
__device__ __forceinline__ unsigned pk_sub16(unsigned a, unsigned b) {
    return __builtin_bit_cast(unsigned, (u16x2)(__builtin_bit_cast(u16x2, a) - __builtin_bit_cast(u16x2, b)));
}
__device__ __forceinline__ unsigned perm(unsigned s0, unsigned s1, unsigned sel) { return __builtin_amdgcn_perm(s0, s1, sel); }
__device__ __forceinline__ int uni(int v) { return __builtin_amdgcn_readfirstlane(v); }
#define WAIT_VM0() asm volatile("s_waitcnt vmcnt(0)" ::: "memory")
#define CBAR() asm volatile("" ::: "memory")

__device__ __forceinline__ void shell44(int i, int& a, int& b) {
    a = (int)((0x3210333210221010ull >> (4 * i)) & 15);
    b = (int)((0x3333210222101100ull >> (4 * i)) & 15);
}
__device__ __forceinline__ void unit_tile(int m, int& tm, int& tn) {
    int bi, bj, i, j; shell44(m >> 4, bi, bj); shell44(m & 15, i, j);
    tm = bi * 4 + i; tn = bj * 4 + j;
}

__device__ __forceinline__ bool give_up(unsigned* tmo, unsigned call, unsigned long long t0) {
    if (__hip_atomic_load(tmo, __ATOMIC_RELAXED, __HIP_MEMORY_SCOPE_AGENT) == call) return true;
    if (__builtin_amdgcn_s_memrealtime() - t0 > DEADLINE) {
        __hip_atomic_store(tmo, call, __ATOMIC_RELAXED, __HIP_MEMORY_SCOPE_AGENT);
        return true;
    }
    return false;
}

__device__ __forceinline__ int robust_top(int e) {
    int t = 0;
#pragma unroll
    for (int bit = 128; bit; bit >>= 1) {
        const int c = t | bit;
        if (__builtin_popcountll(__builtin_amdgcn_ballot_w64(e >= c)) >= 4) t = c;
    }
    return t;
}

__device__ __forceinline__ unsigned img_off(unsigned r, unsigned l) { const unsigned row = 2 * r + (l >> 5); return row * 512 + (((l & 31) ^ (row & 7)) << 4); }

template <bool IS_OUT>
__device__ __forceinline__ void encode_wb(const u32x4 (&d)[16], rsrc_t dst, unsigned slot_off, const LAS unsigned char* img, LAS unsigned* list, int lane) {
    s16x2 pm = __builtin_bit_cast(s16x2, d[0].x); u16x2 nm = __builtin_bit_cast(u16x2, d[0].x);
#pragma unroll
    for (int r = 0; r < 4; ++r)
#pragma unroll
        for (int j = 0; j < 4; ++j) {
            pm = __builtin_elementwise_max(pm, __builtin_bit_cast(s16x2, d[r][j]));
            nm = __builtin_elementwise_max(nm, __builtin_bit_cast(u16x2, d[r][j]));
        }
    const int p16 = pm.x > pm.y ? pm.x : pm.y;
    const unsigned n16 = nm.x > nm.y ? nm.x : nm.y;
    const int pe = p16 > 0 ? (p16 >> 7) & 255 : 0;
    const int ne = (n16 & 0x8000u) ? (int)(n16 >> 7) & 255 : 0;
    const int tp = robust_top(pe), tn = robust_top(ne);
    int off, pb, nb; unsigned T0, T1, nb4;
    if (IS_OUT) {
        off = (tn & 1) ^ 1;
        pb = ((tp + off) >> 1) - 4; nb = ((tn + off) >> 1) - 2;
        pb = pb < 0 ? 0 : (pb > 123 ? 123 : pb); nb = nb < 5 ? 5 : (nb > 125 ? 125 : nb);
        T0 = (unsigned)pb * 0x01010101u + 0x03020100u;
        T1 = (unsigned)(pb + 4) | ((unsigned)(0x80 | nb) << 8) | ((unsigned)(0x80 | (nb + 1)) << 16) | ((unsigned)(0x80 | (nb + 2)) << 24);
        nb4 = (unsigned)(nb - 5) * 0x01010101u;
    } else {
        const int t = tp > tn ? tp : tn;
        off = (t & 1) ^ 1;
        pb = ((t + off) >> 1) - 3; nb = pb;
        pb = pb < 0 ? 0 : (pb > 124 ? 124 : pb); nb = nb < 4 ? 4 : (nb > 124 ? 124 : nb);
        T0 = (unsigned)pb * 0x01010101u + 0x03020100u;
        T1 = ((unsigned)nb * 0x01010101u + 0x03020100u) | 0x80808080u;
        nb4 = (unsigned)(nb - 4) * 0x01010101u;
    }
    const unsigned pb4 = (unsigned)pb * 0x01010101u;
    const unsigned koff = off ? 0x00800080u : 0u;
    unsigned cnt = 0;
#pragma unroll
    for (int s = 0; s < 4; ++s) {
        unsigned c[8]; unsigned L[8]; unsigned mk = 0;
#pragma unroll
        for (int g = 0; g < 8; ++g) {
            const int r = 4 * s + (g >> 1), h = g & 1;
            const unsigned e0 = pk_add16(d[r][2 * h], koff), e1 = pk_add16(d[r][2 * h + 1], koff);
            L[g] = perm(e1, e0, 0x06040200u);
            const unsigned H = perm(e1, e0, 0x07050301u);
            const unsigned M = perm(e1, e0, 0x0b0a0908u);
            const unsigned base = (M & nb4) | (~M & pb4);
            const unsigned cc = ((H | 0x80808080u) - base) & 0x07070707u;
            c[g] = cc;
            const unsigned bad = perm(T1, T0, cc) ^ H;
            const unsigned nz = ((bad & 0x7f7f7f7fu) + 0x7f7f7f7fu) | bad;
            mk |= (nz >> (7 - g)) & (0x01010101u << g);
        }
        u32x3 P;
        P.x = c[0] | (c[1] << 3) | ((c[2] << 6) & 0xc0c0c0c0u);
        P.y = ((c[2] >> 2) & 0x01010101u) | (c[3] << 1) | (c[4] << 4) | ((c[5] << 7) & 0x80808080u);
        P.z = ((c[5] >> 1) & 0x03030303u) | (c[6] << 2) | (c[7] << 5);
        const unsigned o = slot_off + PK_OFF + s * SUBB;
        __builtin_amdgcn_raw_buffer_store_b128((u32x4){L[0], L[1], L[2], L[3]}, dst, o + lane * 16, 0, AUX_SYS);
        __builtin_amdgcn_raw_buffer_store_b128((u32x4){L[4], L[5], L[6], L[7]}, dst, o + 1024 + lane * 16, 0, AUX_SYS);
        __builtin_amdgcn_raw_buffer_store_b96(P, dst, o + 2048 + lane * 12, 0, AUX_SYS);
        unsigned long long m = __builtin_amdgcn_ballot_w64(mk != 0);
        while (m) {
            const unsigned pos = cnt + __builtin_amdgcn_mbcnt_hi((unsigned)(m >> 32), __builtin_amdgcn_mbcnt_lo((unsigned)m, 0));
            if (mk != 0) {
                const unsigned bit = (unsigned)__builtin_ctz(mk), g = bit & 7, j = bit >> 3;
                const unsigned r = 4 * s + (g >> 1), jj = 4 * (g & 1) + j;
                if (pos < PATCH_CAP) list[PATCH0 + pos] = ((unsigned)lane << 23) | (r << 19) | (jj << 16);
                mk &= mk - 1;
            }
            cnt += (unsigned)__builtin_popcountll(m);
            m = __builtin_amdgcn_ballot_w64(mk != 0);
        }
    }
    const bool raw = cnt > PATCH_CAP;
    if (raw) {
        WAIT_VM0();
#pragma unroll
        for (int r = 0; r < 16; ++r) __builtin_amdgcn_raw_buffer_store_b128(d[r], dst, slot_off + PK_OFF + r * 1024 + lane * 16, 0, AUX_SYS);
        cnt = 0;
    }
    const unsigned hdr = cnt | ((unsigned)off << 16) | (raw ? 0x80000000u : 0u);
    for (unsigned b = 0; b < cnt + PATCH0; b += 64) {
        const unsigned idx = b + lane;
        unsigned e = list[idx];
        if (idx >= PATCH0) {
            if (idx < cnt + PATCH0) e |= *(const LAS unsigned short*)(img + img_off((e >> 19) & 15u, e >> 23) + ((e >> 16) & 7u) * 2);
        } else e = idx == 0 ? hdr : (idx == 1 ? T0 : (idx == 2 ? T1 : 0u));
        if (idx < cnt + PATCH0 || b == 0) __builtin_amdgcn_raw_buffer_store_b32(e, dst, slot_off + idx * 4, 0, AUX_SYS);
    }
}

struct Item { rsrc_t dr; unsigned so, base, sh, A, B, s1, s2; unsigned* cnt; };
struct Sub { u32x4 La, Lb; u32x3 P; };

__device__ __forceinline__ void load_sub(Sub& v, rsrc_t src, unsigned so, int s, int lane) {
    const unsigned o = so + PK_OFF + s * SUBB;
    v.La = __builtin_amdgcn_raw_buffer_load_b128(src, o + lane * 16, 0, AUX_SYS);
    v.Lb = __builtin_amdgcn_raw_buffer_load_b128(src, o + 1024 + lane * 16, 0, AUX_SYS);
    v.P = __builtin_amdgcn_raw_buffer_load_b96(src, o + 2048 + lane * 12, 0, AUX_SYS);
}

__device__ __forceinline__ void unpack_sub(const Sub& v, int s, unsigned T0, unsigned T1, unsigned koff, unsigned cnt, const unsigned (&e)[4],
                                           const Item& it, unsigned lanepart, LAS unsigned char* stg, int lane) {
    const u32x3 p = v.P;
    unsigned c[8];
    c[0] = p.x & 0x07070707u;
    c[1] = (p.x >> 3) & 0x07070707u;
    c[2] = ((p.x >> 6) & 0x03030303u) | ((p.y & 0x01010101u) << 2);
    c[3] = (p.y >> 1) & 0x07070707u;
    c[4] = (p.y >> 4) & 0x07070707u;
    c[5] = ((p.y >> 7) & 0x01010101u) | ((p.z & 0x03030303u) << 1);
    c[6] = (p.z >> 2) & 0x07070707u;
    c[7] = (p.z >> 5) & 0x07070707u;
#pragma unroll
    for (int rr = 0; rr < 4; ++rr) {
        const unsigned L0 = rr < 2 ? v.La[2 * rr] : v.Lb[2 * rr - 4], L1 = rr < 2 ? v.La[2 * rr + 1] : v.Lb[2 * rr - 3];
        const unsigned H0 = perm(T1, T0, c[2 * rr]), H1 = perm(T1, T0, c[2 * rr + 1]);
        u32x4 o;
        o.x = pk_sub16(perm(H0, L0, 0x05010400u), koff);
        o.y = pk_sub16(perm(H0, L0, 0x07030602u), koff);
        o.z = pk_sub16(perm(H1, L1, 0x05010400u), koff);
        o.w = pk_sub16(perm(H1, L1, 0x07030602u), koff);
        *(LAS u32x4*)(stg + rr * 1024 + lane * 16) = o;
    }
    CBAR();
#pragma unroll
    for (int q = 0; q < 4; ++q) {
        if (cnt + PATCH0 > (unsigned)(q * 64)) {
            const unsigned idx = q * 64 + lane, x = e[q];
            if (idx >= PATCH0 && idx < cnt + PATCH0 && ((x >> 21) & 3u) == (unsigned)s)
                *(LAS unsigned short*)(stg + ((x >> 19) & 3u) * 1024 + (x >> 23) * 16 + ((x >> 16) & 7u) * 2) = (unsigned short)x;
        }
    }
    CBAR();
#pragma unroll
    for (int rr = 0; rr < 4; ++rr) {
        const int r = 4 * s + rr;
        const u32x4 o = *(const LAS u32x4*)(stg + rr * 1024 + lane * 16);
        __builtin_amdgcn_raw_buffer_store_b128(o, it.dr, lanepart + (r & 7) * it.s1 + (r >> 3) * it.s2, 0, AUX_LOC);
    }
}

constexpr int BM = 256, BK = 64, HALF = 128, HTB = HALF * BK * 2, STAGE_BYTES = 8 * HTB;
__device__ __forceinline__ int lds_byte(int r, int c) { const int st = (r >> 4) * 2 + (c >> 5), rr = r & 15, cc = c & 31, ob = rr * 64 + cc * 2; return st * 1024 + (ob ^ (((ob >> 9) & 1) << 5)); }
__device__ __forceinline__ void stage_rc(int b, int& R, int& C) { const int st = b / 1024, sb = b % 1024, swz = sb ^ (((sb >> 9) & 1) << 5); R = (st >> 1) * 16 + swz / 64; C = (st & 1) * 32 + (swz % 64) / 2; }
__device__ __forceinline__ int perm32(int rho) { const int n = rho >> 4, i = rho & 15; return 8 * (i >> 2) + 4 * n + (i & 3); }

__device__ __forceinline__ float gelu(float y) {
    const float c0 = -2.3022081985f, c1 = -0.1029432397f;
    const float z = y * __builtin_fmaf(y * y, c1, c0);
    return y * __builtin_amdgcn_rcpf(1.0f + __builtin_amdgcn_exp2f(z));
}
__device__ __forceinline__ unsigned cvt2(float a, float b) {
    return __builtin_bit_cast(unsigned, __builtin_convertvector((f32x2){a, b}, bf16x2v));
}

__device__ __forceinline__ void gemm_tile(LAS unsigned char* lds, const char* Abase, const char* Bbase, f32x4 (&acc)[2][2][4][2]) {
    const int tid = threadIdx.x, wid = uni(tid >> 6), lane = tid & 63, wr = wid >> 2, wc = wid & 3, fr = lane & 15, fq = lane >> 4;
    constexpr int K = KK, nt = K / BK;
    unsigned voffA[2], voffB[2];
#pragma unroll
    for (int i = 0; i < 2; ++i) { int R, C; stage_rc(tid * 16 + i * 8192, R, C); const int Rb = (R & ~31) + perm32(R & 31);
        voffA[i] = (unsigned)(R * K + C) * 2u; voffB[i] = (unsigned)(Rb * K + C) * 2u; }
    constexpr size_t kstep = (size_t)(BK * 2), hstep = (size_t)HALF * K * 2;
    const unsigned ldsw = (unsigned)wid * 1024u;
    const int aoff = lds_byte(wr * 64 + fr, fq * 8), boff = lds_byte(wc * 32 + fr, fq * 8);
#define G_SA(b, h) (((b) * 2 + (h)) * HTB)
#define G_SB(b, h) ((4 + (b) * 2 + (h)) * HTB)
#define G_STAGE(bufoff, gbase, voff) do { _Pragma("unroll") for (int _i = 0; _i < 2; ++_i) \
        __builtin_amdgcn_global_load_lds((const unsigned*)((const char*)(gbase) + (voff)[_i]), (LAS unsigned*)(lds + (bufoff) + ldsw + _i * 8192), 16, 0, 0); } while (0)
#define G_LDA(dst, b, h) do { _Pragma("unroll") for (int m = 0; m < 4; ++m) _Pragma("unroll") for (int k = 0; k < 2; ++k) dst[m][k] = *(const LAS bf16x8*)(lds + G_SA(b, h) + aoff + m * 2048 + k * 1024); } while (0)
#define G_LDB(dst, b, h) do { _Pragma("unroll") for (int n = 0; n < 2; ++n) _Pragma("unroll") for (int k = 0; k < 2; ++k) dst[n][k] = *(const LAS bf16x8*)(lds + G_SB(b, h) + boff + n * 2048 + k * 1024); } while (0)
#define G_MMA(ai, bj, At, Bt) do { __builtin_amdgcn_s_setprio(1); _Pragma("unroll") for (int m = 0; m < 4; ++m) _Pragma("unroll") for (int n = 0; n < 2; ++n) _Pragma("unroll") for (int k = 0; k < 2; ++k) \
        acc[ai][bj][m][n] = __builtin_amdgcn_mfma_f32_16x16x32_bf16(Bt[n][k], At[m][k], acc[ai][bj][m][n], 0, 0, 0); __builtin_amdgcn_s_setprio(0); } while (0)
#define G_WAIT_V(n) asm volatile("s_waitcnt vmcnt(" #n ")" ::: "memory")
#define G_WAIT_L(n) asm volatile("s_waitcnt lgkmcnt(" #n ")" ::: "memory")
#define G_BAR __builtin_amdgcn_s_barrier()
#define G_SCHED __builtin_amdgcn_sched_barrier(0)
#pragma unroll
    for (int a = 0; a < 2; ++a)
#pragma unroll
        for (int b = 0; b < 2; ++b)
#pragma unroll
            for (int m = 0; m < 4; ++m)
#pragma unroll
                for (int n = 0; n < 2; ++n) acc[a][b][m][n] = (f32x4){0.f, 0.f, 0.f, 0.f};
    bf16x8 At[4][2], B0[2][2], B1[2][2];
    const char* cA = Abase; const char* cB = Bbase;
    G_STAGE(G_SB(0, 0), cB, voffB); G_STAGE(G_SA(0, 0), cA, voffA); G_STAGE(G_SB(0, 1), cB + hstep, voffB); G_STAGE(G_SA(0, 1), cA + hstep, voffA);
    if (wr == 1) G_BAR;
    G_WAIT_V(4); G_BAR;
    G_STAGE(G_SB(1, 0), cB + kstep, voffB); G_STAGE(G_SA(1, 0), cA + kstep, voffA); G_STAGE(G_SB(1, 1), cB + hstep + kstep, voffB);
    G_WAIT_V(6); G_BAR;
    for (int t = 0; t < nt; t += 2) {
        const bool last = (t == nt - 2);
        const char* a1 = cA + (size_t)(t + 1) * kstep;
        const char* a2 = last ? cA : cA + (size_t)(t + 2) * kstep; const char* b2 = last ? cB : cB + (size_t)(t + 2) * kstep;
        const char* a3 = a2 + kstep; const char* b3 = b2 + kstep;
        G_LDB(B0, 0, 0); G_SCHED; G_LDA(At, 0, 0); G_STAGE(G_SA(1, 1), a1 + hstep, voffA);
        G_WAIT_L(8); G_BAR; G_WAIT_L(0); G_MMA(0, 0, At, B0); G_BAR; G_SCHED;
        G_LDB(B1, 0, 1); G_STAGE(G_SB(0, 0), b2, voffB);
        G_BAR; G_WAIT_L(0); G_MMA(0, 1, At, B1); G_BAR;
        G_LDA(At, 0, 1); G_STAGE(G_SA(0, 0), a2, voffA);
        G_BAR; G_WAIT_L(0); G_MMA(1, 0, At, B0); G_BAR; G_SCHED;
        G_STAGE(G_SB(0, 1), b2 + hstep, voffB);
        G_WAIT_V(6); G_BAR; G_MMA(1, 1, At, B1); G_BAR;
        G_LDB(B0, 1, 0); G_SCHED; G_LDA(At, 1, 0); G_STAGE(G_SA(0, 1), a2 + hstep, voffA);
        G_WAIT_L(8); G_BAR; G_WAIT_L(0); G_MMA(0, 0, At, B0); G_BAR; G_SCHED;
        G_LDB(B1, 1, 1); G_STAGE(G_SB(1, 0), b3, voffB);
        G_BAR; G_WAIT_L(0); G_MMA(0, 1, At, B1); G_BAR;
        G_LDA(At, 1, 1); G_STAGE(G_SA(1, 0), a3, voffA);
        G_BAR; G_WAIT_L(0); G_MMA(1, 0, At, B0); G_BAR; G_SCHED;
        G_STAGE(G_SB(1, 1), b3 + hstep, voffB);
        G_WAIT_V(6); G_BAR; G_MMA(1, 1, At, B1); G_BAR;
    }
    G_WAIT_V(0);
    if (wr == 0) G_BAR;
    G_BAR;
#undef G_SA
#undef G_SB
#undef G_STAGE
#undef G_LDA
#undef G_LDB
#undef G_MMA
}

#define STAMP(p, idx) do { } while (0)
#define STAMPW(p, idx) do { } while (0)

__global__ __launch_bounds__(NTHR, 2) void fused_kernel(Params p) {
    extern __shared__ __attribute__((aligned(16))) unsigned char shm[];
    LAS unsigned char* lds = (LAS unsigned char*)shm;
    const int tid = threadIdx.x, wid = uni(tid >> 6), lane = tid & 63;
    const int rank = p.rank;
    char* const wsl = p.ws[rank];
    unsigned* const tmo = (unsigned*)(wsl + OFF_TMO);
    unsigned* const my_ctr = (unsigned*)(wsl + OFF_TICKET + (size_t)blockIdx.x * 128);
    const unsigned call = (unsigned)uni((int)__hip_atomic_load(my_ctr, __ATOMIC_RELAXED, __HIP_MEMORY_SCOPE_AGENT)) + 1u;
    __syncthreads();
    if (tid == 0) __hip_atomic_store(my_ctr, call, __ATOMIC_RELAXED, __HIP_MEMORY_SCOPE_AGENT);
    const unsigned par = call & 1u;
    const unsigned pk_base = (unsigned)(OFF_PK + par * PAR_BYTES);
    const unsigned xp_off = pk_base, wp_off = pk_base + (unsigned)XP_BYTES, op_off = wp_off + (unsigned)WP_BYTES;
    const unsigned flag_row = (unsigned)(OFF_FLAGS + ((size_t)par * NR) * UNITS * 4);
    const int b = blockIdx.x;
    STAMP(p, 0);

    if (b < NPULL) {
        if (wid >= PULL_WAVES) return;
        const int pi = b % 7, sub = b / 7;
        const int src = (rank + 1 + pi) & 7;
        const rsrc_t rs = mkrsrc(p.ws[src], (unsigned)WS_USED);
        const unsigned* fl = (const unsigned*)(wsl + flag_row + (size_t)src * UNITS * 4);
        LAS unsigned char* stg = lds + wid * 4096;
        constexpr int NWV = NPP * PULL_WAVES;
        constexpr int NITEMS = UNITS * 8;
        unsigned* const tick = (unsigned*)(wsl + OFF_PTICK + pi * 128);
        const unsigned tbase = (call - 1u) * (unsigned)NITEMS - 2u * NWV;
        int known = 0;
        bool dead = false;
        auto make_item = [&](int i, Item& it) {
            const int w = i & 7, u = i >> 3;
            if (u < IN_UNITS) {
                const int c = u >> 2, h = u & 1;
                it.dr = mkrsrc(wsl + OFF_XF, (unsigned)(OFF_PK - OFF_XF));
                it.s1 = 8192;
                if ((u & 2) == 0) {
                    it.so = xp_off + (unsigned)((c * 2 + h) * 8 + w) * SLOT;
                    it.base = (unsigned)(((256 * c + 128 * h + 16 * w) * KK + KS * src) * 2);
                    it.sh = 6; it.A = 0; it.B = 16; it.s2 = 65536;
                    it.cnt = (unsigned*)(wsl + OFF_XCNT + c * 128);
                } else {
                    it.so = wp_off + (unsigned)((rank * 32 + c * 2 + h) * 8 + w) * SLOT;
                    it.base = (unsigned)(MM * KK * 2) + (unsigned)(((256 * c + 64 * (2 * h + (w >> 2))) * KK + KS * src + 128 * (w & 3)) * 2);
                    it.sh = 3; it.A = 16; it.B = 65536; it.s2 = 128;
                    it.cnt = (unsigned*)(wsl + OFF_WCNT + c * 128);
                }
            } else {
                const int m = u - IN_UNITS; int tm, tn; unit_tile(m, tm, tn);
                it.so = op_off + (unsigned)(m * 8 + w) * SLOT;
                it.dr = mkrsrc(p.out, (unsigned)OUT_BYTES);
                it.base = (unsigned)(((size_t)(256 * tm + 32 * w) * NN + NS * src + 256 * tn) * 2);
                it.sh = 5; it.A = 65536; it.B = 16; it.s1 = 131072; it.s2 = 1048576;
                it.cnt = nullptr;
            }
        };
        auto poll = [&](int u) {
            const int uu = u + lane;
            const unsigned f = uu < UNITS ? __hip_atomic_load(fl + uu, __ATOMIC_RELAXED, __HIP_MEMORY_SCOPE_SYSTEM) : 0u;
            const unsigned long long mk = __builtin_amdgcn_ballot_w64(f >= call);
            known = u + (mk == ~0ull ? 64 : __builtin_ctzll(~mk));
            __builtin_amdgcn_fence(__ATOMIC_ACQUIRE, "wavefront");
            CBAR();
        };
        unsigned* pending = nullptr;
        Item it; Sub v; unsigned e[4]; unsigned en0 = 0;
        bool have_next = false;
        bool started = false;
        unsigned hdr = 0, T0 = 0, T1 = 0;
        auto start_item = [&](const Item& x, unsigned e0v) {
            e[0] = e0v;
            hdr = (unsigned)__builtin_amdgcn_readlane((int)e0v, 0); T0 = (unsigned)__builtin_amdgcn_readlane((int)e0v, 1); T1 = (unsigned)__builtin_amdgcn_readlane((int)e0v, 2);
            const unsigned lim = (hdr & 0xffffu) + PATCH0;
            e[1] = e[2] = e[3] = 0;
            if (lim > 64) { if (64 + (unsigned)lane < lim) e[1] = __builtin_amdgcn_raw_buffer_load_b32(rs, x.so + 256 + lane * 4, 0, AUX_SYS); }
            if (lim > 128) { if (128 + (unsigned)lane < lim) e[2] = __builtin_amdgcn_raw_buffer_load_b32(rs, x.so + 512 + lane * 4, 0, AUX_SYS);
                             if (192 + (unsigned)lane < lim) e[3] = __builtin_amdgcn_raw_buffer_load_b32(rs, x.so + 768 + lane * 4, 0, AUX_SYS); }
            load_sub(v, rs, x.so, 0, lane);
        };
        int cur = sub * PULL_WAVES + wid, nxt = cur + NWV;
        while (cur < NITEMS) {
            unsigned tk = 0;
            if (lane == 0) tk = __hip_atomic_fetch_add(tick, 1u, __ATOMIC_RELAXED, __HIP_MEMORY_SCOPE_AGENT);
            if (!started) {
                const int u = cur >> 3;
                if (u >= known && !dead) {
                    if (pending) { WAIT_VM0(); if (lane == 0) __hip_atomic_fetch_add(pending, 1u, __ATOMIC_RELAXED, __HIP_MEMORY_SCOPE_AGENT); pending = nullptr; }
                    const unsigned long long t0 = __builtin_amdgcn_s_memrealtime();
                    for (;;) {
                        poll(u);
                        if (known > u) break;
                        if (__builtin_amdgcn_readfirstlane((int)give_up(tmo, call, t0))) { dead = true; break; }
                        __builtin_amdgcn_s_sleep(4);
                    }
                }
                make_item(cur, it);
                if (!dead) {
                    if (!have_next) en0 = __builtin_amdgcn_raw_buffer_load_b32(rs, it.so + lane * 4, 0, AUX_SYS);
                    start_item(it, en0);
                }
                have_next = false;
            }
            started = false;
            if (dead) {
                if (it.cnt == nullptr) {
                    const unsigned lp = it.base + ((unsigned)lane >> it.sh) * it.A + ((unsigned)lane & ((1u << it.sh) - 1u)) * it.B;
                    for (int r = 0; r < 16; ++r)
                        __builtin_amdgcn_raw_buffer_store_b128((u32x4){0x7fc07fc0u, 0x7fc07fc0u, 0x7fc07fc0u, 0x7fc07fc0u}, it.dr, lp + (r & 7) * it.s1 + (r >> 3) * it.s2, 0, 0);
                } else if (lane == 0) __hip_atomic_fetch_add(it.cnt, 1u, __ATOMIC_RELAXED, __HIP_MEMORY_SCOPE_AGENT);
                cur = nxt; nxt = (int)((unsigned)uni((int)tk) - tbase);
                continue;
            }
            const unsigned lanepart = it.base + ((unsigned)lane >> it.sh) * it.A + ((unsigned)lane & ((1u << it.sh) - 1u)) * it.B;
            const unsigned chdr = hdr, cT0 = T0, cT1 = T1;
            const unsigned koff = (chdr & 0x10000u) ? 0x00800080u : 0u, cnt = chdr & 0xffffu;
            Item nx; bool nx_ok = false;
            if (nxt < NITEMS) {
                const int un = nxt >> 3;
                if (un >= known) poll(un);
                if (un < known) { make_item(nxt, nx); en0 = __builtin_amdgcn_raw_buffer_load_b32(rs, nx.so + lane * 4, 0, AUX_SYS); nx_ok = true; }
            }
            if (chdr & 0x80000000u) {
                if (pending) { WAIT_VM0(); if (lane == 0) __hip_atomic_fetch_add(pending, 1u, __ATOMIC_RELAXED, __HIP_MEMORY_SCOPE_AGENT); pending = nullptr; }
#pragma unroll 4
                for (int r = 0; r < 16; ++r) {
                    const u32x4 x = __builtin_amdgcn_raw_buffer_load_b128(rs, it.so + PK_OFF + r * 1024 + lane * 16, 0, AUX_SYS);
                    __builtin_amdgcn_raw_buffer_store_b128(x, it.dr, lanepart + (r & 7) * it.s1 + (r >> 3) * it.s2, 0, AUX_SYS);
                }
                pending = it.cnt;
                have_next = nx_ok;
                cur = nxt; nxt = (int)((unsigned)uni((int)tk) - tbase);
                continue;
            }
            unsigned ec[4] = {e[0], e[1], e[2], e[3]};
            Sub cv = v;
            CBAR(); load_sub(v, rs, it.so, 1, lane); CBAR();
            unpack_sub(cv, 0, cT0, cT1, koff, cnt, ec, it, lanepart, stg, lane);
            cv = v;
            CBAR(); load_sub(v, rs, it.so, 2, lane); CBAR();
            unpack_sub(cv, 1, cT0, cT1, koff, cnt, ec, it, lanepart, stg, lane);
            cv = v;
            CBAR(); load_sub(v, rs, it.so, 3, lane); CBAR();
            if (pending) {
                asm volatile("s_waitcnt vmcnt(17)" ::: "memory");
                if (lane == 0) __hip_atomic_fetch_add(pending, 1u, __ATOMIC_RELAXED, __HIP_MEMORY_SCOPE_AGENT);
                pending = nullptr;
            }
            unpack_sub(cv, 2, cT0, cT1, koff, cnt, ec, it, lanepart, stg, lane);
            cv = v;
            CBAR();
            if (nx_ok) { start_item(nx, en0); started = true; }
            CBAR();
            unpack_sub(cv, 3, cT0, cT1, koff, cnt, ec, it, lanepart, stg, lane);
            CBAR();
            pending = it.cnt;
            if (nx_ok) it = nx;
            cur = nxt; nxt = (int)((unsigned)uni((int)tk) - tbase);
        }
        if (pending) { WAIT_VM0(); if (lane == 0) __hip_atomic_fetch_add(pending, 1u, __ATOMIC_RELAXED, __HIP_MEMORY_SCOPE_AGENT); }
        return;
    }

    const int g = b - NPULL;
    {
        const rsrc_t rws = mkrsrc(wsl, (unsigned)WS_USED);
        LAS unsigned char* img = lds + wid * 16384;
        LAS unsigned* list = (LAS unsigned*)(lds + STAGE_BYTES + wid * 1024);
        const unsigned long long tstart = __builtin_amdgcn_s_memrealtime();
        for (int rnd = 0; rnd < 2; ++rnd) {
            const int job = rnd == 0 ? g : NGEMM + (NGEMM - 1 - g);
            if (job >= ENC_JOBS) break;
            const int c = job / 18, q = job % 18;
            if (rnd == 0) { while (__builtin_amdgcn_s_memrealtime() - tstart < (unsigned long long)(c * ENC_STAGGER)) __builtin_amdgcn_s_sleep(2); }
            u32x4 d[16];
            if (q < 2) {
                const int h = q;
                const rsrc_t rx = mkrsrc(p.x, (unsigned)(MM * KS * 2));
                const unsigned row0 = 256 * c + 128 * h + 16 * wid;
#pragma unroll
                for (int r = 0; r < 16; ++r) d[r] = __builtin_amdgcn_raw_buffer_load_b128(rx, (row0 + r) * (KS * 2) + lane * 16, 0, 0);
#pragma unroll
                for (int r = 0; r < 16; ++r) *(LAS u32x4*)(img + img_off(r, lane)) = d[r];
                CBAR();
                encode_wb<false>(d, rws, xp_off + (unsigned)((c * 2 + h) * 8 + wid) * SLOT, img, list, lane);
#pragma unroll
                for (int r = 0; r < 16; ++r)
                    __builtin_amdgcn_raw_buffer_store_b128(d[r], rws, (unsigned)OFF_XF + ((row0 + r) * KK + KS * rank) * 2 + lane * 16, 0, AUX_SYS);
                WAIT_VM0();
                if (lane == 0) __hip_atomic_fetch_add((unsigned*)(wsl + OFF_XCNT + c * 128), 1u, __ATOMIC_RELAXED, __HIP_MEMORY_SCOPE_AGENT);
                __syncthreads();
                if (tid < NR && tid != rank)
                    __hip_atomic_store((unsigned*)(p.ws[tid] + flag_row + (size_t)rank * UNITS * 4) + (c * 4 + h), call, __ATOMIC_RELAXED, __HIP_MEMORY_SCOPE_SYSTEM);
            } else {
                const int h = q & 1, dest = (rank + 1 + ((q - 2) >> 1)) & 7;
                const rsrc_t rw = mkrsrc(p.w, (unsigned)((size_t)KS * NN * 2));
                const int ng = 2 * h + (wid >> 2), kq = wid & 3;
                const unsigned k0 = 128 * kq + 8 * (lane >> 3), n0 = NS * dest + 256 * c + 64 * ng + 8 * (lane & 7);
                u32x4 in[16];
#pragma unroll
                for (int i = 0; i < 16; ++i) in[i] = __builtin_amdgcn_raw_buffer_load_b128(rw, ((k0 + 64 * (i >> 3) + (i & 7)) * NN + n0) * 2, 0, 0);
#pragma unroll
                for (int bh = 0; bh < 2; ++bh)
#pragma unroll
                    for (int c8 = 0; c8 < 8; ++c8)
#pragma unroll
                        for (int kk = 0; kk < 4; ++kk)
                            d[bh * 8 + c8][kk] = perm(in[bh * 8 + 2 * kk + 1][c8 >> 1], in[bh * 8 + 2 * kk][c8 >> 1], (c8 & 1) ? 0x07060302u : 0x05040100u);
                if (dest == rank) {
                    const unsigned base = (unsigned)OFF_WT + ((256 * c + 64 * ng + 8 * (lane & 7)) * KK + KS * rank + k0) * 2;
#pragma unroll
                    for (int r = 0; r < 16; ++r) __builtin_amdgcn_raw_buffer_store_b128(d[r], rws, base + (r & 7) * 8192 + (r >> 3) * 128, 0, AUX_SYS);
                    WAIT_VM0();
                    if (lane == 0) __hip_atomic_fetch_add((unsigned*)(wsl + OFF_WCNT + c * 128), 1u, __ATOMIC_RELAXED, __HIP_MEMORY_SCOPE_AGENT);
                } else {
#pragma unroll
                    for (int r = 0; r < 16; ++r) *(LAS u32x4*)(img + img_off(r, lane)) = d[r];
                    CBAR();
                    encode_wb<false>(d, rws, wp_off + (unsigned)((dest * 32 + c * 2 + h) * 8 + wid) * SLOT, img, list, lane);
                    WAIT_VM0();
                    __syncthreads();
                    if (tid == 0)
                        __hip_atomic_store((unsigned*)(p.ws[dest] + flag_row + (size_t)rank * UNITS * 4) + (c * 4 + 2 + h), call, __ATOMIC_RELAXED, __HIP_MEMORY_SCOPE_SYSTEM);
                }
            }
            STAMP(p, 8 + rnd);
        }
    }
    STAMP(p, 1);

    const int xl = b & 7, slot = g >> 3;
    if (slot >= 16) return;
    for (int rnd = 0; rnd < 2; ++rnd) {
        const int blk = xl + 8 * rnd, m = blk * 16 + slot, pos = rnd * GSLOTS;
        int tm, tn; unit_tile(m, tm, tn);
        __syncthreads();
        bool dead = false;
        if (wid == 0) {
            const bool whole = blk >= SYNC_FROM;
            const int ch = lane < 4 ? (whole ? (tm & ~3) + lane : tm) : (whole ? (tn & ~3) + (lane & 3) : tn);
            const unsigned* wp = lane < 8 ? (const unsigned*)(wsl + (lane < 4 ? OFF_XCNT : OFF_WCNT) + ch * 128) : (const unsigned*)(wsl + OFF_PTICK);
            const int pace_unit = IN_UNITS + 16 * blk - PACE_LEAD;
            const unsigned tb0 = (call - 1u) * (unsigned)(UNITS * 8) - 2u * (NPP * PULL_WAVES);
            const unsigned long long t0 = __builtin_amdgcn_s_memrealtime();
            for (;;) {
                const unsigned v = lane <= 8 ? __hip_atomic_load(wp, __ATOMIC_RELAXED, __HIP_MEMORY_SCOPE_AGENT) : 0u;
                const bool wait = lane < 8 ? v < call * 128u : (lane == 8 ? (int)(v - tb0) < pace_unit * 8 : false);
                if (__builtin_amdgcn_ballot_w64(wait) == 0ull) break;
                if (__builtin_amdgcn_readfirstlane((int)give_up(tmo, call, t0))) break;
                __builtin_amdgcn_s_sleep(16);
            }
            __builtin_amdgcn_fence(__ATOMIC_ACQUIRE, "agent");
            WAIT_VM0();
        }
        CBAR(); __builtin_amdgcn_s_barrier(); CBAR();
        STAMP(p, pos < GSLOTS ? 2 : 5);
        f32x4 acc[2][2][4][2];
        gemm_tile(lds, wsl + OFF_XF + (size_t)tm * 256 * KK * 2, wsl + OFF_WT + (size_t)tn * 256 * KK * 2, acc);
        STAMP(p, pos < GSLOTS ? 3 : 6);
        dead = __hip_atomic_load(tmo, __ATOMIC_RELAXED, __HIP_MEMORY_SCOPE_AGENT) == call;
        {
            int tz = threadIdx.x; asm volatile("" : "+v"(tz));
            const int ln = tz & 63, wr = wid >> 2, wc = wid & 3, fr = ln & 15, fq = ln >> 4;
#pragma unroll
            for (int ai = 0; ai < 2; ++ai)
#pragma unroll
                for (int mm = 0; mm < 4; ++mm)
#pragma unroll
                    for (int bj = 0; bj < 2; ++bj) {
                        const f32x4 v0 = acc[ai][bj][mm][0], v1 = acc[ai][bj][mm][1];
                        u32x4 w4;
                        w4.x = cvt2(gelu(v0[0]), gelu(v0[1])); w4.y = cvt2(gelu(v0[2]), gelu(v0[3]));
                        w4.z = cvt2(gelu(v1[0]), gelu(v1[1])); w4.w = cvt2(gelu(v1[2]), gelu(v1[3]));
                        const int row = ai * 128 + wr * 64 + mm * 16 + fr, ch = bj * 16 + wc * 4 + fq;
                        *(LAS u32x4*)(lds + row * 512 + ((ch ^ (row & 7)) << 4)) = w4;
                    }
            __syncthreads();
            u32x4 d[16];
#pragma unroll
            for (int r = 0; r < 16; ++r) {
                const int row = wid * 32 + 2 * r + (ln >> 5), ch = ln & 31;
                d[r] = *(const LAS u32x4*)(lds + row * 512 + ((ch ^ (row & 7)) << 4));
            }
            if (dead) {
#pragma unroll
                for (int r = 0; r < 16; ++r) d[r] = (u32x4){0x7fc07fc0u, 0x7fc07fc0u, 0x7fc07fc0u, 0x7fc07fc0u};
            }
            const rsrc_t ro = mkrsrc(p.out, (unsigned)OUT_BYTES);
            const unsigned ob = (unsigned)(((size_t)(256 * tm + 32 * wid + (ln >> 5)) * NN + NS * rank + 256 * tn + 8 * (ln & 31)) * 2);
#pragma unroll
            for (int r = 0; r < 16; ++r) __builtin_amdgcn_raw_buffer_store_b128(d[r], ro, ob + r * (2 * NN * 2), 0, 0);
            const rsrc_t rws = mkrsrc(wsl, (unsigned)WS_USED);
            encode_wb<true>(d, rws, op_off + (unsigned)(m * 8 + wid) * SLOT, lds + wid * 16384, (LAS unsigned*)(lds + STAGE_BYTES + wid * 1024), ln);
            WAIT_VM0();
            __syncthreads();
            if (tid < NR && tid != rank)
                __hip_atomic_store((unsigned*)(p.ws[tid] + flag_row + (size_t)rank * UNITS * 4) + (IN_UNITS + m), call, __ATOMIC_RELAXED, __HIP_MEMORY_SCOPE_SYSTEM);
        }
        STAMP(p, pos < GSLOTS ? 4 : 7);
    }
}

}

extern "C" void kernel_launch(int rank, int nranks, ncclComm_t comm, hipStream_t stream,
                              void* const* d_in, const int* in_sizes, int n_in,
                              void* d_out, int out_size, const int* in_dtypes, int out_dtype,
                              void* const* d_out_all, void* const* d_ws_all, size_t ws_bytes) {
    (void)comm; (void)in_sizes; (void)n_in; (void)out_size; (void)in_dtypes; (void)out_dtype; (void)d_out_all;
    if (nranks != gk::NR || rank < 0 || rank >= gk::NR || ws_bytes < gk::WS_USED) { fprintf(stderr, "unexpected configuration\n"); return; }
    static bool attr_done[64] = {};
    int dev = 0; HIP_CHECK(hipGetDevice(&dev));
    if (dev >= 0 && dev < 64 && !attr_done[dev]) {
        HIP_CHECK(hipFuncSetAttribute((const void*)gk::fused_kernel, hipFuncAttributeMaxDynamicSharedMemorySize, gk::LDS_BYTES));
        attr_done[dev] = true;
    }
    gk::Params p;
    p.x = d_in[0]; p.w = d_in[1]; p.out = d_out;
    for (int i = 0; i < gk::NR; ++i) p.ws[i] = (char*)d_ws_all[i];
    p.rank = rank; p.pad = 0;
    hipLaunchKernelGGL(gk::fused_kernel, dim3(gk::GRID), dim3(gk::NTHR), gk::LDS_BYTES, stream, p);
    HIP_CHECK(hipGetLastError());
}
```

```cpp
#include <hip/hip_runtime.h>
#include <rccl/rccl.h>
#include <cstdio>

#define HIP_CHECK(expr) do { hipError_t _e = (expr); if (_e != hipSuccess) { \
    fprintf(stderr, "HIP error %s at %s:%d\n", hipGetErrorString(_e), __FILE__, __LINE__); \
    return; } } while (0)

namespace gk {

typedef unsigned u32x4 __attribute__((ext_vector_type(4)));
typedef unsigned u32x3 __attribute__((ext_vector_type(3)));
typedef float f32x4 __attribute__((ext_vector_type(4)));
typedef float f32x2 __attribute__((ext_vector_type(2)));
typedef __bf16 bf16x2v __attribute__((ext_vector_type(2)));
typedef short bf16x8 __attribute__((ext_vector_type(8)));
typedef short s16x2 __attribute__((ext_vector_type(2)));
typedef unsigned short u16x2 __attribute__((ext_vector_type(2)));
typedef __amdgpu_buffer_rsrc_t rsrc_t;
#define LAS __attribute__((address_space(3)))

constexpr int NR = 8;
constexpr int MM = 4096, KK = 4096, NN = 32768, KS = 512, NS = 4096;
constexpr int NTHR = 512, GRID = 256;
#define PULL_PER_PEER 16
#define PULL_WAVES 5
#define PUSH_C 1
constexpr int NPP = PULL_PER_PEER;
constexpr int NPULL = NPP * 7;
constexpr int NGEMM = GRID - NPULL;
static_assert(NPULL % 8 == 0 && NGEMM / 8 >= 16 && NGEMM * 4 >= 256 * PUSH_C, "16 GEMM workgroups per XCD label");
constexpr int IN_UNITS = 64, OUT_UNITS = 256, UNITS = IN_UNITS + OUT_UNITS;
constexpr int NFLAG = IN_UNITS * 8 + OUT_UNITS;
constexpr int ITEM0 = 32 * PUSH_C;
#define PACE_LEAD 70
#define SYNC_FROM 2
#define PEND_LATE 1
#define AUX_LOC 17
#define RAW_CHUNKS 0
#define RAW_FLIGHT 4
#define POLL_SLEEP 8
#define POLL_LANES 16
#define EARLY_UNITS 0
#define ENC_STAGGER 50
#define ENC_LATE 400
constexpr int LDS_BYTES = 131072 + 8192;
#define NDEC 4
#define FLIGHT 4
constexpr int NSLOT = 8, QN = 2, META_LDS = 4224, SLOT_LDS = META_LDS + 8192;
constexpr int CTL_LDS = 0, STG_LDS = 256, RING_LDS = STG_LDS + 6 * 4096;
constexpr int C_QTAIL = 0, C_QHEAD = 1, C_KEND = 2, C_Q = 4, C_FULL = 8, C_FREE = 16, C_DESC = 24, C_META = 32;
static_assert(RING_LDS + NSLOT * SLOT_LDS <= LDS_BYTES && NDEC <= 6 && FLIGHT >= 2 && FLIGHT <= 9 && C_META + NSLOT <= 64, "puller LDS layout");

constexpr unsigned SLOT = 17408;
constexpr unsigned L_OFF = 4096, RAW_OFF = 1024, META_MAX = 4096;
constexpr unsigned LEN_OFF = 32, PAT_OFF = 96, PATCH_CAP = 64;
constexpr unsigned MPF_IN = 3200, MPF_OUT = 2688;

constexpr size_t OFF_TMO = 256, OFF_XCNT = 4096, OFF_WCNT = 8192, OFF_PTICK = 12288, OFF_RCNT = 14336, OFF_TICKET = 16384;
constexpr size_t OFF_FLAGS = 65536;
constexpr size_t OFF_DIAG = 262144;
constexpr size_t OFF_XF = 1u << 20;
constexpr size_t OFF_WT = OFF_XF + (size_t)MM * KK * 2;
constexpr size_t OFF_PK = OFF_WT + (size_t)NS * KK * 2;
constexpr size_t XP_BYTES = (size_t)32 * 8 * SLOT;
constexpr size_t WP_BYTES = (size_t)NR * 32 * 8 * SLOT;
constexpr size_t OP_BYTES = (size_t)OUT_UNITS * 8 * SLOT;
constexpr size_t PAR_BYTES = XP_BYTES + WP_BYTES + OP_BYTES;
constexpr size_t WS_USED = OFF_PK + 2 * PAR_BYTES;
constexpr size_t OUT_BYTES = (size_t)MM * NN * 2;
static_assert(WS_USED < (1ull << 31), "32-bit offsets");
constexpr unsigned long long DEADLINE = 100000000ull;

constexpr int AUX_SYS = 17;

struct Params {
    const void* x; const void* w; void* out;
    char* ws[NR];
    int rank; int pad;
};

__device__ __forceinline__ rsrc_t mkrsrc(const void* p, unsigned bytes) {
    return __builtin_amdgcn_make_buffer_rsrc((void*)p, 0, bytes, 0x00020000);
}
__device__ __forceinline__ unsigned pk_add16(unsigned a, unsigned b) {
    return __builtin_bit_cast(unsigned, (u16x2)(__builtin_bit_cast(u16x2, a) + __builtin_bit_cast(u16x2, b)));
}
__device__ __forceinline__ unsigned pk_sub16(unsigned a, unsigned b) {
    return __builtin_bit_cast(unsigned, (u16x2)(__builtin_bit_cast(u16x2, a) - __builtin_bit_cast(u16x2, b)));
}
__device__ __forceinline__ unsigned perm(unsigned s0, unsigned s1, unsigned sel) { return __builtin_amdgcn_perm(s0, s1, sel); }
__device__ __forceinline__ int uni(int v) { return __builtin_amdgcn_readfirstlane(v); }
#define WAIT_VM0() asm volatile("s_waitcnt vmcnt(0)" ::: "memory")
#define CBAR() asm volatile("" ::: "memory")

__device__ __forceinline__ void shell44(int i, int& a, int& b) {
    a = (int)((0x3210333210221010ull >> (4 * i)) & 15);
    b = (int)((0x3333210222101100ull >> (4 * i)) & 15);
}
__device__ __forceinline__ void unit_tile(int m, int& tm, int& tn) {
    int bi, bj, i, j; shell44(m >> 4, bi, bj); shell44(m & 15, i, j);
    tm = bi * 4 + i; tn = bj * 4 + j;
}

__host__ __device__ constexpr int in_unit(int is_w, int c, int h) { return 2 * (c < 4 ? 2 * c + is_w : 8 + ((c >> 2) - 1) * 8 + is_w * 4 + (c & 3)) + h; }
__device__ __forceinline__ void in_unit_inv(int u, int& is_w, int& c, int& h) {
    const int k = u >> 1; h = u & 1;
    if (k < 8) { is_w = k & 1; c = k >> 1; } else { const int g = (k - 8) >> 2; is_w = g & 1; c = 4 * (1 + (g >> 1)) + (k & 3); }
}
static_assert(PUSH_C >= 0 && PUSH_C <= 4 && OFF_RCNT + 4 * 256 <= OFF_TICKET, "pushed chunks");
static_assert(in_unit(0, 4, 0) == 16 && in_unit(1, 4, 1) == 25 && in_unit(0, 8, 0) == 32 && in_unit(1, 15, 1) == 63, "unit order");

__device__ __forceinline__ unsigned tick_base(unsigned call) { return (call - 1u) * (unsigned)(UNITS * 8 - ITEM0 - NPP) - 2u * NPP - ITEM0; }

__device__ __forceinline__ bool give_up(unsigned* tmo, unsigned call, unsigned long long t0) {
    if (__hip_atomic_load(tmo, __ATOMIC_RELAXED, __HIP_MEMORY_SCOPE_AGENT) == call) return true;
    if (__builtin_amdgcn_s_memrealtime() - t0 > DEADLINE) {
        __hip_atomic_store(tmo, call, __ATOMIC_RELAXED, __HIP_MEMORY_SCOPE_AGENT);
        return true;
    }
    return false;
}

__device__ __forceinline__ int robust_top(int e) {
    int t = 0;
#pragma unroll
    for (int bit = 128; bit; bit >>= 1) {
        const int c = t | bit;
        if (__builtin_popcountll(__builtin_amdgcn_ballot_w64(e >= c)) >= 4) t = c;
    }
    return t;
}

__device__ __forceinline__ unsigned img_off(unsigned r, unsigned l) { const unsigned row = 2 * r + (l >> 5); return row * 512 + (((l & 31) ^ (row & 7)) << 4); }

__device__ __forceinline__ unsigned scan_incl(unsigned v) {
    v += __builtin_amdgcn_update_dpp(0u, v, 0x111, 0xf, 0xf, true);
    v += __builtin_amdgcn_update_dpp(0u, v, 0x112, 0xf, 0xf, true);
    v += __builtin_amdgcn_update_dpp(0u, v, 0x114, 0xf, 0xf, true);
    v += __builtin_amdgcn_update_dpp(0u, v, 0x118, 0xf, 0xf, true);
    v += __builtin_amdgcn_update_dpp(0u, v, 0x142, 0xa, 0xf, false);
    v += __builtin_amdgcn_update_dpp(0u, v, 0x143, 0xc, 0xf, false);
    return v;
}
__device__ __forceinline__ unsigned wave_max(unsigned v) {
    v = max(v, (unsigned)__builtin_amdgcn_update_dpp(0u, v, 0x111, 0xf, 0xf, true));
    v = max(v, (unsigned)__builtin_amdgcn_update_dpp(0u, v, 0x112, 0xf, 0xf, true));
    v = max(v, (unsigned)__builtin_amdgcn_update_dpp(0u, v, 0x114, 0xf, 0xf, true));
    v = max(v, (unsigned)__builtin_amdgcn_update_dpp(0u, v, 0x118, 0xf, 0xf, true));
    v = max(v, (unsigned)__builtin_amdgcn_update_dpp(0u, v, 0x142, 0xa, 0xf, false));
    v = max(v, (unsigned)__builtin_amdgcn_update_dpp(0u, v, 0x143, 0xc, 0xf, false));
    return (unsigned)__builtin_amdgcn_readlane((int)v, 63);
}
__device__ __forceinline__ unsigned lut16(unsigned t0, unsigned t1, unsigned t2, unsigned t3, unsigned sel, unsigned hi) {
    return (perm(t3, t2, sel) & hi) | (perm(t1, t0, sel) & ~hi);
}
__device__ __forceinline__ unsigned hi_mask(unsigned idx) { return perm(0xffffffffu, 0u, (idx >> 1) & 0x04040404u); }

template <bool IS_OUT> struct Codes;
template <> struct Codes<true> {
    static constexpr int NP = 10;
    static constexpr unsigned char RK[16] = {15, 14, 13, 11, 9, 7, 5, 3, 1, 2, 12, 10, 8, 6, 4, 0};
};
template <> struct Codes<false> {
    static constexpr int NP = 8;
    static constexpr unsigned char RK[16] = {14, 12, 10, 8, 6, 4, 0, 2, 15, 13, 11, 9, 7, 5, 1, 3};
};
template <class C> constexpr unsigned rk_word(int q) { return C::RK[4 * q] | (C::RK[4 * q + 1] << 8) | (C::RK[4 * q + 2] << 16) | ((unsigned)C::RK[4 * q + 3] << 24); }
template <class C> constexpr int rk_inv(int rank) { for (int i = 0; i < 16; ++i) if (C::RK[i] == rank) return i; return 0; }

__device__ __forceinline__ void store_raw(const u32x4 (&d)[16], rsrc_t dst, unsigned slot_off, int lane) {
#pragma unroll
    for (int r = 0; r < 16; ++r) __builtin_amdgcn_raw_buffer_store_b128(d[r], dst, slot_off + RAW_OFF + r * 1024 + lane * 16, 0, AUX_SYS);
    if (lane == 0) __builtin_amdgcn_raw_buffer_store_b32(0x80000000u, dst, slot_off, 0, AUX_SYS);
}

#define ESTAMP(k) do { } while (0)
template <bool IS_OUT>
__device__ __forceinline__ void encode_wb(const u32x4 (&d)[16], rsrc_t dst, unsigned slot_off, LAS unsigned char* img, LAS unsigned* list, int lane, unsigned long long* est = nullptr) {
    ESTAMP(0);
    typedef Codes<IS_OUT> C;
    constexpr int NP = C::NP, NNEG = 16 - NP;
    s16x2 pm = __builtin_bit_cast(s16x2, d[0].x); u16x2 nm = __builtin_bit_cast(u16x2, d[0].x);
#pragma unroll
    for (int r = 0; r < 4; ++r)
#pragma unroll
        for (int j = 0; j < 4; ++j) {
            pm = __builtin_elementwise_max(pm, __builtin_bit_cast(s16x2, d[r][j]));
            nm = __builtin_elementwise_max(nm, __builtin_bit_cast(u16x2, d[r][j]));
        }
    const int p16 = pm.x > pm.y ? pm.x : pm.y;
    const unsigned n16 = nm.x > nm.y ? nm.x : nm.y;
    const int pe = p16 > 0 ? (p16 >> 7) & 255 : 0;
    const int ne = (n16 & 0x8000u) ? (int)(n16 >> 7) & 255 : 0;
    int tp = robust_top(pe), tn = robust_top(ne);
    if (!IS_OUT) { tp = tp > tn ? tp : tn; tn = tp; }
    const int off = (tn & 1) ^ 1;
    int pb = ((tp + off) >> 1) - (NP - 1), nb = ((tn + off) >> 1) - (NNEG - 1);
    pb = pb < 0 ? 0 : (pb > 128 - NP ? 128 - NP : pb);
    nb = nb < NP ? NP : (nb > 128 - NNEG ? 128 - NNEG : nb);
    auto cls = [&](int cc) -> unsigned { return cc < NP ? (unsigned)(pb + cc) : (0x80u | (unsigned)(nb + cc - NP)); };
    unsigned CT[4], HT[4];
#pragma unroll
    for (int q = 0; q < 4; ++q) {
        CT[q] = cls(4 * q) | (cls(4 * q + 1) << 8) | (cls(4 * q + 2) << 16) | (cls(4 * q + 3) << 24);
        HT[q] = cls(rk_inv<C>(4 * q)) | (cls(rk_inv<C>(4 * q + 1)) << 8) | (cls(rk_inv<C>(4 * q + 2)) << 16) | (cls(rk_inv<C>(4 * q + 3)) << 24);
    }
    ESTAMP(1);
    const unsigned pb4 = (unsigned)pb * 0x01010101u, nb4 = (unsigned)(nb - NP) * 0x01010101u;
    const unsigned koff = off ? 0x00800080u : 0u;
    unsigned cnt = 0, S = 0, rk[32];
#pragma unroll
    for (int s = 0; s < 4; ++s) {
        unsigned L[8]; unsigned mk = 0;
#pragma unroll
        for (int g = 0; g < 8; ++g) {
            const int r = 4 * s + (g >> 1), h = g & 1;
            const unsigned e0 = pk_add16(d[r][2 * h], koff), e1 = pk_add16(d[r][2 * h + 1], koff);
            L[g] = perm(e1, e0, 0x06040200u);
            const unsigned H = perm(e1, e0, 0x07050301u);
            const unsigned M = perm(e1, e0, 0x0b0a0908u);
            const unsigned base = (M & nb4) | (~M & pb4);
            const unsigned cc = ((H | 0x80808080u) - base) & 0x0f0f0f0fu;
            const unsigned sel = cc & 0x07070707u, hi = hi_mask(cc);
            const unsigned bad = lut16(CT[0], CT[1], CT[2], CT[3], sel, hi) ^ H;
            const unsigned rr = lut16(rk_word<C>(0), rk_word<C>(1), rk_word<C>(2), rk_word<C>(3), sel, hi);
            rk[8 * s + g] = rr;
            S = __builtin_amdgcn_sad_u8(rr, 0u, S);
            const unsigned nz = ((bad & 0x7f7f7f7fu) + 0x7f7f7f7fu) | bad;
            mk |= (nz >> (7 - g)) & (0x01010101u << g);
        }
        const unsigned o = slot_off + L_OFF + s * 2048;
        __builtin_amdgcn_raw_buffer_store_b128((u32x4){L[0], L[1], L[2], L[3]}, dst, o + lane * 16, 0, AUX_SYS);
        __builtin_amdgcn_raw_buffer_store_b128((u32x4){L[4], L[5], L[6], L[7]}, dst, o + 1024 + lane * 16, 0, AUX_SYS);
        unsigned long long m = __builtin_amdgcn_ballot_w64(mk != 0);
        while (m) {
            const unsigned pos = cnt + __builtin_amdgcn_mbcnt_hi((unsigned)(m >> 32), __builtin_amdgcn_mbcnt_lo((unsigned)m, 0));
            if (mk != 0) {
                const unsigned bit = (unsigned)__builtin_ctz(mk), g = bit & 7, j = bit >> 3;
                const unsigned r = 4 * s + (g >> 1), jj = 4 * (g & 1) + j;
                if (pos < PATCH_CAP) list[pos] = ((unsigned)lane << 23) | (r << 19) | (jj << 16);
                mk &= mk - 1;
            }
            cnt += (unsigned)__builtin_popcountll(m);
            m = __builtin_amdgcn_ballot_w64(mk != 0);
        }
    }
    ESTAMP(2);
    const unsigned smax = wave_max(S), smin = 0xffffu - wave_max(0xffffu - S);
    const unsigned bits = 128u + S, incl = scan_incl(bits);
    const unsigned total = (unsigned)__builtin_amdgcn_readlane((int)incl, 63);
    const unsigned code0 = PAT_OFF + 4 * cnt, mbytes = code0 + ((total + 31) >> 5) * 4;
    if (cnt > PATCH_CAP || smax - smin > 255u || mbytes > META_MAX) {
        WAIT_VM0();
        store_raw(d, dst, slot_off, lane);
        return;
    }
    unsigned pe_ = 0;
    if ((unsigned)lane < cnt) { const unsigned e = list[lane]; pe_ = e | *(const LAS unsigned short*)(img + img_off((e >> 19) & 15u, e >> 23) + ((e >> 16) & 7u) * 2); }
    asm volatile("s_waitcnt lgkmcnt(0)" ::: "memory");
#pragma unroll
    for (int q = 0; q < 4; ++q) *(LAS u32x4*)(img + q * 1024 + lane * 16) = (u32x4){0u, 0u, 0u, 0u};
    CBAR();
    if (lane < 6) ((LAS unsigned*)img)[lane] = lane == 0 ? (cnt | ((unsigned)off << 8) | (mbytes << 16)) : (lane == 1 ? smin : (lane == 2 ? HT[0] : (lane == 3 ? HT[1] : (lane == 4 ? HT[2] : HT[3]))));
    img[LEN_OFF + lane] = (unsigned char)(S - smin);
    if ((unsigned)lane < cnt) *(LAS unsigned*)(img + PAT_OFF + 4 * lane) = pe_;
    CBAR();
    ESTAMP(3);
    {
        const unsigned B = code0 * 8 + incl - bits;
        unsigned addr = (B >> 5) * 4, pos = B & 31, acc = 0;
#pragma unroll
        for (int g = 0; g < 32; ++g)
#pragma unroll
            for (int h = 0; h < 2; ++h) {
                const unsigned r0 = (rk[g] >> (16 * h)) & 0xffu, r1 = (rk[g] >> (16 * h + 8)) & 0xffu;
                const unsigned p1 = r0 + r1 + 1;
                const unsigned long long y = (unsigned long long)((1u << r0) | (1u << p1)) << pos;
                acc |= (unsigned)y;
                pos += p1 + 1;
                if (pos >= 32) {
                    __hip_atomic_fetch_or((LAS unsigned*)(img + addr), acc, __ATOMIC_RELAXED, __HIP_MEMORY_SCOPE_WORKGROUP);
                    acc = (unsigned)(y >> 32); addr += 4; pos -= 32;
                }
            }
        if (pos) __hip_atomic_fetch_or((LAS unsigned*)(img + addr), acc, __ATOMIC_RELAXED, __HIP_MEMORY_SCOPE_WORKGROUP);
    }
    ESTAMP(4);
    CBAR();
#pragma unroll
    for (int q = 0; q < 4; ++q) {
        const unsigned o = q * 1024 + lane * 16;
        if (o < mbytes) __builtin_amdgcn_raw_buffer_store_b128(*(const LAS u32x4*)(img + o), dst, slot_off + o, 0, AUX_SYS);
    }
    ESTAMP(5);
}

struct Item { rsrc_t dr; unsigned so, base, sh, A, B, s1, s2, mpf; unsigned* cnt; };
struct Sub { u32x4 La, Lb; };
struct Dec { unsigned T0, T1, T2, T3, koff, cnt, pe, w0, w1, w2, w3, pos; LAS const unsigned char* addr; };

__device__ __forceinline__ void load_sub(Sub& v, rsrc_t src, unsigned so, int s, int lane) {
    const unsigned o = so + L_OFF + s * 2048;
    v.La = __builtin_amdgcn_raw_buffer_load_b128(src, o + lane * 16, 0, AUX_SYS);
    v.Lb = __builtin_amdgcn_raw_buffer_load_b128(src, o + 1024 + lane * 16, 0, AUX_SYS);
}
__device__ __forceinline__ u32x4 load_piece(rsrc_t src, unsigned so, unsigned q, unsigned lim, int lane) {
    u32x4 v = {0u, 0u, 0u, 0u};
    const unsigned o = q * 1024 + lane * 16;
    if (o < lim) v = __builtin_amdgcn_raw_buffer_load_b128(src, so + o, 0, AUX_SYS);
    return v;
}

__device__ __forceinline__ unsigned dec2(Dec& D) {
    unsigned x = __builtin_amdgcn_alignbit(D.w1, D.w0, D.pos);
    const unsigned r0 = (unsigned)__builtin_ctz(x);
    x >>= r0 + 1;
    const unsigned r1 = (unsigned)__builtin_ctz(x);
    const unsigned q = D.pos + r0 + r1 + 2;
    const bool c = q > 31;
    D.pos = q & 31;
    D.w0 = c ? D.w1 : D.w0; D.w1 = c ? D.w2 : D.w1; D.w2 = c ? D.w3 : D.w2;
    D.addr += c ? 4 : 0;
    D.w3 = *(const LAS unsigned*)D.addr;
    return r0 | (r1 << 8);
}

__device__ __forceinline__ void unpack_sub(const Sub& v, const unsigned (&H)[8], int s, const Dec& D, const Item& it, unsigned lanepart, LAS unsigned char* stg, int lane) {
    u32x4 o[4];
#pragma unroll
    for (int rr = 0; rr < 4; ++rr) {
        const unsigned L0 = rr < 2 ? v.La[2 * rr] : v.Lb[2 * rr - 4], L1 = rr < 2 ? v.La[2 * rr + 1] : v.Lb[2 * rr - 3];
        const unsigned H0 = H[2 * rr], H1 = H[2 * rr + 1];
        o[rr].x = pk_sub16(perm(H0, L0, 0x05010400u), D.koff);
        o[rr].y = pk_sub16(perm(H0, L0, 0x07030602u), D.koff);
        o[rr].z = pk_sub16(perm(H1, L1, 0x05010400u), D.koff);
        o[rr].w = pk_sub16(perm(H1, L1, 0x07030602u), D.koff);
    }
    const bool mine = (unsigned)lane < D.cnt && ((D.pe >> 21) & 3u) == (unsigned)s;
    if (__builtin_amdgcn_ballot_w64(mine)) {
#pragma unroll
        for (int rr = 0; rr < 4; ++rr) *(LAS u32x4*)(stg + rr * 1024 + lane * 16) = o[rr];
        CBAR();
        if (mine) *(LAS unsigned short*)(stg + ((D.pe >> 19) & 3u) * 1024 + (D.pe >> 23) * 16 + ((D.pe >> 16) & 7u) * 2) = (unsigned short)D.pe;
        CBAR();
#pragma unroll
        for (int rr = 0; rr < 4; ++rr) o[rr] = *(const LAS u32x4*)(stg + rr * 1024 + lane * 16);
    }
#pragma unroll
    for (int rr = 0; rr < 4; ++rr) {
        const int r = 4 * s + rr;
        __builtin_amdgcn_raw_buffer_store_b128(o[rr], it.dr, lanepart + (r & 7) * it.s1 + (r >> 3) * it.s2, 0, AUX_LOC);
    }
}
__device__ __forceinline__ void decode_highs(Dec& D, unsigned (&H)[8]) {
#pragma unroll
    for (int g = 0; g < 8; ++g) {
        const unsigned a = dec2(D), b = dec2(D);
        const unsigned rk = a | (b << 16);
        H[g] = lut16(D.T0, D.T1, D.T2, D.T3, rk & 0x07070707u, hi_mask(rk));
    }
}

constexpr int BM = 256, BK = 64, HALF = 128, HTB = HALF * BK * 2, STAGE_BYTES = 8 * HTB;
__device__ __forceinline__ int lds_byte(int r, int c) { const int st = (r >> 4) * 2 + (c >> 5), rr = r & 15, cc = c & 31, ob = rr * 64 + cc * 2; return st * 1024 + (ob ^ (((ob >> 9) & 1) << 5)); }
__device__ __forceinline__ void stage_rc(int b, int& R, int& C) { const int st = b / 1024, sb = b % 1024, swz = sb ^ (((sb >> 9) & 1) << 5); R = (st >> 1) * 16 + swz / 64; C = (st & 1) * 32 + (swz % 64) / 2; }
__device__ __forceinline__ int perm32(int rho) { const int n = rho >> 4, i = rho & 15; return 8 * (i >> 2) + 4 * n + (i & 3); }

__device__ __forceinline__ unsigned gelu2(float a, float b) {
    const f32x2 y = {a, b};
    const f32x2 c0 = {-2.3022081985f, -2.3022081985f}, c1 = {-0.1029432397f, -0.1029432397f};
    const f32x2 z = y * __builtin_elementwise_fma(y * y, c1, c0);
    f32x2 e; e.x = __builtin_amdgcn_exp2f(z.x); e.y = __builtin_amdgcn_exp2f(z.y);
    const f32x2 dn = e + (f32x2){1.0f, 1.0f};
    f32x2 r; r.x = __builtin_amdgcn_rcpf(dn.x); r.y = __builtin_amdgcn_rcpf(dn.y);
    return __builtin_bit_cast(unsigned, __builtin_convertvector(y * r, bf16x2v));
}

__device__ __forceinline__ void gemm_tile(LAS unsigned char* lds, const char* Abase, const char* Bbase, f32x4 (&acc)[2][2][4][2]) {
    const int tid = threadIdx.x, wid = uni(tid >> 6), lane = tid & 63, wr = wid >> 2, wc = wid & 3, fr = lane & 15, fq = lane >> 4;
    constexpr int K = KK, nt = K / BK;
    unsigned voffA[2], voffB[2];
#pragma unroll
    for (int i = 0; i < 2; ++i) { int R, C; stage_rc(tid * 16 + i * 8192, R, C); const int Rb = (R & ~31) + perm32(R & 31);
        voffA[i] = (unsigned)(R * K + C) * 2u; voffB[i] = (unsigned)(Rb * K + C) * 2u; }
    constexpr size_t kstep = (size_t)(BK * 2), hstep = (size_t)HALF * K * 2;
    const unsigned ldsw = (unsigned)wid * 1024u;
    const int aoff = lds_byte(wr * 64 + fr, fq * 8), boff = lds_byte(wc * 32 + fr, fq * 8);
#define G_SA(b, h) (((b) * 2 + (h)) * HTB)
#define G_SB(b, h) ((4 + (b) * 2 + (h)) * HTB)
#define G_STAGE(bufoff, gbase, voff) do { _Pragma("unroll") for (int _i = 0; _i < 2; ++_i) \
        __builtin_amdgcn_global_load_lds((const unsigned*)((const char*)(gbase) + (voff)[_i]), (LAS unsigned*)(lds + (bufoff) + ldsw + _i * 8192), 16, 0, 0); } while (0)
#define G_LDA(dst, b, h) do { _Pragma("unroll") for (int m = 0; m < 4; ++m) _Pragma("unroll") for (int k = 0; k < 2; ++k) dst[m][k] = *(const LAS bf16x8*)(lds + G_SA(b, h) + aoff + m * 2048 + k * 1024); } while (0)
#define G_LDB(dst, b, h) do { _Pragma("unroll") for (int n = 0; n < 2; ++n) _Pragma("unroll") for (int k = 0; k < 2; ++k) dst[n][k] = *(const LAS bf16x8*)(lds + G_SB(b, h) + boff + n * 2048 + k * 1024); } while (0)
#define G_MMA(ai, bj, At, Bt) do { __builtin_amdgcn_s_setprio(1); _Pragma("unroll") for (int m = 0; m < 4; ++m) _Pragma("unroll") for (int n = 0; n < 2; ++n) _Pragma("unroll") for (int k = 0; k < 2; ++k) \
        acc[ai][bj][m][n] = __builtin_amdgcn_mfma_f32_16x16x32_bf16(Bt[n][k], At[m][k], acc[ai][bj][m][n], 0, 0, 0); __builtin_amdgcn_s_setprio(0); } while (0)
#define G_WAIT_V(n) asm volatile("s_waitcnt vmcnt(" #n ")" ::: "memory")
#define G_WAIT_L(n) asm volatile("s_waitcnt lgkmcnt(" #n ")" ::: "memory")
#define G_BAR __builtin_amdgcn_s_barrier()
#define G_SCHED __builtin_amdgcn_sched_barrier(0)
#pragma unroll
    for (int a = 0; a < 2; ++a)
#pragma unroll
        for (int b = 0; b < 2; ++b)
#pragma unroll
            for (int m = 0; m < 4; ++m)
#pragma unroll
                for (int n = 0; n < 2; ++n) acc[a][b][m][n] = (f32x4){0.f, 0.f, 0.f, 0.f};
    bf16x8 At[4][2], B0[2][2], B1[2][2];
    const char* cA = Abase; const char* cB = Bbase;
    G_STAGE(G_SB(0, 0), cB, voffB); G_STAGE(G_SA(0, 0), cA, voffA); G_STAGE(G_SB(0, 1), cB + hstep, voffB); G_STAGE(G_SA(0, 1), cA + hstep, voffA);
    if (wr == 1) G_BAR;
    G_WAIT_V(4); G_BAR;
    G_STAGE(G_SB(1, 0), cB + kstep, voffB); G_STAGE(G_SA(1, 0), cA + kstep, voffA); G_STAGE(G_SB(1, 1), cB + hstep + kstep, voffB);
    G_WAIT_V(6); G_BAR;
    for (int t = 0; t < nt; t += 2) {
        const bool last = (t == nt - 2);
        const char* a1 = cA + (size_t)(t + 1) * kstep;
        const char* a2 = last ? cA : cA + (size_t)(t + 2) * kstep; const char* b2 = last ? cB : cB + (size_t)(t + 2) * kstep;
        const char* a3 = a2 + kstep; const char* b3 = b2 + kstep;
        G_LDB(B0, 0, 0); G_SCHED; G_LDA(At, 0, 0); G_STAGE(G_SA(1, 1), a1 + hstep, voffA);
        G_WAIT_L(8); G_BAR; G_WAIT_L(0); G_MMA(0, 0, At, B0); G_BAR; G_SCHED;
        G_LDB(B1, 0, 1); G_STAGE(G_SB(0, 0), b2, voffB);
        G_BAR; G_WAIT_L(0); G_MMA(0, 1, At, B1); G_BAR;
        G_LDA(At, 0, 1); G_STAGE(G_SA(0, 0), a2, voffA);
        G_BAR; G_WAIT_L(0); G_MMA(1, 0, At, B0); G_BAR; G_SCHED;
        G_STAGE(G_SB(0, 1), b2 + hstep, voffB);
        G_WAIT_V(6); G_BAR; G_MMA(1, 1, At, B1); G_BAR;
        G_LDB(B0, 1, 0); G_SCHED; G_LDA(At, 1, 0); G_STAGE(G_SA(0, 1), a2 + hstep, voffA);
        G_WAIT_L(8); G_BAR; G_WAIT_L(0); G_MMA(0, 0, At, B0); G_BAR; G_SCHED;
        G_LDB(B1, 1, 1); G_STAGE(G_SB(1, 0), b3, voffB);
        G_BAR; G_WAIT_L(0); G_MMA(0, 1, At, B1); G_BAR;
        G_LDA(At, 1, 1); G_STAGE(G_SA(1, 0), a3, voffA);
        G_BAR; G_WAIT_L(0); G_MMA(1, 0, At, B0); G_BAR; G_SCHED;
        G_STAGE(G_SB(1, 1), b3 + hstep, voffB);
        G_WAIT_V(6); G_BAR; G_MMA(1, 1, At, B1); G_BAR;
    }
    G_WAIT_V(0);
    if (wr == 0) G_BAR;
    G_BAR;
#undef G_SA
#undef G_SB
#undef G_STAGE
#undef G_LDA
#undef G_LDB
#undef G_MMA
}

#define STAMP(p, idx) do { } while (0)
#define STAMPW(p, idx) do { } while (0)
#define WSTAMP(n, k) do { } while (0)

__global__ __launch_bounds__(NTHR, 2) void fused_kernel(Params p) {
    extern __shared__ __attribute__((aligned(16))) unsigned char shm[];
    LAS unsigned char* lds = (LAS unsigned char*)shm;
    const int tid = threadIdx.x, wid = uni(tid >> 6), lane = tid & 63;
    const int rank = p.rank;
    char* const wsl = p.ws[rank];
    unsigned* const tmo = (unsigned*)(wsl + OFF_TMO);
    unsigned* const my_ctr = (unsigned*)(wsl + OFF_TICKET + (size_t)blockIdx.x * 128);
    const unsigned call = (unsigned)uni((int)__hip_atomic_load(my_ctr, __ATOMIC_RELAXED, __HIP_MEMORY_SCOPE_AGENT)) + 1u;
    __syncthreads();
    if (tid == 0) __hip_atomic_store(my_ctr, call, __ATOMIC_RELAXED, __HIP_MEMORY_SCOPE_AGENT);
    const unsigned par = call & 1u;
    const unsigned pk_base = (unsigned)(OFF_PK + par * PAR_BYTES);
    const unsigned xp_off = pk_base, wp_off = pk_base + (unsigned)XP_BYTES, op_off = wp_off + (unsigned)WP_BYTES;
    const unsigned flag_row = (unsigned)(OFF_FLAGS + ((size_t)par * NR) * NFLAG * 4);
    const int b = blockIdx.x;
    STAMP(p, 0);

    if (b < NPULL) {
        volatile LAS unsigned* const ctl = (volatile LAS unsigned*)(lds + CTL_LDS);
        if (tid < 64) ctl[tid] = tid == C_KEND ? 0x7fffffffu : 0u;
        __syncthreads();
        if (wid >= 2 + NDEC) return;
        const int pi = b % 7, sub = b / 7;
        const int src = (rank + 1 + pi) & 7;
        const rsrc_t rs = mkrsrc(p.ws[src], (unsigned)WS_USED);
        constexpr int NITEMS = UNITS * 8;
        auto item_so = [&](int i) -> unsigned {
            const int w = i & 7, u = i >> 3;
            if (u < IN_UNITS) {
                int is_w, c, h; in_unit_inv(u, is_w, c, h);
                return is_w ? wp_off + (unsigned)((rank * 32 + c * 2 + h) * 8 + w) * SLOT : xp_off + (unsigned)((c * 2 + h) * 8 + w) * SLOT;
            }
            return op_off + (unsigned)((u - IN_UNITS) * 8 + w) * SLOT;
        };
        if (wid == 1) {
            const unsigned* fl = (const unsigned*)(wsl + flag_row + (size_t)src * NFLAG * 4);
            unsigned* const tick = (unsigned*)(wsl + OFF_PTICK + pi * 128);
            const unsigned tbase = tick_base(call);
            int known = 0; bool dead = false;
            for (int n = 0;; ++n) {
                int item;
                if (n < 2) item = ITEM0 + sub + n * NPP;
                else { unsigned tk = 0; if (lane == 0) tk = __hip_atomic_fetch_add(tick, 1u, __ATOMIC_RELAXED, __HIP_MEMORY_SCOPE_AGENT); item = (int)((unsigned)uni((int)tk) - tbase); }
                unsigned e = 0xffffffffu;
                if (item < NITEMS) {
                    const int u = item < IN_UNITS * 8 ? item : IN_UNITS * 8 + ((item - IN_UNITS * 8) >> 3);
                    if (u >= known && !dead) {
                        const unsigned long long t0 = __builtin_amdgcn_s_memrealtime();
                        for (unsigned spins = 0;; ++spins) {
                            const int uu = u + lane;
                            const unsigned f = uu < NFLAG ? __hip_atomic_load(fl + uu, __ATOMIC_RELAXED, __HIP_MEMORY_SCOPE_SYSTEM) : 0u;
                            const unsigned long long mk = __builtin_amdgcn_ballot_w64(f >= call);
                            known = u + __builtin_ctzll(~mk | (1ull << 63));
                            if (known > u) break;
                            if ((spins & 15u) == 15u && __builtin_amdgcn_readfirstlane((int)give_up(tmo, call, t0))) { dead = true; break; }
                            __builtin_amdgcn_s_sleep(POLL_SLEEP);
                        }
                    }
                    e = (unsigned)item | (dead ? 0x40000000u : 0u);
                }
                while ((unsigned)n - ctl[C_QHEAD] >= QN) __builtin_amdgcn_s_sleep(2);
                CBAR();
                if (lane == 0) { ctl[C_Q + (n % QN)] = e; asm volatile("s_waitcnt lgkmcnt(0)" ::: "memory"); ctl[C_QTAIL] = (unsigned)n + 1u; }
                if (item >= NITEMS) break;
            }
            return;
        }
        if (wid == 0) {
            int pend = -1;
            for (int k = 0;; ++k) {
                const int slot = k % NSLOT;
                if (ctl[C_QTAIL] <= (unsigned)k || (k >= NSLOT && ctl[C_FREE + slot] < (unsigned)(k - NSLOT + 1))) {
                    WAIT_VM0();
                    if (pend >= 0) { ctl[C_FULL + pend % NSLOT] = (unsigned)pend + 1u; pend = -1; }
                    while (ctl[C_QTAIL] <= (unsigned)k) __builtin_amdgcn_s_sleep(1);
                    while (k >= NSLOT && ctl[C_FREE + slot] < (unsigned)(k - NSLOT + 1)) __builtin_amdgcn_s_sleep(1);
                }
                CBAR();
                const unsigned e = (unsigned)uni((int)ctl[C_Q + (k % QN)]);
                asm volatile("s_waitcnt lgkmcnt(0)" ::: "memory");
                ctl[C_QHEAD] = (unsigned)k + 1u;
                if (e == 0xffffffffu) {
                    WAIT_VM0();
                    if (pend >= 0) ctl[C_FULL + pend % NSLOT] = (unsigned)pend + 1u;
                    ctl[C_KEND] = (unsigned)k;
                    break;
                }
                ctl[C_DESC + slot] = e;
                if (e & 0x40000000u) {
                    WAIT_VM0();
                    if (pend >= 0) { ctl[C_FULL + pend % NSLOT] = (unsigned)pend + 1u; pend = -1; }
                    asm volatile("s_waitcnt lgkmcnt(0)" ::: "memory");
                    ctl[C_META + slot] = (unsigned)k + 1u;
                    ctl[C_FULL + slot] = (unsigned)k + 1u;
                    continue;
                }
                const int item = (int)e;
                const unsigned so = item_so(item) + lane * 16;
                const unsigned mpf = item < IN_UNITS * 8 ? MPF_IN : MPF_OUT;
                LAS unsigned char* sb = lds + RING_LDS + slot * SLOT_LDS;
#pragma unroll
                for (int j = 0; j < 12; ++j) {
                    if (j < 4) {
                        if ((unsigned)(j * 1024) >= mpf) continue;
                        if ((unsigned)(j * 1024 + lane * 16) < mpf)
                            __builtin_amdgcn_raw_ptr_buffer_load_lds(rs, (LAS void*)(sb + j * 1024), 16, so + j * 1024, 0, 0, AUX_SYS);
                    } else {
                        __builtin_amdgcn_raw_ptr_buffer_load_lds(rs, (LAS void*)(sb + META_LDS + (j - 4) * 1024), 16, so + L_OFF + (j - 4) * 1024, 0, 0, AUX_SYS);
                    }
                    asm volatile("s_waitcnt vmcnt(%0)" :: "n"(FLIGHT - 1) : "memory");
                    if (j >= FLIGHT - 1 && pend >= 0) { ctl[C_FULL + pend % NSLOT] = (unsigned)pend + 1u; pend = -1; }
                    if (j == 3 + (FLIGHT - 1)) ctl[C_META + slot] = (unsigned)k + 1u;
                }
                pend = k;
            }
            return;
        }
        const int dw = wid - 2;
        LAS unsigned char* stg = lds + STG_LDS + dw * 4096;
        for (int k = dw;; k += NDEC) {
            const int slot = k % NSLOT;
            bool stop = false;
            while (ctl[C_META + slot] < (unsigned)k + 1u) {
                if (ctl[C_KEND] <= (unsigned)k) { stop = true; break; }
                __builtin_amdgcn_s_sleep(1);
            }
            if (stop) break;
            CBAR();
            const unsigned e = (unsigned)uni((int)ctl[C_DESC + slot]);
            const int cur = (int)(e & 0x3fffffffu);
            Item it;
            {
                const int w = cur & 7, u = cur >> 3;
                it.so = item_so(cur);
                if (u < IN_UNITS) {
                    int is_w, c, h; in_unit_inv(u, is_w, c, h);
                    it.dr = mkrsrc(wsl + OFF_XF, (unsigned)(OFF_PK - OFF_XF));
                    it.s1 = 8192; it.mpf = MPF_IN;
                    if (!is_w) {
                        it.base = (unsigned)(((256 * c + 128 * h + 16 * w) * KK + KS * src) * 2);
                        it.sh = 6; it.A = 0; it.B = 16; it.s2 = 65536;
                        it.cnt = (unsigned*)(wsl + OFF_XCNT + c * 128);
                    } else {
                        it.base = (unsigned)(MM * KK * 2) + (unsigned)(((256 * c + 64 * (2 * h + (w >> 2))) * KK + KS * src + 128 * (w & 3)) * 2);
                        it.sh = 3; it.A = 16; it.B = 65536; it.s2 = 128;
                        it.cnt = (unsigned*)(wsl + OFF_WCNT + c * 128);
                    }
                } else {
                    const int m = u - IN_UNITS; int tm, tn; unit_tile(m, tm, tn);
                    it.dr = mkrsrc(p.out, (unsigned)OUT_BYTES);
                    it.base = (unsigned)(((size_t)(256 * tm + 32 * w) * NN + NS * src + 256 * tn) * 2);
                    it.sh = 5; it.A = 65536; it.B = 16; it.s1 = 131072; it.s2 = 1048576;
                    it.cnt = nullptr; it.mpf = MPF_OUT;
                }
            }
            const unsigned lanepart = it.base + ((unsigned)lane >> it.sh) * it.A + ((unsigned)lane & ((1u << it.sh) - 1u)) * it.B;
            LAS unsigned char* mc = lds + RING_LDS + slot * SLOT_LDS;
            const unsigned hdr = (e & 0x40000000u) ? 0u : (unsigned)uni((int)*(const LAS unsigned*)mc);
            if (e & 0x40000000u) {
                ctl[C_FREE + slot] = (unsigned)k + 1u;
                if (it.cnt == nullptr) {
                    for (int r = 0; r < 16; ++r)
                        __builtin_amdgcn_raw_buffer_store_b128((u32x4){0x7fc07fc0u, 0x7fc07fc0u, 0x7fc07fc0u, 0x7fc07fc0u}, it.dr, lanepart + (r & 7) * it.s1 + (r >> 3) * it.s2, 0, 0);
                }
            } else if (hdr & 0x80000000u) {
                while (ctl[C_FULL + slot] < (unsigned)k + 1u) __builtin_amdgcn_s_sleep(1);
                ctl[C_FREE + slot] = (unsigned)k + 1u;
#pragma unroll 1
                for (int r0 = 0; r0 < 16; r0 += 4) {
                    u32x4 x[4];
#pragma unroll
                    for (int r = 0; r < 4; ++r) x[r] = __builtin_amdgcn_raw_buffer_load_b128(rs, it.so + RAW_OFF + (r0 + r) * 1024 + lane * 16, 0, AUX_SYS);
#pragma unroll
                    for (int r = 0; r < 4; ++r) __builtin_amdgcn_raw_buffer_store_b128(x[r], it.dr, lanepart + ((r0 + r) & 7) * it.s1 + ((r0 + r) >> 3) * it.s2, 0, AUX_LOC);
                }
            } else {
                Dec D;
                const unsigned mbytes = (hdr >> 16) & 0x1fffu;
                if (mbytes > it.mpf) {
                    for (unsigned q = it.mpf >> 10; q < 4; ++q) {
                        const unsigned o = q * 1024 + lane * 16;
                        if (o >= it.mpf && o < mbytes) *(LAS u32x4*)(mc + o) = __builtin_amdgcn_raw_buffer_load_b128(rs, it.so + o, 0, AUX_SYS);
                    }
                    CBAR();
                }
                const u32x4 h0 = *(const LAS u32x4*)mc; const unsigned h4 = *(const LAS unsigned*)(mc + 16), h5 = *(const LAS unsigned*)(mc + 20);
                D.cnt = hdr & 0xffu; D.koff = (hdr & 0x100u) ? 0x00800080u : 0u;
                D.T0 = (unsigned)uni((int)h0.z); D.T1 = (unsigned)uni((int)h0.w); D.T2 = (unsigned)uni((int)h4); D.T3 = (unsigned)uni((int)h5);
                const unsigned bits = 128u + (unsigned)uni((int)h0.y) + mc[LEN_OFF + lane];
                const unsigned B = (PAT_OFF + 4 * D.cnt) * 8 + scan_incl(bits) - bits;
                const LAS unsigned* wp = (const LAS unsigned*)(mc + (B >> 5) * 4);
                D.pos = B & 31; D.w0 = wp[0]; D.w1 = wp[1]; D.w2 = wp[2]; D.w3 = wp[3]; D.addr = (const LAS unsigned char*)(wp + 3);
                D.pe = *(const LAS unsigned*)(mc + PAT_OFF + 4 * lane);
                unsigned H[4][8];
#pragma unroll
                for (int s = 0; s < 4; ++s) decode_highs(D, H[s]);
                while (ctl[C_FULL + slot] < (unsigned)k + 1u) __builtin_amdgcn_s_sleep(1);
                CBAR();
#pragma unroll
                for (int s = 0; s < 4; ++s) {
                    Sub cv;
                    cv.La = *(const LAS u32x4*)(mc + META_LDS + s * 2048 + lane * 16);
                    cv.Lb = *(const LAS u32x4*)(mc + META_LDS + s * 2048 + 1024 + lane * 16);
                    if (s == 3) { asm volatile("s_waitcnt lgkmcnt(0)" ::: "memory"); ctl[C_FREE + slot] = (unsigned)k + 1u; }
                    unpack_sub(cv, H[s], s, D, it, lanepart, stg, lane);
                }
            }
            if (it.cnt) { WAIT_VM0(); if (lane == 0) __hip_atomic_fetch_add(it.cnt, 1u, __ATOMIC_RELAXED, __HIP_MEMORY_SCOPE_AGENT); }
        }
        return;
    }

    const int g = b - NPULL;
    const int xl = b & 7, slot = g >> 3;
    {
        const rsrc_t rws = mkrsrc(wsl, (unsigned)WS_USED);
        LAS unsigned char* img = lds + wid * 16384;
        LAS unsigned* list = (LAS unsigned*)(lds + STAGE_BYTES + wid * 1024);
        const unsigned long long tstart = __builtin_amdgcn_s_memrealtime();
        constexpr int FAST_C = (NGEMM * 4) / 144, REST_JOBS = (16 - PUSH_C - FAST_C) * 144;
        const bool early = xl < 2 && slot < 16;
        const int li = slot < 16 ? slot * 6 + (xl - 2) : 96 + (slot - 16) * 8 + xl;
        constexpr int NLATE = NGEMM - 32;
        for (int rnd = 0; rnd < 4; ++rnd) {
            int c, j; bool push = false;
            if (rnd == 0) {
                if (wid < 4) { const int f = g * 4 + wid; if (f >= FAST_C * 144) continue; c = PUSH_C + f / 144; j = f % 144; }
                else { const int f = (NGEMM - 1 - g) * 4 + (wid - 4); if (f >= 256 * PUSH_C) continue; push = true; c = f >> 8; j = f & 255; }
            } else {
                if (early) break;
                const int f = (rnd - 1) * (NLATE * 8) + (NLATE - 1 - li) * 8 + (wid ^ 4);
                if (f >= REST_JOBS) break;
                c = PUSH_C + FAST_C + f / 144; j = f % 144;
            }
            {
                const unsigned long long delay = push ? 0ull : (rnd == 0 ? (unsigned long long)((c - PUSH_C) * ENC_STAGGER) : (unsigned long long)ENC_LATE);
                while (__builtin_amdgcn_s_memrealtime() - tstart < delay) __builtin_amdgcn_s_sleep(2);
            }
            if (rnd == 0) { if (push || c == PUSH_C) __builtin_amdgcn_s_setprio(3); else __builtin_amdgcn_s_setprio(1); }
            int is_w, dest, h, w;
            if (push) { is_w = j >> 7; dest = (rank + 1 + ((j >> 4) & 7)) & 7; h = (j >> 3) & 1; w = j & 7; }
            else if (j < 16) { is_w = 0; dest = rank; h = j >> 3; w = j & 7; }
            else { is_w = 1; dest = (rank + 1 + ((j - 16) >> 4)) & 7; h = (j >> 3) & 1; w = j & 7; }
            u32x4 d[16]; unsigned base, s2;
            if (!is_w) {
                const rsrc_t rx = mkrsrc(p.x, (unsigned)(MM * KS * 2));
                const unsigned row0 = 256 * c + 128 * h + 16 * w;
#pragma unroll
                for (int r = 0; r < 16; ++r) d[r] = __builtin_amdgcn_raw_buffer_load_b128(rx, (row0 + r) * (KS * 2) + lane * 16, 0, 0);
                base = (unsigned)OFF_XF + (row0 * KK + KS * rank) * 2 + lane * 16; s2 = 65536;
            } else {
                const rsrc_t rw = mkrsrc(p.w, (unsigned)((size_t)KS * NN * 2));
                const int ng = 2 * h + (w >> 2), kq = w & 3;
                const unsigned k0 = 128 * kq + 8 * (lane >> 3), n0 = NS * dest + 256 * c + 64 * ng + 8 * (lane & 7);
                u32x4 in[16];
#pragma unroll
                for (int i = 0; i < 16; ++i) in[i] = __builtin_amdgcn_raw_buffer_load_b128(rw, ((k0 + 64 * (i >> 3) + (i & 7)) * NN + n0) * 2, 0, 0);
#pragma unroll
                for (int bh = 0; bh < 2; ++bh)
#pragma unroll
                    for (int c8 = 0; c8 < 8; ++c8)
#pragma unroll
                        for (int kk = 0; kk < 4; ++kk)
                            d[bh * 8 + c8][kk] = perm(in[bh * 8 + 2 * kk + 1][c8 >> 1], in[bh * 8 + 2 * kk][c8 >> 1], (c8 & 1) ? 0x07060302u : 0x05040100u);
                base = (unsigned)OFF_WT + ((256 * c + 64 * ng + 8 * (lane & 7)) * KK + KS * rank + k0) * 2; s2 = 128;
            }
            if (!push && !(is_w && dest == rank)) {
#pragma unroll
                for (int r = 0; r < 16; ++r) *(LAS u32x4*)(img + img_off(r, lane)) = d[r];
                CBAR();
                const unsigned so = is_w ? wp_off + (unsigned)((dest * 32 + c * 2 + h) * 8 + w) * SLOT : xp_off + (unsigned)((c * 2 + h) * 8 + w) * SLOT;
                encode_wb<false>(d, rws, so, img, list, lane);
                WAIT_VM0();
                if (is_w ? lane == 0 : (lane < NR && lane != rank))
                    __hip_atomic_store((unsigned*)(p.ws[is_w ? dest : lane] + flag_row + (size_t)rank * NFLAG * 4) + in_unit(is_w, c, h) * 8 + w, call, __ATOMIC_RELAXED, __HIP_MEMORY_SCOPE_SYSTEM);
            }
            if (push || !is_w || dest == rank) {
                const rsrc_t rd = mkrsrc(p.ws[dest], (unsigned)WS_USED);
#pragma unroll
                for (int r = 0; r < 16; ++r) __builtin_amdgcn_raw_buffer_store_b128(d[r], rd, base + (r & 7) * 8192 + (r >> 3) * s2, 0, AUX_SYS);
                WAIT_VM0();
                if (lane == 0) {
                    if (dest == rank) __hip_atomic_fetch_add((unsigned*)(wsl + (is_w ? OFF_WCNT : OFF_XCNT) + c * 128), 1u, __ATOMIC_RELAXED, __HIP_MEMORY_SCOPE_AGENT);
                    else __hip_atomic_fetch_add((unsigned*)(p.ws[dest] + OFF_RCNT + c * 256 + is_w * 128), 1u, __ATOMIC_RELAXED, __HIP_MEMORY_SCOPE_SYSTEM);
                }
            }
            if (rnd == 0) __builtin_amdgcn_s_setprio(0);
        }
    }
    STAMP(p, 1);

    if (slot >= 16) return;
    for (int rnd = 0; rnd < 2; ++rnd) {
        const int blk = xl + 8 * rnd, m = blk * 16 + slot;
        int tm, tn; unit_tile(m, tm, tn);
        __syncthreads();
        bool dead = false;
        if (wid == 0) {
            const bool whole = blk >= SYNC_FROM;
            const int l7 = lane & 7, is_w = l7 >> 2;
            const int ch = is_w ? (whole ? (tn & ~3) + (lane & 3) : tn) : (whole ? (tm & ~3) + (lane & 3) : tm);
            const bool pushed = ch < PUSH_C;
            const unsigned* wp = lane < 8 ? (const unsigned*)(wsl + (is_w ? OFF_WCNT : OFF_XCNT) + ch * 128)
                                          : (lane < 16 ? (const unsigned*)(wsl + OFF_RCNT + (pushed ? ch : 0) * 256 + is_w * 128) : (const unsigned*)(wsl + OFF_PTICK));
            const unsigned want = lane < 8 ? call * (pushed ? 16u : 128u) : (pushed ? call * 112u : 0u);
            const int pace_unit = IN_UNITS + 16 * blk - PACE_LEAD;
            const unsigned tb0 = tick_base(call);
            const unsigned long long t0 = __builtin_amdgcn_s_memrealtime();
            unsigned spins = 0;
            for (;;) {
                unsigned v = 0;
                if (lane < 8 || lane == 16) v = __hip_atomic_load(wp, __ATOMIC_RELAXED, __HIP_MEMORY_SCOPE_AGENT);
                else if (lane < 16 && pushed) v = __hip_atomic_load(wp, __ATOMIC_RELAXED, __HIP_MEMORY_SCOPE_SYSTEM);
                const bool wait = lane < 16 ? v < want : (lane == 16 ? (int)(v - tb0) < pace_unit * 8 : false);
                if (__builtin_amdgcn_ballot_w64(wait) == 0ull) break;
                if ((++spins & 15u) == 0u && __builtin_amdgcn_readfirstlane((int)give_up(tmo, call, t0))) break;
                if (blk < 2) __builtin_amdgcn_s_sleep(2); else __builtin_amdgcn_s_sleep(16);
            }
            if (__builtin_amdgcn_ballot_w64(lane < 8 && pushed)) __builtin_amdgcn_fence(__ATOMIC_ACQUIRE, "");
            else
            __builtin_amdgcn_fence(__ATOMIC_ACQUIRE, "agent");
            WAIT_VM0();
        }
        CBAR(); __builtin_amdgcn_s_barrier(); CBAR();
        STAMP(p, rnd == 0 ? 2 : 5);
        f32x4 acc[2][2][4][2];
        gemm_tile(lds, wsl + OFF_XF + (size_t)tm * 256 * KK * 2, wsl + OFF_WT + (size_t)tn * 256 * KK * 2, acc);
        STAMP(p, rnd == 0 ? 3 : 6);
        dead = __hip_atomic_load(tmo, __ATOMIC_RELAXED, __HIP_MEMORY_SCOPE_AGENT) == call;
        {
            int tz = threadIdx.x; asm volatile("" : "+v"(tz));
            const int ln = tz & 63, wr = wid >> 2, wc = wid & 3, fr = ln & 15, fq = ln >> 4;
#pragma unroll
            for (int ai = 0; ai < 2; ++ai)
#pragma unroll
                for (int mm = 0; mm < 4; ++mm)
#pragma unroll
                    for (int bj = 0; bj < 2; ++bj) {
                        const f32x4 v0 = acc[ai][bj][mm][0], v1 = acc[ai][bj][mm][1];
                        u32x4 w4;
                        w4.x = gelu2(v0[0], v0[1]); w4.y = gelu2(v0[2], v0[3]);
                        w4.z = gelu2(v1[0], v1[1]); w4.w = gelu2(v1[2], v1[3]);
                        const int row = ai * 128 + wr * 64 + mm * 16 + fr, ch = bj * 16 + wc * 4 + fq;
                        *(LAS u32x4*)(lds + row * 512 + ((ch ^ (row & 7)) << 4)) = w4;
                    }
            __syncthreads();
            u32x4 d[16];
#pragma unroll
            for (int r = 0; r < 16; ++r) {
                const int row = wid * 32 + 2 * r + (ln >> 5), ch = ln & 31;
                d[r] = *(const LAS u32x4*)(lds + row * 512 + ((ch ^ (row & 7)) << 4));
            }
            if (dead) {
#pragma unroll
                for (int r = 0; r < 16; ++r) d[r] = (u32x4){0x7fc07fc0u, 0x7fc07fc0u, 0x7fc07fc0u, 0x7fc07fc0u};
            }
            const rsrc_t ro = mkrsrc(p.out, (unsigned)OUT_BYTES);
            const unsigned ob = (unsigned)(((size_t)(256 * tm + 32 * wid + (ln >> 5)) * NN + NS * rank + 256 * tn + 8 * (ln & 31)) * 2);
#pragma unroll
            for (int r = 0; r < 16; ++r) __builtin_amdgcn_raw_buffer_store_b128(d[r], ro, ob + r * (2 * NN * 2), 0, 0);
            const rsrc_t rws = mkrsrc(wsl, (unsigned)WS_USED);
            encode_wb<true>(d, rws, op_off + (unsigned)(m * 8 + wid) * SLOT, lds + wid * 16384, (LAS unsigned*)(lds + STAGE_BYTES + wid * 1024), ln);
            WAIT_VM0();
            __syncthreads();
            if (tid < NR && tid != rank)
                __hip_atomic_store((unsigned*)(p.ws[tid] + flag_row + (size_t)rank * NFLAG * 4) + (IN_UNITS * 8 + m), call, __ATOMIC_RELAXED, __HIP_MEMORY_SCOPE_SYSTEM);
        }
        STAMP(p, rnd == 0 ? 4 : 7);
    }
}

}

extern "C" void kernel_launch(int rank, int nranks, ncclComm_t comm, hipStream_t stream,
                              void* const* d_in, const int* in_sizes, int n_in,
                              void* d_out, int out_size, const int* in_dtypes, int out_dtype,
                              void* const* d_out_all, void* const* d_ws_all, size_t ws_bytes) {
    (void)comm; (void)in_sizes; (void)n_in; (void)out_size; (void)in_dtypes; (void)out_dtype; (void)d_out_all;
    if (nranks != gk::NR || rank < 0 || rank >= gk::NR || ws_bytes < gk::WS_USED) { fprintf(stderr, "unexpected configuration\n"); return; }
    static bool attr_done[64] = {};
    int dev = 0; HIP_CHECK(hipGetDevice(&dev));
    if (dev >= 0 && dev < 64 && !attr_done[dev]) {
        HIP_CHECK(hipFuncSetAttribute((const void*)gk::fused_kernel, hipFuncAttributeMaxDynamicSharedMemorySize, gk::LDS_BYTES));
        attr_done[dev] = true;
    }
    gk::Params p;
    p.x = d_in[0]; p.w = d_in[1]; p.out = d_out;
    for (int i = 0; i < gk::NR; ++i) p.ws[i] = (char*)d_ws_all[i];
    p.rank = rank; p.pad = 0;
    hipLaunchKernelGGL(gk::fused_kernel, dim3(gk::GRID), dim3(gk::NTHR), gk::LDS_BYTES, stream, p);
    HIP_CHECK(hipGetLastError());
}
```

```cpp
#include <hip/hip_runtime.h>
#include <rccl/rccl.h>
#include <cstdio>
#include <type_traits>

#define HIP_CHECK(expr) do { hipError_t _e = (expr); if (_e != hipSuccess) { \
    fprintf(stderr, "HIP error %s at %s:%d\n", hipGetErrorString(_e), __FILE__, __LINE__); \
    return; } } while (0)

namespace gk {

typedef unsigned u32x4 __attribute__((ext_vector_type(4)));
typedef float f32x4 __attribute__((ext_vector_type(4)));
typedef float f32x2 __attribute__((ext_vector_type(2)));
typedef __bf16 bf16x2v __attribute__((ext_vector_type(2)));
typedef short bf16x8 __attribute__((ext_vector_type(8)));
typedef short s16x2 __attribute__((ext_vector_type(2)));
typedef unsigned short u16x2 __attribute__((ext_vector_type(2)));
typedef __amdgpu_buffer_rsrc_t rsrc_t;
#define LAS __attribute__((address_space(3)))

constexpr int NR = 8;
constexpr int MM = 4096, KK = 4096, NN = 32768, KS = 512, NS = 4096;
constexpr int NTHR = 512, GRID = 256;
constexpr int NPP = 16;
constexpr int NPULL = NPP * 7;
constexpr int NGEMM = GRID - NPULL;
constexpr int PUSH_C = 1;
static_assert(NPULL % 8 == 0 && NGEMM / 8 >= 16 && NGEMM * 4 >= 256 * PUSH_C, "16 GEMM workgroups per XCD label");
constexpr int IN_UNITS = 64, OUT_UNITS = 256;
constexpr int NFLAG = IN_UNITS * 8 + OUT_UNITS;
constexpr int ITEM0 = 32 * PUSH_C;
constexpr int TAILT = 4, TAIL0 = (IN_UNITS + OUT_UNITS - TAILT) * 8, NITEMS = TAIL0 + TAILT * 16;
constexpr int NSPLIT = 3;
constexpr int PACE_LEAD = 80;
constexpr int SYNC_FROM = 2;
constexpr int POLL_SLEEP = 8;
constexpr int ENC_STAGGER = 100, ENC_LATE = 1200;
constexpr int ENC_LDS = 131072 + 8192;
constexpr int LDS_BYTES = ENC_LDS + 2112;
constexpr int NDEC = 4;
constexpr int FLIGHT = 4, FLIGHT_IN = 4, FLIGHT_EARLY = 3, EARLY_ITEMS = 128;
constexpr int NSLOT = 8, QN = 2, SLOT_LDS = 12288 + 128;
constexpr int CTL_LDS = 0, LUT_LDS = 1024  , TAB_LDS = LUT_LDS + 6 * 1024, STG_LDS = TAB_LDS + 2048, RING_LDS = STG_LDS + 6 * 4096;
constexpr int C_QTAIL = 0, C_QHEAD = 1, C_KEND = 2, C_Q = 4, C_FULL = 8, C_FREE = 16, C_DESC = 24;
static_assert(RING_LDS + NSLOT * SLOT_LDS <= LDS_BYTES && NDEC <= 6 && FLIGHT >= 2 && FLIGHT <= 8 && C_DESC + NSLOT <= 64, "puller LDS layout");

constexpr unsigned SLOT = 17408;
constexpr unsigned RAW_OFF = 1024, BLOB_MAX = 12288;
constexpr unsigned LEN_OFF = 8, PAT_OFF = 72, PATCH_CAP = 64;
constexpr unsigned MPF_IN = 10880, MPF_OUT = 10240;
constexpr int NP_IN = (MPF_IN + 1023) / 1024, NP_OUT = (MPF_OUT + 1023) / 1024;
static_assert(MPF_IN % 16 == 0 && MPF_OUT % 16 == 0 && MPF_IN <= BLOB_MAX && MPF_OUT <= BLOB_MAX && NP_IN >= 8 && NP_OUT >= 8, "prefetch size");

constexpr size_t OFF_TMO = 256, OFF_XCNT = 4096, OFF_WCNT = 8192, OFF_PTICK = 12288, OFF_RCNT = 14336, OFF_TICKET = 16384;
constexpr size_t OFF_PFLAG = 49152;
constexpr size_t OFF_FLAGS = 65536;
constexpr size_t OFF_XF = 1u << 20;
constexpr size_t OFF_WT = OFF_XF + (size_t)MM * KK * 2;
constexpr size_t OFF_PK = OFF_WT + (size_t)NS * KK * 2;
constexpr size_t XP_BYTES = (size_t)32 * 8 * SLOT;
constexpr size_t WP_BYTES = (size_t)NR * 32 * 8 * SLOT;
constexpr size_t OP_BYTES = (size_t)OUT_UNITS * 8 * SLOT;
constexpr size_t PAR_BYTES = XP_BYTES + WP_BYTES + OP_BYTES;
constexpr size_t OFF_PART = OFF_PK + 2 * PAR_BYTES;
constexpr size_t WS_USED = OFF_PART + (size_t)NSPLIT * 16 * 262144;
constexpr size_t OUT_BYTES = (size_t)MM * NN * 2;
static_assert(WS_USED < (1ull << 31), "32-bit offsets");
constexpr unsigned long long DEADLINE = 100000000ull;

constexpr int AUX_SYS = 17, AUX_LOC = 17;

struct Params {
    const void* x; const void* w; void* out;
    char* ws[NR];
    int rank; int pad;
};

__device__ __forceinline__ rsrc_t mkrsrc(const void* p, unsigned bytes) {
    return __builtin_amdgcn_make_buffer_rsrc((void*)p, 0, bytes, 0x00020000);
}
__device__ __forceinline__ unsigned perm(unsigned s0, unsigned s1, unsigned sel) { return __builtin_amdgcn_perm(s0, s1, sel); }
__device__ __forceinline__ int uni(int v) { return __builtin_amdgcn_readfirstlane(v); }
#define WAIT_VM0() asm volatile("s_waitcnt vmcnt(0)" ::: "memory")
#define CBAR() asm volatile("" ::: "memory")

__device__ __forceinline__ void shell44(int i, int& a, int& b) {
    a = (int)((0x3210333210221010ull >> (4 * i)) & 15);
    b = (int)((0x3333210222101100ull >> (4 * i)) & 15);
}
__device__ __forceinline__ void unit_tile(int m, int& tm, int& tn) {
    int bi, bj, i, j; shell44(m >> 4, bi, bj); shell44(m & 15, i, j);
    tm = bi * 4 + i; tn = bj * 4 + j;
}

__host__ __device__ constexpr int in_unit(int is_w, int c, int h) { return 2 * (c < 4 ? 2 * c + is_w : 8 + ((c >> 2) - 1) * 8 + is_w * 4 + (c & 3)) + h; }
__device__ __forceinline__ void in_unit_inv(int u, int& is_w, int& c, int& h) {
    const int k = u >> 1; h = u & 1;
    if (k < 8) { is_w = k & 1; c = k >> 1; } else { const int g = (k - 8) >> 2; is_w = g & 1; c = 4 * (1 + (g >> 1)) + (k & 3); }
}
static_assert(PUSH_C >= 0 && PUSH_C <= 4 && OFF_RCNT + 4 * 256 <= OFF_TICKET, "pushed chunks");
static_assert(in_unit(0, 4, 0) == 16 && in_unit(1, 4, 1) == 25 && in_unit(0, 8, 0) == 32 && in_unit(1, 15, 1) == 63, "unit order");

__device__ __forceinline__ unsigned tick_base(unsigned call) { return (call - 1u) * (unsigned)(NITEMS - ITEM0 - NPP) - 2u * NPP - ITEM0; }

__device__ __forceinline__ bool give_up(unsigned* tmo, unsigned call, unsigned long long t0) {
    if (__hip_atomic_load(tmo, __ATOMIC_RELAXED, __HIP_MEMORY_SCOPE_AGENT) == call) return true;
    if (__builtin_amdgcn_s_memrealtime() - t0 > DEADLINE) {
        __hip_atomic_store(tmo, call, __ATOMIC_RELAXED, __HIP_MEMORY_SCOPE_AGENT);
        return true;
    }
    return false;
}

__device__ __forceinline__ int robust_top(int e) {
    int t = 0;
#pragma unroll
    for (int bit = 128; bit; bit >>= 1) {
        const int c = t | bit;
        if (__builtin_popcountll(__builtin_amdgcn_ballot_w64(e >= c)) >= 4) t = c;
    }
    return t;
}

__device__ __forceinline__ unsigned img_off(unsigned r, unsigned l) { const unsigned row = 2 * r + (l >> 5); return row * 512 + (((l & 31) ^ (row & 7)) << 4); }

__device__ __forceinline__ unsigned scan_incl(unsigned v) {
    v += __builtin_amdgcn_update_dpp(0u, v, 0x111, 0xf, 0xf, true);
    v += __builtin_amdgcn_update_dpp(0u, v, 0x112, 0xf, 0xf, true);
    v += __builtin_amdgcn_update_dpp(0u, v, 0x114, 0xf, 0xf, true);
    v += __builtin_amdgcn_update_dpp(0u, v, 0x118, 0xf, 0xf, true);
    v += __builtin_amdgcn_update_dpp(0u, v, 0x142, 0xa, 0xf, false);
    v += __builtin_amdgcn_update_dpp(0u, v, 0x143, 0xc, 0xf, false);
    return v;
}
__device__ __forceinline__ unsigned wave_max(unsigned v) {
    v = max(v, (unsigned)__builtin_amdgcn_update_dpp(0u, v, 0x111, 0xf, 0xf, true));
    v = max(v, (unsigned)__builtin_amdgcn_update_dpp(0u, v, 0x112, 0xf, 0xf, true));
    v = max(v, (unsigned)__builtin_amdgcn_update_dpp(0u, v, 0x114, 0xf, 0xf, true));
    v = max(v, (unsigned)__builtin_amdgcn_update_dpp(0u, v, 0x118, 0xf, 0xf, true));
    v = max(v, (unsigned)__builtin_amdgcn_update_dpp(0u, v, 0x142, 0xa, 0xf, false));
    v = max(v, (unsigned)__builtin_amdgcn_update_dpp(0u, v, 0x143, 0xc, 0xf, false));
    return (unsigned)__builtin_amdgcn_readlane((int)v, 63);
}
__device__ const unsigned ENC_TAB[528] = {
    0x800224a4, 0x800224a4, 0x800224a4, 0x800224a4, 0x800224a4, 0x800224a4, 0x800224a4, 0x800224a4, 0x800224a4, 0x800224a4, 0x800224a4, 0x800224a4,
    0x800224a4, 0x800224a4, 0x800224a4, 0x800224a4, 0x800224a4, 0x800224a4, 0x800224a4, 0x800224a4, 0x800224a4, 0x800224a4, 0x800224a4, 0x800224a4,
    0x800224a4, 0x800224a4, 0x800224a4, 0x800224a4, 0x800224a4, 0x800224a4, 0x800224a4, 0x800224a4, 0x800224a4, 0x800224a4, 0x800224a4, 0x800224a4,
    0x800224a4, 0x800224a4, 0x800224a4, 0x800224a4, 0x800224a4, 0x800224a4, 0x800224a4, 0x800224a4, 0x800224a4, 0x800224a4, 0x800224a4, 0x800224a4,
    0x800224a4, 0x800224a4, 0x800224a4, 0x800224a4, 0x800224a4, 0x800224a4, 0x800224a4, 0x800224a4, 0x800224a4, 0x800224a4, 0x800224a4, 0x800224a4,
    0x800224a4, 0x800224a4, 0x800224a4, 0x800224a4, 0x009f3ce8, 0x00ff3ce8, 0x009f3ce8, 0x00ff3ce8, 0x009f3ce8, 0x00ff3ce8, 0x009f3ce8, 0x00ff3ce8,
    0x009f3ce8, 0x00ff3ce8, 0x009f3ce8, 0x00ff3ce8, 0x009f3ce8, 0x00ff3ce8, 0x009f3ce8, 0x00ff3ce8, 0x001f3ce8, 0x007f3ce8, 0x001f3ce8, 0x007f3ce8,
    0x001f3ce8, 0x007f3ce8, 0x001f3ce8, 0x007f3ce8, 0x001f3ce8, 0x007f3ce8, 0x001f3ce8, 0x007f3ce8, 0x001f3ce8, 0x007f3ce8, 0x001f3ce8, 0x007f3ce8,
    0x00ef3ce8, 0x00bf3ce8, 0x00ef3ce8, 0x00bf3ce8, 0x00ef3ce8, 0x00bf3ce8, 0x00ef3ce8, 0x00bf3ce8, 0x00ef3ce8, 0x00bf3ce8, 0x00ef3ce8, 0x00bf3ce8,
    0x00ef3ce8, 0x00bf3ce8, 0x00ef3ce8, 0x00bf3ce8, 0x006f3ce8, 0x003f3ce8, 0x006f3ce8, 0x003f3ce8, 0x006f3ce8, 0x003f3ce8, 0x006f3ce8, 0x003f3ce8,
    0x006f3ce8, 0x003f3ce8, 0x006f3ce8, 0x003f3ce8, 0x006f3ce8, 0x003f3ce8, 0x006f3ce8, 0x003f3ce8, 0x000f38e7, 0x004f38e7, 0x000f38e7, 0x004f38e7,
    0x000f38e7, 0x004f38e7, 0x000f38e7, 0x004f38e7, 0x000f38e7, 0x004f38e7, 0x000f38e7, 0x004f38e7, 0x000f38e7, 0x004f38e7, 0x000f38e7, 0x004f38e7,
    0x003b34e6, 0x001734e6, 0x003b34e6, 0x001734e6, 0x003b34e6, 0x001734e6, 0x003b34e6, 0x001734e6, 0x003b34e6, 0x001734e6, 0x003b34e6, 0x001734e6,
    0x003b34e6, 0x001734e6, 0x003b34e6, 0x001734e6, 0x001d30e5, 0x000b30e5, 0x001d30e5, 0x000b30e5, 0x001d30e5, 0x000b30e5, 0x001d30e5, 0x000b30e5,
    0x001d30e5, 0x000b30e5, 0x001d30e5, 0x000b30e5, 0x001d30e5, 0x000b30e5, 0x001d30e5, 0x000b30e5, 0x00062ce4, 0x00092ce4, 0x00062ce4, 0x00092ce4,
    0x00062ce4, 0x00092ce4, 0x00062ce4, 0x00092ce4, 0x00062ce4, 0x00092ce4, 0x00062ce4, 0x00092ce4, 0x00062ce4, 0x00092ce4, 0x00062ce4, 0x00092ce4,
    0x000a28c4, 0x00012ce4, 0x000a28c4, 0x00012ce4, 0x000a28c4, 0x00012ce4, 0x000a28c4, 0x00012ce4, 0x000d2cc5, 0x00012ce4, 0x000d2cc5, 0x00012ce4,
    0x000d2cc5, 0x00012ce4, 0x000d2cc5, 0x00012ce4, 0x000028e3, 0x000428e3, 0x000028e3, 0x000428e3, 0x000028e3, 0x000428e3, 0x000028e3, 0x000428e3,
    0x000028e3, 0x000428e3, 0x000028e3, 0x000428e3, 0x000028e3, 0x000428e3, 0x000028e3, 0x000428e3, 0x000224a4, 0x000e24a4, 0x000224a4, 0x000e24a4,
    0x000528a5, 0x000328a5, 0x000528a5, 0x000328a5, 0x001528a5, 0x001328a5, 0x001528a5, 0x001328a5, 0x001b2ca6, 0x00272ca6, 0x001b2ca6, 0x00272ca6,
    0x00372c87, 0x00072ca6, 0x00772c87, 0x00072ca6, 0x002f34a8, 0x005f34a8, 0x002f34a8, 0x005f34a8, 0x00af38c8, 0x00df38c8, 0x00af38c8, 0x00df38c8,
    0x00af38c8, 0x00df38c8, 0x00af38c8, 0x00df38c8, 0x800224a4, 0x800224a4, 0x00000000, 0x00000000, 0x00000000, 0x00000000, 0x00000000, 0x00000000,
    0x80021c83, 0x80021c83, 0x80021c83, 0x80021c83, 0x80021c83, 0x80021c83, 0x80021c83, 0x80021c83, 0x80021c83, 0x80021c83, 0x80021c83, 0x80021c83,
    0x80021c83, 0x80021c83, 0x80021c83, 0x80021c83, 0x80021c83, 0x80021c83, 0x80021c83, 0x80021c83, 0x80021c83, 0x80021c83, 0x80021c83, 0x80021c83,
    0x80021c83, 0x80021c83, 0x80021c83, 0x80021c83, 0x80021c83, 0x80021c83, 0x80021c83, 0x80021c83, 0x80021c83, 0x80021c83, 0x80021c83, 0x80021c83,
    0x80021c83, 0x80021c83, 0x80021c83, 0x80021c83, 0x80021c83, 0x80021c83, 0x80021c83, 0x80021c83, 0x80021c83, 0x80021c83, 0x80021c83, 0x80021c83,
    0x009f3ce8, 0x80021c83, 0x009f3ce8, 0x80021c83, 0x009f3ce8, 0x80021c83, 0x009f3ce8, 0x80021c83, 0x009f3ce8, 0x80021c83, 0x009f3ce8, 0x80021c83,
    0x009f3ce8, 0x80021c83, 0x009f3ce8, 0x80021c83, 0x001f3ce8, 0x80021c83, 0x001f3ce8, 0x80021c83, 0x001f3ce8, 0x80021c83, 0x001f3ce8, 0x80021c83,
    0x001f3ce8, 0x80021c83, 0x001f3ce8, 0x80021c83, 0x001f3ce8, 0x80021c83, 0x001f3ce8, 0x80021c83, 0x00ef3ce8, 0x80021c83, 0x00ef3ce8, 0x80021c83,
    0x00ef3ce8, 0x80021c83, 0x00ef3ce8, 0x80021c83, 0x00ef3ce8, 0x80021c83, 0x00ef3ce8, 0x80021c83, 0x00ef3ce8, 0x80021c83, 0x00ef3ce8, 0x80021c83,
    0x006f3ce8, 0x80021c83, 0x006f3ce8, 0x80021c83, 0x006f3ce8, 0x80021c83, 0x006f3ce8, 0x80021c83, 0x006f3ce8, 0x80021c83, 0x006f3ce8, 0x80021c83,
    0x006f3ce8, 0x80021c83, 0x006f3ce8, 0x80021c83, 0x000f38e7, 0x00ff3ce8, 0x000f38e7, 0x00ff3ce8, 0x000f38e7, 0x00ff3ce8, 0x000f38e7, 0x00ff3ce8,
    0x000f38e7, 0x00ff3ce8, 0x000f38e7, 0x00ff3ce8, 0x000f38e7, 0x00ff3ce8, 0x000f38e7, 0x00ff3ce8, 0x000734e6, 0x007f3ce8, 0x000734e6, 0x007f3ce8,
    0x000734e6, 0x007f3ce8, 0x000734e6, 0x007f3ce8, 0x000734e6, 0x007f3ce8, 0x000734e6, 0x007f3ce8, 0x000734e6, 0x007f3ce8, 0x000734e6, 0x007f3ce8,
    0x003b30c6, 0x00bf3ce8, 0x003b30c6, 0x00bf3ce8, 0x003b30c6, 0x00bf3ce8, 0x003b30c6, 0x00bf3ce8, 0x007734c7, 0x00bf3ce8, 0x007734c7, 0x00bf3ce8,
    0x007734c7, 0x00bf3ce8, 0x007734c7, 0x00bf3ce8, 0x001330e5, 0x003f3ce8, 0x001330e5, 0x003f3ce8, 0x001330e5, 0x003f3ce8, 0x001330e5, 0x003f3ce8,
    0x001330e5, 0x003f3ce8, 0x001330e5, 0x003f3ce8, 0x001330e5, 0x003f3ce8, 0x001330e5, 0x003f3ce8, 0x00052ce4, 0x004f38e7, 0x00052ce4, 0x004f38e7,
    0x00052ce4, 0x004f38e7, 0x00052ce4, 0x004f38e7, 0x00052ce4, 0x004f38e7, 0x00052ce4, 0x004f38e7, 0x00052ce4, 0x004f38e7, 0x00052ce4, 0x004f38e7,
    0x000928c4, 0x002734e6, 0x000928c4, 0x002734e6, 0x000928c4, 0x002734e6, 0x000928c4, 0x002734e6, 0x000328a5, 0x002734e6, 0x000328a5, 0x002734e6,
    0x00572c87, 0x002734e6, 0x00372c87, 0x002734e6, 0x000028e3, 0x000d2ce4, 0x000028e3, 0x000d2ce4, 0x000028e3, 0x000d2ce4, 0x000028e3, 0x000d2ce4,
    0x000028e3, 0x000d2ce4, 0x000028e3, 0x000d2ce4, 0x000028e3, 0x000d2ce4, 0x000028e3, 0x000d2ce4, 0x000128c4, 0x000628e3, 0x000128c4, 0x000628e3,
    0x000128c4, 0x000628e3, 0x000128c4, 0x000628e3, 0x002b2886, 0x000628e3, 0x00172c87, 0x000628e3, 0x001b2ca6, 0x000628e3, 0x001b2ca6, 0x000628e3,
    0x000b2ca6, 0x000420a3, 0x000b2ca6, 0x000420a3, 0x002f34a8, 0x00021c83, 0x002f34a8, 0x005f3088, 0x00af38c8, 0x00df38c8, 0x00af38c8, 0x00df38c8,
    0x00af38c8, 0x00df38c8, 0x00af38c8, 0x00df38c8, 0x80021c83, 0x80021c83, 0x00000000, 0x00000000, 0x00000000, 0x00000000, 0x00000000, 0x00000000,
};
__device__ const unsigned DEC_TAB[512] = {
    0xff0038ca, 0x7e80b90b, 0xff802909, 0x7fa0a94a, 0x7f00b8ca, 0xffa0294a, 0xfe00390b, 0x8000a98b, 0xff0038ca, 0x7e00b90b, 0xfe80310a, 0x7d80b94c,
    0x7f00b8ca, 0xfec0314b, 0x7f80a909, 0xfc8039ce, 0xff0038ca, 0x7e80b90b, 0xff802909, 0x7fc0a94a, 0x7f00b8ca, 0xffc0294a, 0xfe00390b, 0x7d00b98d,
    0xff0038ca, 0x7e00b90b, 0xfe80310a, 0xffe0298b, 0x7f00b8ca, 0xfd80394c, 0x7f80a909, 0xfb003a0f, 0xff0038ca, 0x7e80b90b, 0xff802909, 0x7fa0a94a,
    0x7f00b8ca, 0xffa0294a, 0xfe00390b, 0x7fe0a98b, 0xff0038ca, 0x7e00b90b, 0xfe80310a, 0x7d80b94c, 0x7f00b8ca, 0xfec0314b, 0x7f80a909, 0x00202a0d,
    0xff0038ca, 0x7e80b90b, 0xff802909, 0x7fc0a94a, 0x7f00b8ca, 0xffc0294a, 0xfe00390b, 0x000021cb, 0xff0038ca, 0x7e00b90b, 0xfe80310a, 0xfd00398d,
    0x7f00b8ca, 0xfd80394c, 0x7f80a909, 0x7c00ba0f, 0xff0038ca, 0x7e80b90b, 0xff802909, 0x7fa0a94a, 0x7f00b8ca, 0xffa0294a, 0xfe00390b, 0x8000a98b,
    0xff0038ca, 0x7e00b90b, 0xfe80310a, 0x7d80b94c, 0x7f00b8ca, 0xfec0314b, 0x7f80a909, 0x7c80b9ce, 0xff0038ca, 0x7e80b90b, 0xff802909, 0x7fc0a94a,
    0x7f00b8ca, 0xffc0294a, 0xfe00390b, 0x7d00b98d, 0xff0038ca, 0x7e00b90b, 0xfe80310a, 0xffe0298b, 0x7f00b8ca, 0xfd80394c, 0x7f80a909, 0x8020aa0d,
    0xff0038ca, 0x7e80b90b, 0xff802909, 0x7fa0a94a, 0x7f00b8ca, 0xffa0294a, 0xfe00390b, 0x7fe0a98b, 0xff0038ca, 0x7e00b90b, 0xfe80310a, 0x7d80b94c,
    0x7f00b8ca, 0xfec0314b, 0x7f80a909, 0xfc003a0f, 0xff0038ca, 0x7e80b90b, 0xff802909, 0x7fc0a94a, 0x7f00b8ca, 0xffc0294a, 0xfe00390b, 0x001021cb,
    0xff0038ca, 0x7e00b90b, 0xfe80310a, 0xfd00398d, 0x7f00b8ca, 0xfd80394c, 0x7f80a909, 0x7b00ba0f, 0xff0038ca, 0x7e80b90b, 0xff802909, 0x7fa0a94a,
    0x7f00b8ca, 0xffa0294a, 0xfe00390b, 0x8000a98b, 0xff0038ca, 0x7e00b90b, 0xfe80310a, 0x7d80b94c, 0x7f00b8ca, 0xfec0314b, 0x7f80a909, 0xfc8039ce,
    0xff0038ca, 0x7e80b90b, 0xff802909, 0x7fc0a94a, 0x7f00b8ca, 0xffc0294a, 0xfe00390b, 0x7d00b98d, 0xff0038ca, 0x7e00b90b, 0xfe80310a, 0xffe0298b,
    0x7f00b8ca, 0xfd80394c, 0x7f80a909, 0xfa803a0f, 0xff0038ca, 0x7e80b90b, 0xff802909, 0x7fa0a94a, 0x7f00b8ca, 0xffa0294a, 0xfe00390b, 0x7fe0a98b,
    0xff0038ca, 0x7e00b90b, 0xfe80310a, 0x7d80b94c, 0x7f00b8ca, 0xfec0314b, 0x7f80a909, 0x0040320e, 0xff0038ca, 0x7e80b90b, 0xff802909, 0x7fc0a94a,
    0x7f00b8ca, 0xffc0294a, 0xfe00390b, 0x000021cb, 0xff0038ca, 0x7e00b90b, 0xfe80310a, 0xfd00398d, 0x7f00b8ca, 0xfd80394c, 0x7f80a909, 0x7b80ba0f,
    0xff0038ca, 0x7e80b90b, 0xff802909, 0x7fa0a94a, 0x7f00b8ca, 0xffa0294a, 0xfe00390b, 0x8000a98b, 0xff0038ca, 0x7e00b90b, 0xfe80310a, 0x7d80b94c,
    0x7f00b8ca, 0xfec0314b, 0x7f80a909, 0x7c80b9ce, 0xff0038ca, 0x7e80b90b, 0xff802909, 0x7fc0a94a, 0x7f00b8ca, 0xffc0294a, 0xfe00390b, 0x7d00b98d,
    0xff0038ca, 0x7e00b90b, 0xfe80310a, 0xffe0298b, 0x7f00b8ca, 0xfd80394c, 0x7f80a909, 0x8040b20e, 0xff0038ca, 0x7e80b90b, 0xff802909, 0x7fa0a94a,
    0x7f00b8ca, 0xffa0294a, 0xfe00390b, 0x7fe0a98b, 0xff0038ca, 0x7e00b90b, 0xfe80310a, 0x7d80b94c, 0x7f00b8ca, 0xfec0314b, 0x7f80a909, 0xfb803a0f,
    0xff0038ca, 0x7e80b90b, 0xff802909, 0x7fc0a94a, 0x7f00b8ca, 0xffc0294a, 0xfe00390b, 0x001021cb, 0xff0038ca, 0x7e00b90b, 0xfe80310a, 0xfd00398d,
    0x7f00b8ca, 0xfd80394c, 0x7f80a909, 0x7a80ba0f, 0xff0038ca, 0xff80310a, 0x8020a0c7, 0xfec0294a, 0x8000a8c8, 0xfe00390b, 0x7f80b8ca, 0xfc80398d,
    0xff0038ca, 0xfe80310a, 0x8020a0c7, 0x0000298b, 0x8000a8c8, 0x7f00b90b, 0x7f80b8ca, 0xfc0039ce, 0xff0038ca, 0xff80310a, 0x8020a0c7, 0xfd80394c,
    0x8000a8c8, 0xfe00390b, 0x7f80b8ca, 0xffd021cb, 0xff0038ca, 0xfe80310a, 0x8020a0c7, 0xffe0298b, 0x8000a8c8, 0x7f00b90b, 0x7f80b8ca, 0xfa803a0f,
    0xff0038ca, 0xff80310a, 0x8020a0c7, 0xfec0294a, 0x8000a8c8, 0xfe00390b, 0x7f80b8ca, 0x7e80b98d, 0xff0038ca, 0xfe80310a, 0x8020a0c7, 0xffc0218a,
    0x8000a8c8, 0x7f00b90b, 0x7f80b8ca, 0x00202a0d, 0xff0038ca, 0xff80310a, 0x8020a0c7, 0xfd80394c, 0x8000a8c8, 0xfe00390b, 0x7f80b8ca, 0xfef021cb,
    0xff0038ca, 0xfe80310a, 0x8020a0c7, 0xfd00318c, 0x8000a8c8, 0x7f00b90b, 0x7f80b8ca, 0x7d80ba0f, 0xff0038ca, 0xff80310a, 0x8020a0c7, 0xfec0294a,
    0x8000a8c8, 0xfe00390b, 0x7f80b8ca, 0xfc80398d, 0xff0038ca, 0xfe80310a, 0x8020a0c7, 0x0000298b, 0x8000a8c8, 0x7f00b90b, 0x7f80b8ca, 0x7e00b9ce,
    0xff0038ca, 0xff80310a, 0x8020a0c7, 0xfd80394c, 0x8000a8c8, 0xfe00390b, 0x7f80b8ca, 0xfee021cb, 0xff0038ca, 0xfe80310a, 0x8020a0c7, 0xffe0298b,
    0x8000a8c8, 0x7f00b90b, 0x7f80b8ca, 0x8030a20c, 0xff0038ca, 0xff80310a, 0x8020a0c7, 0xfec0294a, 0x8000a8c8, 0xfe00390b, 0x7f80b8ca, 0x7e80b98d,
    0xff0038ca, 0xfe80310a, 0x8020a0c7, 0xffc0218a, 0x8000a8c8, 0x7f00b90b, 0x7f80b8ca, 0xfb803a0f, 0xff0038ca, 0xff80310a, 0x8020a0c7, 0xfd80394c,
    0x8000a8c8, 0xfe00390b, 0x7f80b8ca, 0xfd4031cd, 0xff0038ca, 0xfe80310a, 0x8020a0c7, 0xfd00318c, 0x8000a8c8, 0x7f00b90b, 0x7f80b8ca, 0x7c80ba0f,
    0xff0038ca, 0xff80310a, 0x8020a0c7, 0xfec0294a, 0x8000a8c8, 0xfe00390b, 0x7f80b8ca, 0xfc80398d, 0xff0038ca, 0xfe80310a, 0x8020a0c7, 0x0000298b,
    0x8000a8c8, 0x7f00b90b, 0x7f80b8ca, 0xfc0039ce, 0xff0038ca, 0xff80310a, 0x8020a0c7, 0xfd80394c, 0x8000a8c8, 0xfe00390b, 0x7f80b8ca, 0xffd021cb,
    0xff0038ca, 0xfe80310a, 0x8020a0c7, 0xffe0298b, 0x8000a8c8, 0x7f00b90b, 0x7f80b8ca, 0xfa003a0f, 0xff0038ca, 0xff80310a, 0x8020a0c7, 0xfec0294a,
    0x8000a8c8, 0xfe00390b, 0x7f80b8ca, 0x7e80b98d, 0xff0038ca, 0xfe80310a, 0x8020a0c7, 0xffc0218a, 0x8000a8c8, 0x7f00b90b, 0x7f80b8ca, 0x0040320e,
    0xff0038ca, 0xff80310a, 0x8020a0c7, 0xfd80394c, 0x8000a8c8, 0xfe00390b, 0x7f80b8ca, 0xfef021cb, 0xff0038ca, 0xfe80310a, 0x8020a0c7, 0xfd00318c,
    0x8000a8c8, 0x7f00b90b, 0x7f80b8ca, 0x7d00ba0f, 0xff0038ca, 0xff80310a, 0x8020a0c7, 0xfec0294a, 0x8000a8c8, 0xfe00390b, 0x7f80b8ca, 0xfc80398d,
    0xff0038ca, 0xfe80310a, 0x8020a0c7, 0x0000298b, 0x8000a8c8, 0x7f00b90b, 0x7f80b8ca, 0x7e00b9ce, 0xff0038ca, 0xff80310a, 0x8020a0c7, 0xfd80394c,
    0x8000a8c8, 0xfe00390b, 0x7f80b8ca, 0xfee021cb, 0xff0038ca, 0xfe80310a, 0x8020a0c7, 0xffe0298b, 0x8000a8c8, 0x7f00b90b, 0x7f80b8ca, 0x8040b20e,
    0xff0038ca, 0xff80310a, 0x8020a0c7, 0xfec0294a, 0x8000a8c8, 0xfe00390b, 0x7f80b8ca, 0x7e80b98d, 0xff0038ca, 0xfe80310a, 0x8020a0c7, 0xffc0218a,
    0x8000a8c8, 0x7f00b90b, 0x7f80b8ca, 0xfb003a0f, 0xff0038ca, 0xff80310a, 0x8020a0c7, 0xfd80394c, 0x8000a8c8, 0xfe00390b, 0x7f80b8ca, 0xfd4031cd,
    0xff0038ca, 0xfe80310a, 0x8020a0c7, 0xfd00318c, 0x8000a8c8, 0x7f00b90b, 0x7f80b8ca, 0x7c00ba0f,
};

__device__ __forceinline__ void fetch_enc_tabs(unsigned (&t)[9], int lane) {
#pragma unroll
    for (int i = 0; i < 9; ++i) t[i] = i * 64 + lane < 528 ? ENC_TAB[i * 64 + lane] : 0u;
}
__device__ __forceinline__ void commit_enc_tabs(const unsigned (&t)[9], LAS unsigned char* lds, int lane) {
#pragma unroll
    for (int i = 0; i < 9; ++i) if (i * 64 + lane < 528) ((LAS unsigned*)(lds + ENC_LDS))[i * 64 + lane] = t[i];
}

template <bool IS_OUT>
__device__ __forceinline__ void encode_wb(const u32x4 (&d)[16], rsrc_t dst, unsigned slot_off, LAS unsigned char* lds, LAS unsigned char* img, LAS unsigned* aux, int lane) {
    s16x2 pm = __builtin_bit_cast(s16x2, d[0].x); u16x2 nm = __builtin_bit_cast(u16x2, d[0].x);
#pragma unroll
    for (int r = 0; r < 4; ++r)
#pragma unroll
        for (int j = 0; j < 4; ++j) {
            pm = __builtin_elementwise_max(pm, __builtin_bit_cast(s16x2, d[r][j]));
            nm = __builtin_elementwise_max(nm, __builtin_bit_cast(u16x2, d[r][j]));
        }
    const int p16 = pm.x > pm.y ? pm.x : pm.y;
    const unsigned n16 = nm.x > nm.y ? nm.x : nm.y;
    const int pe = p16 > 0 ? (p16 >> 7) & 255 : 0;
    const int ne = (n16 & 0x8000u) ? (int)(n16 >> 7) & 255 : 0;
    int tp = robust_top(pe), tn = robust_top(ne);
    if (!IS_OUT) { tp = tp > tn ? tp : tn; tn = tp; }
    const unsigned bp2 = (((unsigned)(tp - 15) << 3) & 0xffffu) * 0x00010001u, bn2 = (((unsigned)(tn - 15) << 3) & 0xffffu) * 0x00010001u;
    const LAS unsigned char* tab = lds + ENC_LDS + (IS_OUT ? 264 * 4 : 0);
    unsigned cnt = 0, S = 0, sym[64];
#pragma unroll
    for (int s = 0; s < 4; ++s) {
        unsigned mk = 0;
#pragma unroll
        for (int q0 = 0; q0 < 16; q0 += 8) {
            unsigned e0[8], e1[8];
#pragma unroll
            for (int qq = 0; qq < 8; ++qq) {
                const int q = q0 + qq;
                const unsigned dv = d[4 * s + (q >> 2)][q & 3];
                const unsigned M = __builtin_bit_cast(unsigned, (s16x2)(__builtin_bit_cast(s16x2, dv) >> (s16x2){15, 15}));
                const unsigned bs = IS_OUT ? ((M & bn2) | (~M & bp2)) : bp2;
                const u16x2 u = __builtin_elementwise_min((u16x2)(__builtin_bit_cast(u16x2, (dv >> 4) & 0x07ff07ffu) - __builtin_bit_cast(u16x2, bs)), (u16x2){128, 128});
                const unsigned ib = (__builtin_bit_cast(unsigned, u) << 3) | (M & 0x00040004u);
                e0[qq] = *(const LAS unsigned*)(tab + (ib & 0xffffu));
                e1[qq] = *(const LAS unsigned*)(tab + (ib >> 16));
            }
#pragma unroll
            for (int qq = 0; qq < 8; ++qq) {
                const int q = q0 + qq;
                const unsigned dv = d[4 * s + (q >> 2)][q & 3];
                const unsigned y0 = (__builtin_amdgcn_ubfe(dv, 0, e0[qq] >> 5) << (e0[qq] & 31u)) | __builtin_amdgcn_ubfe(e0[qq], 16, 8);
                const unsigned y1 = (__builtin_amdgcn_ubfe(dv, 16, e1[qq] >> 5) << (e1[qq] & 31u)) | __builtin_amdgcn_ubfe(e1[qq], 16, 8);
                const unsigned t0 = __builtin_amdgcn_ubfe(e0[qq], 10, 4), t = t0 + __builtin_amdgcn_ubfe(e1[qq], 10, 4);
                unsigned sy = (y0 | (y1 << t0)) | (1u << t);
                S += t;
                mk = __builtin_amdgcn_alignbit(mk, e0[qq], 31);
                mk = __builtin_amdgcn_alignbit(mk, e1[qq], 31);
                asm volatile("" : "+v"(sy), "+v"(S), "+v"(mk));
                sym[16 * s + q] = sy;
            }
            __builtin_amdgcn_sched_barrier(0);
        }
        unsigned long long m = __builtin_amdgcn_ballot_w64(mk != 0);
        while (m) {
            const unsigned pos = cnt + __builtin_amdgcn_mbcnt_hi((unsigned)(m >> 32), __builtin_amdgcn_mbcnt_lo((unsigned)m, 0));
            if (mk != 0) {
                const unsigned v = 31u - (unsigned)__builtin_ctz(mk);
                if (pos < PATCH_CAP) aux[pos] = ((unsigned)lane << 23) | ((4 * s + (v >> 3)) << 19) | ((v & 7u) << 16);
                mk &= mk - 1;
            }
            cnt += (unsigned)__builtin_popcountll(m);
            m = __builtin_amdgcn_ballot_w64(mk != 0);
        }
    }
    const unsigned smax = wave_max(S), smin = 0xffffu - wave_max(0xffffu - S);
    const unsigned incl = scan_incl(S);
    const unsigned total = (unsigned)__builtin_amdgcn_readlane((int)incl, 63);
    const unsigned code0 = PAT_OFF + 4 * cnt, mbytes = code0 + ((total + 31) >> 5) * 4;
    if (cnt > PATCH_CAP || smax - smin > 255u || mbytes > BLOB_MAX) {
#pragma unroll 4
        for (int r = 0; r < 16; ++r) __builtin_amdgcn_raw_buffer_store_b128(*(const LAS u32x4*)(img + img_off(r, lane)), dst, slot_off + RAW_OFF + r * 1024 + lane * 16, 0, AUX_SYS);
        if (lane == 0) __builtin_amdgcn_raw_buffer_store_b32(0x80000000u, dst, slot_off, 0, AUX_SYS);
        return;
    }
    unsigned pe_ = 0;
    if ((unsigned)lane < cnt) { const unsigned e = aux[lane]; pe_ = e | *(const LAS unsigned short*)(img + img_off((e >> 19) & 15u, e >> 23) + ((e >> 16) & 7u) * 2); }
    asm volatile("s_waitcnt lgkmcnt(0)" ::: "memory");
    unsigned zero = 0; asm volatile("" : "+v"(zero));
#pragma unroll
    for (int q = 0; q < 12; ++q) if ((unsigned)(q * 1024) < mbytes) *(LAS u32x4*)(img + q * 1024 + lane * 16) = (u32x4){zero, zero, zero, zero};
    CBAR();
    if (lane < 2) ((LAS unsigned*)img)[lane] = lane == 0 ? (cnt | (mbytes << 16)) : (smin | ((unsigned)tp << 16) | ((unsigned)tn << 24));
    img[LEN_OFF + lane] = (unsigned char)(S - smin);
    if ((unsigned)lane < cnt) *(LAS unsigned*)(img + PAT_OFF + 4 * lane) = pe_;
    CBAR();
    {
        const unsigned B = code0 * 8 + incl - S;
        unsigned addr = (B >> 5) * 4, pos = B & 31, acc = 0;
#pragma unroll
        for (int g = 0; g < 64; ++g) {
            const unsigned t = (unsigned)__builtin_clz(sym[g]) ^ 31u;
            const unsigned long long y = (unsigned long long)__builtin_amdgcn_ubfe(sym[g], 0, t) << pos;
            acc |= (unsigned)y;
            pos += t;
            const bool c = pos >= 32;
            __hip_atomic_fetch_or((LAS unsigned*)(img + addr), c ? acc : 0u, __ATOMIC_RELAXED, __HIP_MEMORY_SCOPE_WORKGROUP);
            acc = c ? (unsigned)(y >> 32) : acc;
            addr += c ? 4u : 0u;
            pos &= 31u;
        }
        if (pos) __hip_atomic_fetch_or((LAS unsigned*)(img + addr), acc, __ATOMIC_RELAXED, __HIP_MEMORY_SCOPE_WORKGROUP);
    }
    CBAR();
#pragma unroll
    for (int q = 0; q < 12; ++q) {
        const unsigned o = q * 1024 + lane * 16;
        if (o < mbytes) __builtin_amdgcn_raw_buffer_store_b128(*(const LAS u32x4*)(img + o), dst, slot_off + o, 0, AUX_SYS);
    }
}

struct Item { rsrc_t dr; unsigned so, base, sh, A, B, s1, s2, mpf; unsigned* cnt; };
struct Dec { unsigned cnt, pe, w0, w1, w2, w3, pos, lut; LAS const unsigned char* addr; };

__device__ __forceinline__ unsigned dec2(Dec& D) {
    const unsigned x = __builtin_amdgcn_alignbit(D.w1, D.w0, D.pos);
    const unsigned e0 = *(const LAS unsigned*)(size_t)((x & 0x3fcu) | D.lut);
    const unsigned r0 = __builtin_amdgcn_ubfe(x, e0 >> 6, e0 >> 11);
    const unsigned x1 = x >> (e0 & 31u);
    const unsigned e1 = *(const LAS unsigned*)(size_t)((x1 & 0x3fcu) | D.lut);
    const unsigned r1 = __builtin_amdgcn_ubfe(x1, e1 >> 6, e1 >> 11);
    const unsigned q = D.pos + e0 + e1;
    const bool c = (q & 32u) != 0;
    D.pos = q & 31u;
    D.w0 = c ? D.w1 : D.w0; D.w1 = c ? D.w2 : D.w1; D.w2 = c ? D.w3 : D.w2;
    D.addr += c ? 4 : 0;
    D.w3 = *(const LAS unsigned*)D.addr;
    return perm(e1, e0, 0x07060302u) | r0 | (r1 << 16);
}

__device__ __forceinline__ void store_sub(u32x4 (&o)[4], int s, const Dec& D, const Item& it, unsigned lanepart, LAS unsigned char* stg, int lane) {
    const bool mine = (unsigned)lane < D.cnt && ((D.pe >> 21) & 3u) == (unsigned)s;
    if (__builtin_amdgcn_ballot_w64(mine)) {
#pragma unroll
        for (int rr = 0; rr < 4; ++rr) *(LAS u32x4*)(stg + rr * 1024 + lane * 16) = o[rr];
        CBAR();
        if (mine) *(LAS unsigned short*)(stg + ((D.pe >> 19) & 3u) * 1024 + (D.pe >> 23) * 16 + ((D.pe >> 16) & 7u) * 2) = (unsigned short)D.pe;
        CBAR();
#pragma unroll
        for (int rr = 0; rr < 4; ++rr) o[rr] = *(const LAS u32x4*)(stg + rr * 1024 + lane * 16);
    }
#pragma unroll
    for (int rr = 0; rr < 4; ++rr) {
        const int r = 4 * s + rr;
        __builtin_amdgcn_raw_buffer_store_b128(o[rr], it.dr, lanepart + (r & 7) * it.s1 + (r >> 3) * it.s2, 0, AUX_LOC);
    }
}

constexpr int BK = 64, HALF = 128, HTB = HALF * BK * 2, STAGE_BYTES = 8 * HTB;
__device__ __forceinline__ int lds_byte(int r, int c) { const int st = (r >> 4) * 2 + (c >> 5), rr = r & 15, cc = c & 31, ob = rr * 64 + cc * 2; return st * 1024 + (ob ^ (((ob >> 9) & 1) << 5)); }
__device__ __forceinline__ void stage_rc(int b, int& R, int& C) { const int st = b / 1024, sb = b % 1024, swz = sb ^ (((sb >> 9) & 1) << 5); R = (st >> 1) * 16 + swz / 64; C = (st & 1) * 32 + (swz % 64) / 2; }
__device__ __forceinline__ int perm32(int rho) { const int n = rho >> 4, i = rho & 15; return 8 * (i >> 2) + 4 * n + (i & 3); }

__device__ __forceinline__ unsigned gelu2(float a, float b) {
    const f32x2 y = {a, b};
    const f32x2 c0 = {-2.3022081985f, -2.3022081985f}, c1 = {-0.1029432397f, -0.1029432397f};
    const f32x2 z = y * __builtin_elementwise_fma(y * y, c1, c0);
    f32x2 e; e.x = __builtin_amdgcn_exp2f(z.x); e.y = __builtin_amdgcn_exp2f(z.y);
    const f32x2 dn = e + (f32x2){1.0f, 1.0f};
    f32x2 r; r.x = __builtin_amdgcn_rcpf(dn.x); r.y = __builtin_amdgcn_rcpf(dn.y);
    return __builtin_bit_cast(unsigned, __builtin_convertvector(y * r, bf16x2v));
}

__device__ __forceinline__ void gemm_tile(LAS unsigned char* lds, const char* Abase, const char* Bbase, int nt, f32x4 (&acc)[2][2][4][2], int tid) {
    const int wid = uni(tid >> 6), lane = tid & 63, wr = wid >> 2, wc = wid & 3, fr = lane & 15, fq = lane >> 4;
    constexpr int K = KK;
    unsigned voffA[2], voffB[2];
#pragma unroll
    for (int i = 0; i < 2; ++i) { int R, C; stage_rc(tid * 16 + i * 8192, R, C); const int Rb = (R & ~31) + perm32(R & 31);
        voffA[i] = (unsigned)(R * K + C) * 2u; voffB[i] = (unsigned)(Rb * K + C) * 2u; }
    constexpr size_t kstep = (size_t)(BK * 2), hstep = (size_t)HALF * K * 2;
    const unsigned ldsw = (unsigned)wid * 1024u;
    const int aoff = lds_byte(wr * 64 + fr, fq * 8), boff = lds_byte(wc * 32 + fr, fq * 8);
#define G_SA(b, h) (((b) * 2 + (h)) * HTB)
#define G_SB(b, h) ((4 + (b) * 2 + (h)) * HTB)
#define G_STAGE(bufoff, gbase, voff) do { _Pragma("unroll") for (int _i = 0; _i < 2; ++_i) \
        __builtin_amdgcn_global_load_lds((const unsigned*)((const char*)(gbase) + (voff)[_i]), (LAS unsigned*)(lds + (bufoff) + ldsw + _i * 8192), 16, 0, 0); } while (0)
#define G_LDA(dst, b, h) do { _Pragma("unroll") for (int m = 0; m < 4; ++m) _Pragma("unroll") for (int k = 0; k < 2; ++k) dst[m][k] = *(const LAS bf16x8*)(lds + G_SA(b, h) + aoff + m * 2048 + k * 1024); } while (0)
#define G_LDB(dst, b, h) do { _Pragma("unroll") for (int n = 0; n < 2; ++n) _Pragma("unroll") for (int k = 0; k < 2; ++k) dst[n][k] = *(const LAS bf16x8*)(lds + G_SB(b, h) + boff + n * 2048 + k * 1024); } while (0)
#define G_MMA(ai, bj, At, Bt) do { __builtin_amdgcn_s_setprio(1); _Pragma("unroll") for (int m = 0; m < 4; ++m) _Pragma("unroll") for (int n = 0; n < 2; ++n) _Pragma("unroll") for (int k = 0; k < 2; ++k) \
        acc[ai][bj][m][n] = __builtin_amdgcn_mfma_f32_16x16x32_bf16(Bt[n][k], At[m][k], acc[ai][bj][m][n], 0, 0, 0); __builtin_amdgcn_s_setprio(0); } while (0)
#define G_WAIT_V(n) asm volatile("s_waitcnt vmcnt(" #n ")" ::: "memory")
#define G_WAIT_L(n) asm volatile("s_waitcnt lgkmcnt(" #n ")" ::: "memory")
#define G_BAR __builtin_amdgcn_s_barrier()
#define G_SCHED __builtin_amdgcn_sched_barrier(0)
#pragma unroll
    for (int a = 0; a < 2; ++a)
#pragma unroll
        for (int b = 0; b < 2; ++b)
#pragma unroll
            for (int m = 0; m < 4; ++m)
#pragma unroll
                for (int n = 0; n < 2; ++n) acc[a][b][m][n] = (f32x4){0.f, 0.f, 0.f, 0.f};
    bf16x8 At[4][2], B0[2][2], B1[2][2];
    const char* cA = Abase; const char* cB = Bbase;
    G_STAGE(G_SB(0, 0), cB, voffB); G_STAGE(G_SA(0, 0), cA, voffA); G_STAGE(G_SB(0, 1), cB + hstep, voffB); G_STAGE(G_SA(0, 1), cA + hstep, voffA);
    if (wr == 1) G_BAR;
    G_WAIT_V(4); G_BAR;
    G_STAGE(G_SB(1, 0), cB + kstep, voffB); G_STAGE(G_SA(1, 0), cA + kstep, voffA); G_STAGE(G_SB(1, 1), cB + hstep + kstep, voffB);
    G_WAIT_V(6); G_BAR;
    for (int t = 0; t < nt; t += 2) {
        const bool last = (t == nt - 2);
        const char* a1 = cA + (size_t)(t + 1) * kstep;
        const char* a2 = last ? cA : cA + (size_t)(t + 2) * kstep; const char* b2 = last ? cB : cB + (size_t)(t + 2) * kstep;
        const char* a3 = a2 + kstep; const char* b3 = b2 + kstep;
        G_LDB(B0, 0, 0); G_SCHED; G_LDA(At, 0, 0); G_STAGE(G_SA(1, 1), a1 + hstep, voffA);
        G_WAIT_L(8); G_BAR; G_WAIT_L(0); G_MMA(0, 0, At, B0); G_BAR; G_SCHED;
        G_LDB(B1, 0, 1); G_STAGE(G_SB(0, 0), b2, voffB);
        G_BAR; G_WAIT_L(0); G_MMA(0, 1, At, B1); G_BAR;
        G_LDA(At, 0, 1); G_STAGE(G_SA(0, 0), a2, voffA);
        G_BAR; G_WAIT_L(0); G_MMA(1, 0, At, B0); G_BAR; G_SCHED;
        G_STAGE(G_SB(0, 1), b2 + hstep, voffB);
        G_WAIT_V(6); G_BAR; G_MMA(1, 1, At, B1); G_BAR;
        G_LDB(B0, 1, 0); G_SCHED; G_LDA(At, 1, 0); G_STAGE(G_SA(0, 1), a2 + hstep, voffA);
        G_WAIT_L(8); G_BAR; G_WAIT_L(0); G_MMA(0, 0, At, B0); G_BAR; G_SCHED;
        G_LDB(B1, 1, 1); G_STAGE(G_SB(1, 0), b3, voffB);
        G_BAR; G_WAIT_L(0); G_MMA(0, 1, At, B1); G_BAR;
        G_LDA(At, 1, 1); G_STAGE(G_SA(1, 0), a3, voffA);
        G_BAR; G_WAIT_L(0); G_MMA(1, 0, At, B0); G_BAR; G_SCHED;
        G_STAGE(G_SB(1, 1), b3 + hstep, voffB);
        G_WAIT_V(6); G_BAR; G_MMA(1, 1, At, B1); G_BAR;
    }
    G_WAIT_V(0);
    if (wr == 0) G_BAR;
    G_BAR;
#undef G_SA
#undef G_SB
#undef G_STAGE
#undef G_LDA
#undef G_LDB
#undef G_MMA
}


__global__ __launch_bounds__(NTHR, 2) void fused_kernel(Params p) {
    extern __shared__ __attribute__((aligned(16))) unsigned char shm[];
    LAS unsigned char* lds = (LAS unsigned char*)shm;
    const int tid = threadIdx.x, wid = uni(tid >> 6), lane = tid & 63;
    const int rank = p.rank;
    char* const wsl = p.ws[rank];
    unsigned* const tmo = (unsigned*)(wsl + OFF_TMO);
    unsigned* const my_ctr = (unsigned*)(wsl + OFF_TICKET + (size_t)blockIdx.x * 128 + wid * 16);
    const unsigned ctr_raw = __hip_atomic_load(my_ctr, __ATOMIC_RELAXED, __HIP_MEMORY_SCOPE_AGENT);
    unsigned call = 0, xp_off = 0, wp_off = 0, op_off = 0, flag_row = 0;
    auto resolve = [&]() {
        call = (unsigned)uni((int)ctr_raw) + 1u;
        if (lane == 0) __hip_atomic_store(my_ctr, call, __ATOMIC_RELAXED, __HIP_MEMORY_SCOPE_AGENT);
        const unsigned par = call & 1u;
        xp_off = (unsigned)(OFF_PK + par * PAR_BYTES); wp_off = xp_off + (unsigned)XP_BYTES; op_off = wp_off + (unsigned)WP_BYTES;
        flag_row = (unsigned)(OFF_FLAGS + ((size_t)par * NR) * NFLAG * 4);
    };
    const int b = blockIdx.x;

    if (b < NPULL) {
        volatile LAS unsigned* const ctl = (volatile LAS unsigned*)(lds + CTL_LDS);
        if (tid < 64) ctl[tid] = tid == C_KEND ? 0x7fffffffu : 0u;
        ((LAS unsigned*)(lds + TAB_LDS))[tid] = DEC_TAB[tid];
        __syncthreads();
        if (wid >= 2 + NDEC) return;
        resolve();
        const int pi = b % 7, sub = b / 7;
        const int src = (rank + 1 + pi) & 7;
        const rsrc_t rs = mkrsrc(p.ws[src], (unsigned)WS_USED);
        auto item_so = [&](int i) -> unsigned {
            if (i >= TAIL0) return op_off + (unsigned)((OUT_UNITS - TAILT) * 8 + ((i - TAIL0) >> 1)) * SLOT + RAW_OFF + ((i - TAIL0) & 1) * 8192;
            const int w = i & 7, u = i >> 3;
            if (u < IN_UNITS) {
                int is_w, c, h; in_unit_inv(u, is_w, c, h);
                return is_w ? wp_off + (unsigned)((rank * 32 + c * 2 + h) * 8 + w) * SLOT : xp_off + (unsigned)((c * 2 + h) * 8 + w) * SLOT;
            }
            return op_off + (unsigned)((u - IN_UNITS) * 8 + w) * SLOT;
        };
        if (wid == 1) {
            const unsigned* fl = (const unsigned*)(wsl + flag_row + (size_t)src * NFLAG * 4);
            unsigned* const tick = (unsigned*)(wsl + OFF_PTICK + pi * 128);
            const unsigned tbase = tick_base(call);
            int known = 0; bool dead = false;
            for (int n = 0;; ++n) {
                int item;
                if (n < 2) item = ITEM0 + sub + n * NPP;
                else { unsigned tk = 0; if (lane == 0) tk = __hip_atomic_fetch_add(tick, 1u, __ATOMIC_RELAXED, __HIP_MEMORY_SCOPE_AGENT); item = (int)((unsigned)uni((int)tk) - tbase); }
                unsigned e = 0xffffffffu;
                if (item < NITEMS) {
                    const int u = item < IN_UNITS * 8 ? item : (item < TAIL0 ? IN_UNITS * 8 + ((item - IN_UNITS * 8) >> 3) : IN_UNITS * 8 + OUT_UNITS - TAILT + ((item - TAIL0) >> 4));
                    if (u >= known && !dead) {
                        const unsigned long long t0 = __builtin_amdgcn_s_memrealtime();
                        for (unsigned spins = 0;; ++spins) {
                            const int uu = u + lane;
                            const unsigned f = uu < NFLAG ? __hip_atomic_load(fl + uu, __ATOMIC_RELAXED, __HIP_MEMORY_SCOPE_SYSTEM) : 0u;
                            const unsigned long long mk = __builtin_amdgcn_ballot_w64(f >= call);
                            known = u + __builtin_ctzll(~mk | (1ull << 63));
                            if (known > u) break;
                            if ((spins & 15u) == 15u && __builtin_amdgcn_readfirstlane((int)give_up(tmo, call, t0))) { dead = true; break; }
                            __builtin_amdgcn_s_sleep(POLL_SLEEP);
                        }
                    }
                    e = (unsigned)item | (dead ? 0x40000000u : 0u);
                }
                while ((unsigned)n - ctl[C_QHEAD] >= QN) __builtin_amdgcn_s_sleep(2);
                CBAR();
                if (lane == 0) { ctl[C_Q + (n % QN)] = e; asm volatile("s_waitcnt lgkmcnt(0)" ::: "memory"); ctl[C_QTAIL] = (unsigned)n + 1u; }
                if (item >= NITEMS) break;
            }
            return;
        }
        if (wid == 0) {
            __builtin_amdgcn_s_setprio(3);
            int pend = -1;
            for (int k = 0;; ++k) {
                const int slot = k % NSLOT;
                if (ctl[C_QTAIL] <= (unsigned)k || (k >= NSLOT && ctl[C_FREE + slot] < (unsigned)(k - NSLOT + 1))) {
                    WAIT_VM0();
                    if (pend >= 0) { ctl[C_FULL + pend % NSLOT] = (unsigned)pend + 1u; pend = -1; }
                    while (ctl[C_QTAIL] <= (unsigned)k) __builtin_amdgcn_s_sleep(1);
                    while (k >= NSLOT && ctl[C_FREE + slot] < (unsigned)(k - NSLOT + 1)) __builtin_amdgcn_s_sleep(1);
                }
                CBAR();
                const unsigned e = (unsigned)uni((int)ctl[C_Q + (k % QN)]);
                asm volatile("s_waitcnt lgkmcnt(0)" ::: "memory");
                ctl[C_QHEAD] = (unsigned)k + 1u;
                if (e == 0xffffffffu) {
                    WAIT_VM0();
                    if (pend >= 0) ctl[C_FULL + pend % NSLOT] = (unsigned)pend + 1u;
                    ctl[C_KEND] = (unsigned)k;
                    break;
                }
                ctl[C_DESC + slot] = e;
                if (e & 0x40000000u) {
                    WAIT_VM0();
                    if (pend >= 0) { ctl[C_FULL + pend % NSLOT] = (unsigned)pend + 1u; pend = -1; }
                    asm volatile("s_waitcnt lgkmcnt(0)" ::: "memory");
                    ctl[C_FULL + slot] = (unsigned)k + 1u;
                    continue;
                }
                const int item = (int)e;
                const unsigned so = item_so(item) + lane * 16;
                const unsigned mpf = item < IN_UNITS * 8 ? MPF_IN : (item < TAIL0 ? MPF_OUT : 8192u);
                LAS unsigned char* sb = lds + RING_LDS + slot * SLOT_LDS;
                auto fill = [&](auto flc, auto npc) {
                    constexpr int FL = decltype(flc)::value, NP = decltype(npc)::value;
#pragma unroll
                    for (int j = 0; j < NP; ++j) {
                        if (j < NP - 1 || (unsigned)(j * 1024 + lane * 16) < mpf)
                            __builtin_amdgcn_raw_ptr_buffer_load_lds(rs, (LAS void*)(sb + j * 1024), 16, so + j * 1024, 0, 0, AUX_SYS);
                        asm volatile("s_waitcnt vmcnt(%0)" :: "n"(FL - 1) : "memory");
                        if (j >= FL - 1 && pend >= 0) { ctl[C_FULL + pend % NSLOT] = (unsigned)pend + 1u; pend = -1; }
                    }
                };
                if (item < EARLY_ITEMS) fill(std::integral_constant<int, FLIGHT_EARLY>{}, std::integral_constant<int, NP_IN>{});
                else if (item < IN_UNITS * 8) fill(std::integral_constant<int, FLIGHT_IN>{}, std::integral_constant<int, NP_IN>{});
                else if (item < TAIL0) fill(std::integral_constant<int, FLIGHT>{}, std::integral_constant<int, NP_OUT>{});
                else fill(std::integral_constant<int, FLIGHT>{}, std::integral_constant<int, 8>{});
                pend = k;
            }
            return;
        }
        const int dw = wid - 2;
        LAS unsigned char* stg = lds + STG_LDS + dw * 4096;
        LAS unsigned char* lut = lds + LUT_LDS + dw * 1024;
        unsigned lut_key = 0xffffffffu;
        for (int k = dw;; k += NDEC) {
            const int slot = k % NSLOT;
            bool stop = false;
            while (ctl[C_FULL + slot] < (unsigned)k + 1u) {
                if (ctl[C_KEND] <= (unsigned)k) { stop = true; break; }
                __builtin_amdgcn_s_sleep(1);
            }
            if (stop) break;
            CBAR();
            const unsigned e = (unsigned)uni((int)ctl[C_DESC + slot]);
            const int cur0 = (int)(e & 0x3fffffffu);
            const bool tail = cur0 >= TAIL0;
            const int cur = tail ? (IN_UNITS + OUT_UNITS - TAILT) * 8 + ((cur0 - TAIL0) >> 1) : cur0;
            Item it;
            {
                const int w = cur & 7, u = cur >> 3;
                it.so = item_so(cur0);
                if (u < IN_UNITS) {
                    int is_w, c, h; in_unit_inv(u, is_w, c, h);
                    it.dr = mkrsrc(wsl + OFF_XF, (unsigned)(OFF_PK - OFF_XF));
                    it.s1 = 8192; it.mpf = MPF_IN;
                    if (!is_w) {
                        it.base = (unsigned)(((256 * c + 128 * h + 16 * w) * KK + KS * src) * 2);
                        it.sh = 6; it.A = 0; it.B = 16; it.s2 = 65536;
                        it.cnt = (unsigned*)(wsl + OFF_XCNT + c * 128);
                    } else {
                        it.base = (unsigned)(MM * KK * 2) + (unsigned)(((256 * c + 64 * (2 * h + (w >> 2))) * KK + KS * src + 128 * (w & 3)) * 2);
                        it.sh = 3; it.A = 16; it.B = 65536; it.s2 = 128;
                        it.cnt = (unsigned*)(wsl + OFF_WCNT + c * 128);
                    }
                } else {
                    const int m = u - IN_UNITS; int tm, tn; unit_tile(m, tm, tn);
                    it.dr = mkrsrc(p.out, (unsigned)OUT_BYTES);
                    it.base = (unsigned)(((size_t)(256 * tm + 32 * w) * NN + NS * src + 256 * tn) * 2);
                    it.sh = 5; it.A = 65536; it.B = 16; it.s1 = 131072; it.s2 = 1048576;
                    it.cnt = nullptr; it.mpf = MPF_OUT;
                }
            }
            const unsigned lanepart = it.base + ((unsigned)lane >> it.sh) * it.A + ((unsigned)lane & ((1u << it.sh) - 1u)) * it.B;
            LAS unsigned char* mc = lds + RING_LDS + slot * SLOT_LDS;
            const unsigned hdr = ((e & 0x40000000u) || tail) ? 0u : (unsigned)uni((int)*(const LAS unsigned*)mc);
            if (e & 0x40000000u) {
                ctl[C_FREE + slot] = (unsigned)k + 1u;
                if (it.cnt == nullptr) {
                    for (int r = 0; r < 16; ++r)
                        __builtin_amdgcn_raw_buffer_store_b128((u32x4){0x7fc07fc0u, 0x7fc07fc0u, 0x7fc07fc0u, 0x7fc07fc0u}, it.dr, lanepart + (r & 7) * it.s1 + (r >> 3) * it.s2, 0, 0);
                }
            } else if (tail) {
                u32x4 x[8];
#pragma unroll
                for (int r = 0; r < 8; ++r) x[r] = *(const LAS u32x4*)(mc + r * 1024 + lane * 16);
                asm volatile("s_waitcnt lgkmcnt(0)" ::: "memory");
                ctl[C_FREE + slot] = (unsigned)k + 1u;
                const unsigned hb = lanepart + (unsigned)((cur0 - TAIL0) & 1) * it.s2;
#pragma unroll
                for (int r = 0; r < 8; ++r) __builtin_amdgcn_raw_buffer_store_b128(x[r], it.dr, hb + r * it.s1, 0, AUX_LOC);
            } else if (hdr & 0x80000000u) {
                ctl[C_FREE + slot] = (unsigned)k + 1u;
#pragma unroll 1
                for (int r0 = 0; r0 < 16; r0 += 8) {
                    u32x4 x[8];
#pragma unroll
                    for (int r = 0; r < 8; ++r) x[r] = __builtin_amdgcn_raw_buffer_load_b128(rs, it.so + RAW_OFF + (r0 + r) * 1024 + lane * 16, 0, AUX_SYS);
#pragma unroll
                    for (int r = 0; r < 8; ++r) __builtin_amdgcn_raw_buffer_store_b128(x[r], it.dr, lanepart + ((r0 + r) & 7) * it.s1 + ((r0 + r) >> 3) * it.s2, 0, AUX_LOC);
                }
            } else {
                Dec D;
                const unsigned mbytes = (hdr >> 16) & 0x3fffu;
                if (mbytes > it.mpf) {
                    for (unsigned q = it.mpf >> 10; q < 12; ++q) {
                        const unsigned o = q * 1024 + lane * 16;
                        if (o >= it.mpf && o < mbytes) *(LAS u32x4*)(mc + o) = __builtin_amdgcn_raw_buffer_load_b128(rs, it.so + o, 0, AUX_SYS);
                    }
                    CBAR();
                }
                const unsigned h1 = (unsigned)uni((int)*(const LAS unsigned*)(mc + 4));
                const unsigned key = (h1 >> 16) | (cur < IN_UNITS * 8 ? 0u : 0x10000u);
                if (key != lut_key) {
                    lut_key = key;
                    u32x4 t = *(const LAS u32x4*)(lds + TAB_LDS + (key >> 16) * 1024 + lane * 16);
                    const unsigned tp = (h1 >> 16) & 0xffu, tn = h1 >> 24;
#pragma unroll
                    for (int i = 0; i < 4; ++i) t[i] = (t[i] & ~0x8000u) + (2u << 6) + (((t[i] & 0x8000u) ? tn : tp) << 23);
                    *(LAS u32x4*)(lut + lane * 16) = t;
                    CBAR();
                }
                D.cnt = hdr & 0xffu; D.lut = (unsigned)(size_t)lut;
                const unsigned bits = (h1 & 0xffffu) + mc[LEN_OFF + lane];
                const unsigned B = (PAT_OFF + 4 * D.cnt) * 8 + scan_incl(bits) - bits - 2;
                const LAS unsigned* wp = (const LAS unsigned*)(mc + (B >> 5) * 4);
                D.pos = B & 31; D.w0 = wp[0]; D.w1 = wp[1]; D.w2 = wp[2]; D.w3 = wp[3]; D.addr = (const LAS unsigned char*)(wp + 3);
                D.pe = *(const LAS unsigned*)(mc + PAT_OFF + 4 * lane);
#pragma unroll
                for (int s = 0; s < 4; ++s) {
                    u32x4 o[4];
#pragma unroll
                    for (int rr = 0; rr < 4; ++rr)
#pragma unroll
                        for (int j = 0; j < 4; ++j) o[rr][j] = dec2(D);
                    if (s == 3) { asm volatile("s_waitcnt lgkmcnt(0)" ::: "memory"); ctl[C_FREE + slot] = (unsigned)k + 1u; }
                    store_sub(o, s, D, it, lanepart, stg, lane);
                }
            }
            if (it.cnt) { WAIT_VM0(); if (lane == 0) __hip_atomic_fetch_add(it.cnt, 1u, __ATOMIC_RELAXED, __HIP_MEMORY_SCOPE_AGENT); }
        }
        return;
    }

    const int g = b - NPULL;
    const int xl = b & 7, slot = g >> 3;
    {
        const rsrc_t rws = mkrsrc(wsl, (unsigned)WS_USED);
        LAS unsigned char* img = lds + wid * 16384;
        LAS unsigned* list = (LAS unsigned*)(lds + STAGE_BYTES + wid * 1024);
        const unsigned long long tstart = __builtin_amdgcn_s_memrealtime();
        unsigned tabs[9]; fetch_enc_tabs(tabs, lane);
        bool first = true;
        constexpr int FAST_C = (NGEMM * 4) / 144, REST_JOBS = (16 - PUSH_C - FAST_C) * 144;
        constexpr int HX0 = 8 - NSPLIT;
        const bool early = slot < 16 && (xl < 2 || (NSPLIT > 0 && xl == HX0));
        constexpr int NE = NSPLIT > 0 ? 3 : 2;
        const int xr = xl < 2 ? 0 : (NSPLIT > 0 && xl > HX0 ? xl - 3 : xl - 2);
        const int li = slot < 16 ? slot * (8 - NE) + xr : 16 * (8 - NE) + (slot - 16) * 8 + xl;
        constexpr int NLATE = NGEMM - 16 * NE;
        static_assert(NSPLIT == 0 || (HX0 >= 2 && NSPLIT <= 6), "helpers");
        for (int rnd = 0; rnd < 5; ++rnd) {
            int c, j; bool push = false;
            if (rnd == 0) {
                if (wid < 4) { const int f = g * 4 + wid; if (f >= FAST_C * 144) continue; c = PUSH_C + f / 144; j = f % 144; }
                else { const int f = (NGEMM - 1 - g) * 4 + (wid - 4); if (f >= 256 * PUSH_C) continue; push = true; c = f >> 8; j = f & 255; }
            } else {
                if (early) break;
                const int f = (rnd - 1) * (NLATE * 8) + (NLATE - 1 - li) * 8 + (wid ^ 4);
                if (f >= REST_JOBS) break;
                c = PUSH_C + FAST_C + f / 144; j = f % 144;
            }
            {
                const unsigned long long delay = push ? 0ull : (rnd == 0 ? (unsigned long long)((c - PUSH_C) * ENC_STAGGER) : (unsigned long long)ENC_LATE);
                while (__builtin_amdgcn_s_memrealtime() - tstart < delay) __builtin_amdgcn_s_sleep(2);
            }
            if (rnd == 0) { if (push || c == PUSH_C) __builtin_amdgcn_s_setprio(3); else __builtin_amdgcn_s_setprio(1); }
            int is_w, dest, h, w;
            if (push) { is_w = j >> 7; dest = (rank + 1 + ((j >> 4) & 7)) & 7; h = (j >> 3) & 1; w = j & 7; }
            else if (j < 16) { is_w = 0; dest = rank; h = j >> 3; w = j & 7; }
            else { is_w = 1; dest = (rank + 1 + ((j - 16) >> 4)) & 7; h = (j >> 3) & 1; w = j & 7; }
            u32x4 d[16]; unsigned base, s2;
            if (!is_w) {
                const rsrc_t rx = mkrsrc(p.x, (unsigned)(MM * KS * 2));
                const unsigned row0 = 256 * c + 128 * h + 16 * w;
#pragma unroll
                for (int r = 0; r < 16; ++r) d[r] = __builtin_amdgcn_raw_buffer_load_b128(rx, (row0 + r) * (KS * 2) + lane * 16, 0, 0);
                base = (unsigned)OFF_XF + (row0 * KK + KS * rank) * 2 + lane * 16; s2 = 65536;
            } else {
                const rsrc_t rw = mkrsrc(p.w, (unsigned)((size_t)KS * NN * 2));
                const int ng = 2 * h + (w >> 2), kq = w & 3;
                const unsigned k0 = 128 * kq + 8 * (lane >> 3), n0 = NS * dest + 256 * c + 64 * ng + 8 * (lane & 7);
                u32x4 in[16];
#pragma unroll
                for (int i = 0; i < 16; ++i) in[i] = __builtin_amdgcn_raw_buffer_load_b128(rw, ((k0 + 64 * (i >> 3) + (i & 7)) * NN + n0) * 2, 0, 0);
#pragma unroll
                for (int bh = 0; bh < 2; ++bh)
#pragma unroll
                    for (int c8 = 0; c8 < 8; ++c8)
#pragma unroll
                        for (int kk = 0; kk < 4; ++kk)
                            d[bh * 8 + c8][kk] = perm(in[bh * 8 + 2 * kk + 1][c8 >> 1], in[bh * 8 + 2 * kk][c8 >> 1], (c8 & 1) ? 0x07060302u : 0x05040100u);
                base = (unsigned)OFF_WT + ((256 * c + 64 * ng + 8 * (lane & 7)) * KK + KS * rank + k0) * 2; s2 = 128;
            }
            if (first) { first = false; resolve(); commit_enc_tabs(tabs, lds, lane); }
            const bool rawcopy = push || !is_w || dest == rank, pack = !push && !(is_w && dest == rank);
            if (rawcopy) {
                const rsrc_t rd = mkrsrc(p.ws[dest], (unsigned)WS_USED);
#pragma unroll
                for (int r = 0; r < 16; ++r) __builtin_amdgcn_raw_buffer_store_b128(d[r], rd, base + (r & 7) * 8192 + (r >> 3) * s2, 0, AUX_SYS);
            }
            if (pack) {
#pragma unroll
                for (int r = 0; r < 16; ++r) *(LAS u32x4*)(img + img_off(r, lane)) = d[r];
                CBAR();
                const unsigned so = is_w ? wp_off + (unsigned)((dest * 32 + c * 2 + h) * 8 + w) * SLOT : xp_off + (unsigned)((c * 2 + h) * 8 + w) * SLOT;
                encode_wb<false>(d, rws, so, lds, img, list, lane);
            }
            WAIT_VM0();
            if (pack && (is_w ? lane == 0 : (lane < NR && lane != rank)))
                __hip_atomic_store((unsigned*)(p.ws[is_w ? dest : lane] + flag_row + (size_t)rank * NFLAG * 4) + in_unit(is_w, c, h) * 8 + w, call, __ATOMIC_RELAXED, __HIP_MEMORY_SCOPE_SYSTEM);
            if (rawcopy && lane == 0) {
                if (dest == rank) __hip_atomic_fetch_add((unsigned*)(wsl + (is_w ? OFF_WCNT : OFF_XCNT) + c * 128), 1u, __ATOMIC_RELAXED, __HIP_MEMORY_SCOPE_AGENT);
                else __hip_atomic_fetch_add((unsigned*)(p.ws[dest] + OFF_RCNT + c * 256 + is_w * 128), 1u, __ATOMIC_RELAXED, __HIP_MEMORY_SCOPE_SYSTEM);
            }
            if (rnd == 0) __builtin_amdgcn_s_setprio(0);
        }
        if (first) { resolve(); commit_enc_tabs(tabs, lds, lane); }
    }

    if (slot >= 16) return;
    for (int rnd = -1; rnd < 2; ++rnd) {
        const bool helper = rnd < 0;
        if (helper && xl < 8 - NSPLIT) continue;
        const int blk = helper ? xl - (8 - NSPLIT) : xl + 8 * rnd, m = blk * 16 + slot;
        const bool split = blk < NSPLIT;
        int tm, tn; unit_tile(m, tm, tn);
        int tid = threadIdx.x; asm volatile("" : "+v"(tid));
        const int lane = tid & 63;
        __syncthreads();
        bool dead = false;
        if (wid == 0) {
            const bool whole = blk >= SYNC_FROM;
            const int l7 = lane & 7, is_w = l7 >> 2;
            const int ch = is_w ? (whole ? (tn & ~3) + (lane & 3) : tn) : (whole ? (tm & ~3) + (lane & 3) : tm);
            const bool pushed = ch < PUSH_C;
            const unsigned* wp = lane < 8 ? (const unsigned*)(wsl + (is_w ? OFF_WCNT : OFF_XCNT) + ch * 128)
                                          : (lane < 16 ? (const unsigned*)(wsl + OFF_RCNT + (pushed ? ch : 0) * 256 + is_w * 128) : (const unsigned*)(wsl + OFF_PTICK));
            const unsigned want = lane < 8 ? call * (pushed ? 16u : 128u) : (pushed ? call * 112u : 0u);
            const int pace_unit = IN_UNITS + 16 * blk - PACE_LEAD;
            const unsigned tb0 = tick_base(call);
            const unsigned long long t0 = __builtin_amdgcn_s_memrealtime();
            unsigned spins = 0;
            for (;;) {
                unsigned v = 0;
                if (lane < 8 || lane == 16) v = __hip_atomic_load(wp, __ATOMIC_RELAXED, __HIP_MEMORY_SCOPE_AGENT);
                else if (lane < 16 && pushed) v = __hip_atomic_load(wp, __ATOMIC_RELAXED, __HIP_MEMORY_SCOPE_SYSTEM);
                const bool wait = lane < 16 ? v < want : (lane == 16 ? (int)(v - tb0) < pace_unit * 8 : false);
                if (__builtin_amdgcn_ballot_w64(wait) == 0ull) break;
                if ((++spins & 15u) == 0u && __builtin_amdgcn_readfirstlane((int)give_up(tmo, call, t0))) break;
                if (blk < 3) __builtin_amdgcn_s_sleep(2); else __builtin_amdgcn_s_sleep(16);
            }
            if (__builtin_amdgcn_ballot_w64(lane < 8 && pushed)) __builtin_amdgcn_fence(__ATOMIC_ACQUIRE, "");
            else
            __builtin_amdgcn_fence(__ATOMIC_ACQUIRE, "agent");
            WAIT_VM0();
        }
        CBAR(); __builtin_amdgcn_s_barrier(); CBAR();
        f32x4 acc[2][2][4][2];
        {
            const size_t koff = helper ? (size_t)KK : 0;
            gemm_tile(lds, wsl + OFF_XF + (size_t)tm * 256 * KK * 2 + koff, wsl + OFF_WT + (size_t)tn * 256 * KK * 2 + koff, split ? 32 : 64, acc, tid);
        }
        const rsrc_t rp = mkrsrc(wsl + OFF_PART + (size_t)(split ? m : 0) * 262144, split ? 262144u : 0u);
        {
            unsigned* const pf = (unsigned*)(wsl + OFF_PFLAG + (split ? m : 0) * 128);
            if (helper) {
#pragma unroll
                for (int j = 0; j < 32; ++j) __builtin_amdgcn_raw_buffer_store_b128(__builtin_bit_cast(u32x4, acc[j >> 4][(j >> 3) & 1][(j >> 1) & 3][j & 1]), rp, tid * 16, j * 8192, AUX_SYS);
                WAIT_VM0();
                __syncthreads();
                if (tid == 0) __hip_atomic_store(pf, call, __ATOMIC_RELAXED, __HIP_MEMORY_SCOPE_AGENT);
                continue;
            }
            if (split) {
                if (wid == 0) {
                    const unsigned long long t0 = __builtin_amdgcn_s_memrealtime();
                    for (unsigned spins = 1;; ++spins) {
                        if ((unsigned)uni((int)__hip_atomic_load(pf, __ATOMIC_RELAXED, __HIP_MEMORY_SCOPE_AGENT)) >= call) break;
                        if ((spins & 15u) == 0u && __builtin_amdgcn_readfirstlane((int)give_up(tmo, call, t0))) break;
                        __builtin_amdgcn_s_sleep(1);
                    }
                    __builtin_amdgcn_fence(__ATOMIC_ACQUIRE, "agent");
                    WAIT_VM0();
                }
                CBAR(); __builtin_amdgcn_s_barrier(); CBAR();
            }
        }
        dead = __hip_atomic_load(tmo, __ATOMIC_RELAXED, __HIP_MEMORY_SCOPE_AGENT) == call;
        {
            int tz = threadIdx.x; asm volatile("" : "+v"(tz));
            const int ln = tz & 63, wr = wid >> 2, wc = wid & 3, fr = ln & 15, fq = ln >> 4;
#pragma unroll
            for (int ai = 0; ai < 2; ++ai) {
                u32x4 pt[16];
#pragma unroll
                for (int j = 0; j < 16; ++j) pt[j] = __builtin_amdgcn_raw_buffer_load_b128(rp, tz * 16, (ai * 16 + j) * 8192, 0);
#pragma unroll
                for (int mm = 0; mm < 4; ++mm)
#pragma unroll
                    for (int bj = 0; bj < 2; ++bj) {
                        const f32x4 v0 = acc[ai][bj][mm][0] + __builtin_bit_cast(f32x4, pt[bj * 8 + mm * 2]), v1 = acc[ai][bj][mm][1] + __builtin_bit_cast(f32x4, pt[bj * 8 + mm * 2 + 1]);
                        u32x4 w4;
                        w4.x = gelu2(v0[0], v0[1]); w4.y = gelu2(v0[2], v0[3]);
                        w4.z = gelu2(v1[0], v1[1]); w4.w = gelu2(v1[2], v1[3]);
                        const int row = ai * 128 + wr * 64 + mm * 16 + fr, ch = bj * 16 + wc * 4 + fq;
                        *(LAS u32x4*)(lds + row * 512 + ((ch ^ (row & 7)) << 4)) = w4;
                    }
                CBAR();
            }
            __syncthreads();
            u32x4 d[16];
#pragma unroll
            for (int r = 0; r < 16; ++r) {
                const int row = wid * 32 + 2 * r + (ln >> 5), ch = ln & 31;
                d[r] = *(const LAS u32x4*)(lds + row * 512 + ((ch ^ (row & 7)) << 4));
            }
            if (dead) {
#pragma unroll
                for (int r = 0; r < 16; ++r) d[r] = (u32x4){0x7fc07fc0u, 0x7fc07fc0u, 0x7fc07fc0u, 0x7fc07fc0u};
            }
            const rsrc_t ro = mkrsrc(p.out, (unsigned)OUT_BYTES);
            const unsigned ob = (unsigned)(((size_t)(256 * tm + 32 * wid + (ln >> 5)) * NN + NS * rank + 256 * tn + 8 * (ln & 31)) * 2);
#pragma unroll
            for (int r = 0; r < 16; ++r) __builtin_amdgcn_raw_buffer_store_b128(d[r], ro, ob, r * (2 * NN * 2), AUX_LOC);
            const rsrc_t rws = mkrsrc(wsl, (unsigned)WS_USED);
            if (m >= OUT_UNITS - TAILT) {
#pragma unroll
                for (int r = 0; r < 16; ++r) __builtin_amdgcn_raw_buffer_store_b128(d[r], rws, op_off + (unsigned)(m * 8 + wid) * SLOT + RAW_OFF + ln * 16, r * 1024, AUX_SYS);
            } else encode_wb<true>(d, rws, op_off + (unsigned)(m * 8 + wid) * SLOT, lds, lds + wid * 16384, (LAS unsigned*)(lds + STAGE_BYTES + wid * 1024), ln);
            WAIT_VM0();
            __syncthreads();
            if (tid < NR && tid != rank)
                __hip_atomic_store((unsigned*)(p.ws[tid] + flag_row + (size_t)rank * NFLAG * 4) + (IN_UNITS * 8 + m), call, __ATOMIC_RELAXED, __HIP_MEMORY_SCOPE_SYSTEM);
        }
    }
}

}

extern "C" void kernel_launch(int rank, int nranks, ncclComm_t comm, hipStream_t stream,
                              void* const* d_in, const int* in_sizes, int n_in,
                              void* d_out, int out_size, const int* in_dtypes, int out_dtype,
                              void* const* d_out_all, void* const* d_ws_all, size_t ws_bytes) {
    (void)comm; (void)in_sizes; (void)n_in; (void)out_size; (void)in_dtypes; (void)out_dtype; (void)d_out_all;
    if (nranks != gk::NR || rank < 0 || rank >= gk::NR || ws_bytes < gk::WS_USED) { fprintf(stderr, "unexpected configuration\n"); return; }
    static bool attr_done[64] = {};
    int dev = 0; HIP_CHECK(hipGetDevice(&dev));
    if (dev >= 0 && dev < 64 && !attr_done[dev]) {
        HIP_CHECK(hipFuncSetAttribute((const void*)gk::fused_kernel, hipFuncAttributeMaxDynamicSharedMemorySize, gk::LDS_BYTES));
        attr_done[dev] = true;
    }
    gk::Params p;
    p.x = d_in[0]; p.w = d_in[1]; p.out = d_out;
    for (int i = 0; i < gk::NR; ++i) p.ws[i] = (char*)d_ws_all[i];
    p.rank = rank; p.pad = 0;
    hipLaunchKernelGGL(gk::fused_kernel, dim3(gk::GRID), dim3(gk::NTHR), gk::LDS_BYTES, stream, p);
    HIP_CHECK(hipGetLastError());
}
```
